```python
import numpy as np

kernel_name = "gpu_gemm_a2a_m4096_k4096_n4096_e4m3e4m3_dqrelu"


ABSMAX_RTOL = 0.03

COMPUTE_DTYPE = "float32"
EPILOGUE = "dequant_relu"
IN_QUANT = None
OUT_QUANT = None
IN_FP8 = "e4m3e4m3"
IN_DTYPES = {"x": "e4m3", "w_mat": "e4m3",
             "scale_x": "float32", "scale_w": "float32"}
WORLD_SIZE = 8

def _snap_fp8(x, mant_bits, max_abs, min_normal):
    a = np.abs(x.astype(np.float64))
    _m, _e = np.frexp(a)
    q = np.where(a >= min_normal, np.ldexp(1.0, _e - 1 - mant_bits),
                 min_normal * 2.0 ** -mant_bits)
    snapped = np.minimum(np.round(a / q) * q, max_abs)
    return (np.sign(x) * snapped).astype(np.float32)

def setup_inputs():

    x = _snap_fp8(np.random.randn(4096, 4096).astype(np.float32), 3, 448.0, 2.0 ** -6)
    w_mat = _snap_fp8(np.random.randn(4096, 4096).astype(np.float32), 3, 448.0, 2.0 ** -6)
    scale_x = (0.02 * (1.0 + np.random.rand(1))).astype(np.float32)
    scale_w = (0.02 * (1.0 + np.random.rand(1))).astype(np.float32)
    return {"x": x, "w_mat": w_mat, "scale_x": scale_x, "scale_w": scale_w}

def reference(x, w_mat, scale_x, scale_w):
    acc = (x @ w_mat).astype(np.float32)
    y = (acc * (scale_x * scale_w)).astype(np.float32)
    return np.maximum(y, 0.0).astype(np.float32)
```

```cpp
#include <hip/hip_runtime.h>
#include <rccl/rccl.h>
#include <cstdio>

#define HIP_CHECK(expr) do { hipError_t _e = (expr); if (_e != hipSuccess) { \
    fprintf(stderr, "HIP error %s at %s:%d\n", hipGetErrorString(_e), __FILE__, __LINE__); \
    return; } } while (0)

extern "C" __device__ unsigned llvm_writelane(unsigned, unsigned, unsigned) __asm("llvm.amdgcn.writelane.i32");

namespace {

typedef unsigned u32x4 __attribute__((ext_vector_type(4)));
typedef int v2i __attribute__((ext_vector_type(2)));
typedef int v4i __attribute__((ext_vector_type(4)));
typedef int v8i __attribute__((ext_vector_type(8)));
typedef float v4f __attribute__((ext_vector_type(4)));
typedef unsigned long long u64;
#define LAS __attribute__((address_space(3)))
#define GAS __attribute__((address_space(1)))

constexpr int NR = 8;
constexpr int MS = 512, KD = 4096, ND = 4096, NS = 512;
constexpr int DEPTH = 5, STAGE = 24576, NSTEP = 32;
constexpr int LDS_MISC = DEPTH * STAGE;
constexpr int LDS_BYTES = LDS_MISC + 64;
constexpr int HDR = 1040, HDR16 = HDR / 16;
constexpr int SLOT = 34048;
constexpr int P_HDR = 0, P_VAL = 2048;
constexpr size_t WS_CTR = 0;
constexpr size_t WS_TMO = 32768;
constexpr size_t WS_FLAG = 65536;
constexpr size_t WS_DBG = 262144;
constexpr size_t WS_IN = 1u << 20;
constexpr size_t WS_OUT = WS_IN + (size_t)2 * NR * 32 * SLOT;
constexpr size_t WS_END = WS_OUT + (size_t)2 * NR * 32 * SLOT;
constexpr u64 DEADLINE = 20000000ull * 2;

#define DELAY_B 400
#define PULL_B 0

struct Args {
  const unsigned char* x;
  const unsigned char* w;
  const float* sx;
  const float* sw;
  float* out;
  char* ws[NR];
  int rank;
  int pad;
};

__device__ __forceinline__ void glds(unsigned m0, unsigned voff, __amdgpu_buffer_rsrc_t rs, unsigned soff) {
  asm volatile("s_mov_b32 m0, %0\n\ts_nop 0\n\tbuffer_load_dwordx4 %1, %2, %3 offen lds" ::"s"(m0), "v"(voff), "s"(rs), "s"(soff) : "memory");
}
__device__ __forceinline__ void glds_sys(unsigned m0, const void* p) {
  asm volatile("s_mov_b32 m0, %0\n\ts_nop 0\n\tglobal_load_lds_dwordx4 %1, off sc0 sc1" ::"s"(m0), "v"(p) : "memory");
}
__device__ __forceinline__ void st16_sys(void* p, u32x4 v) {
  asm volatile("global_store_dwordx4 %0, %1, off sc0 sc1\n\ts_nop 1" ::"v"(p), "v"(v) : "memory");
}
template <int N> __device__ __forceinline__ void wait_vm() { asm volatile("s_waitcnt vmcnt(%0)" ::"i"(N) : "memory"); }
#define SB() __builtin_amdgcn_sched_barrier(0)
#define BAR() do { asm volatile("s_barrier" ::: "memory"); SB(); } while (0)
#define WAIT_LDS() do { __builtin_amdgcn_s_waitcnt(0xC07F); SB(); } while (0)
#define LBAR() do { asm volatile("s_waitcnt lgkmcnt(0)\n\ts_barrier" ::: "memory"); SB(); } while (0)

#define STAMP(i) do {} while (0)

struct Frag { v8i x[2]; v8i w[4]; };

__device__ __forceinline__ void kstep(v4f (&acc)[4][2], const Frag& c, Frag& n, bool mm, unsigned so, unsigned ax0, unsigned ax1, const unsigned (&aw)[4]) {
  v4i xa[2], xb[2]; v2i t[4][4];
#define RX(d, a) d = *(LAS v4i*)(size_t)(a)
#define RW(nb, u, off) t[nb][u] = __builtin_amdgcn_ds_read_tr8_b64_v2i32((LAS v2i*)(size_t)(aw[nb] + so + off))
#define MM(nb, mb) if (mm) acc[nb][mb] = __builtin_amdgcn_mfma_scale_f32_16x16x128_f8f6f4(c.w[nb], c.x[mb], acc[nb][mb], 0, 0, 0, 0, 0, 0)
  MM(0, 0); SB();
  RX(xa[0], ax0 + so); RX(xb[0], ax1 + so); RW(0, 0, 0); SB();
  MM(0, 1); SB();
  RW(0, 1, 1024); RW(0, 2, 8192); RW(0, 3, 9216); SB();
  MM(1, 0); SB();
  RX(xa[1], ax0 + so + 2048); RX(xb[1], ax1 + so + 2048); RW(1, 0, 0); SB();
  MM(1, 1); SB();
  RW(1, 1, 1024); RW(1, 2, 8192); RW(1, 3, 9216); SB();
  MM(2, 0); SB();
  RW(2, 0, 0); RW(2, 1, 1024); RW(2, 2, 8192); RW(2, 3, 9216); SB();
  MM(2, 1); SB();
  RW(3, 0, 0); RW(3, 1, 1024); RW(3, 2, 8192); RW(3, 3, 9216); SB();
  MM(3, 0); SB();
  MM(3, 1); SB();
#pragma unroll
  for (int mb = 0; mb < 2; ++mb) n.x[mb] = __builtin_shufflevector(xa[mb], xb[mb], 0, 1, 2, 3, 4, 5, 6, 7);
#pragma unroll
  for (int nb = 0; nb < 4; ++nb) n.w[nb] = v8i{t[nb][0][0], t[nb][0][1], t[nb][1][0], t[nb][1][1], t[nb][2][0], t[nb][2][1], t[nb][3][0], t[nb][3][1]};
  WAIT_LDS();
#undef RX
#undef RW
#undef MM
}

__global__ void __launch_bounds__(512) fused_gemm_a2a(Args a) {
  extern __shared__ __attribute__((aligned(16))) char lds[];
  const int tid = threadIdx.x, l = tid & 63;
  const int wv = __builtin_amdgcn_readfirstlane(tid >> 6);
  const int j = blockIdx.x & 7, idx = blockIdx.x >> 3, grp = idx >> 4, i16 = idx & 15;
  const int rb = i16 & 7, cb = (i16 >> 3) + 2 * grp, tile = rb * 4 + cb;
  const unsigned dst = (unsigned)(a.rank + j) & 7u, src = (unsigned)(a.rank - j) & 7u;
  const bool pull = grp == 1 && i16 < PULL_B;
  const unsigned ldsb = (unsigned)(size_t)lds;
  char* const me = a.ws[a.rank];
  LAS volatile unsigned* misc = (LAS volatile unsigned*)(size_t)(ldsb + LDS_MISC);
  STAMP(0);

  if (wv < 4) {
    const u64 t0 = __builtin_amdgcn_s_memrealtime();
    const __amdgpu_buffer_rsrc_t rsW = __builtin_amdgcn_make_buffer_rsrc((void*)a.w, 0, KD * ND, 0x00020000);
    const __amdgpu_buffer_rsrc_t rsX = __builtin_amdgcn_make_buffer_rsrc((void*)a.x, 0, MS * KD, 0x00020000);
    const int xr = 16 * wv + (l >> 3);
    const unsigned vx0 = (unsigned)(xr * KD + 16 * ((l & 7) ^ ((xr >> 1) & 7)));
    const unsigned vx1 = (unsigned)((xr + 8) * KD + 16 * ((l & 7) ^ (((xr + 8) >> 1) & 7)));
    const unsigned cw = (unsigned)((l & 7) ^ ((l >> 4) & 3));
    const unsigned vw0 = (unsigned)((l >> 3) * ND) + 16 * cw, vw1 = (unsigned)((l >> 3) * ND) + 16 * (cw ^ 4);
    const int koff = 4 * rb + 16 * (i16 >> 3) + j;
    const unsigned bx = (unsigned)(64 * rb * KD), bw = (unsigned)(32 * wv * ND) + NS * dst + 128 * cb;
    unsigned slot = ldsb;
    int st = 0;
    auto issue = [&]() {
      const unsigned kb = (unsigned)((st + koff) & 31);
      const unsigned sx = bx + 128 * kb, sw = bw + kb * (128 * ND);
      glds(slot + wv * 2048, vx0, rsX, sx); glds(slot + wv * 2048 + 1024, vx1, rsX, sx);
      const unsigned wb = slot + 8192 + wv * 4096;
      glds(wb, vw0, rsW, sw); glds(wb + 1024, vw0, rsW, sw + 8 * ND); glds(wb + 2048, vw1, rsW, sw + 16 * ND); glds(wb + 3072, vw1, rsW, sw + 24 * ND);
      ++st;
      slot += STAGE; if (slot == ldsb + DEPTH * STAGE) slot = ldsb;
    };
    if (grp == 1 && DELAY_B > 0) { while (__builtin_amdgcn_s_memrealtime() - t0 < (u64)DELAY_B) __builtin_amdgcn_s_sleep(1); }
#pragma unroll
    for (int i = 0; i < DEPTH - 1; ++i) issue();
#pragma unroll 1
    for (int s = 0; s < NSTEP - (DEPTH - 1); ++s) {
      wait_vm<(DEPTH - 2) * 6>();
      BAR();
      issue();
    }
    static_assert(DEPTH == 5, "tail below is written for DEPTH 5");
    wait_vm<18>(); BAR();
    wait_vm<12>(); BAR();
    wait_vm<6>(); BAR();
    wait_vm<0>(); BAR();
    BAR();
  } else {
    unsigned ticket = 0;
    if (tid == 256) {
      GAS unsigned* ctr = (GAS unsigned*)(me + WS_CTR + blockIdx.x * 128);
      asm volatile("" : "+v"(ctr));
      ticket = __hip_atomic_fetch_add(ctr, 1u, __ATOMIC_RELAXED, __HIP_MEMORY_SCOPE_AGENT);
    }
    const float scale = a.sx[0] * a.sw[0];
    const int pw = wv - 4, mh = pw & 1, nh = pw >> 1;
    const int g = l >> 4, q = (l & 15) >> 1, p = l & 1;
    const unsigned f = (l & 15) >> 1;
    const unsigned ax0 = ldsb + (32 * mh + (l & 15)) * 128 + ((g ^ f) * 16), ax1 = ldsb + (32 * mh + (l & 15)) * 128 + ((g ^ f ^ 4) * 16);
    const unsigned hW = ((q >> 1) & 3) | ((g & 1) << 2);
    unsigned aw[4];
#pragma unroll
    for (int nb = 0; nb < 4; ++nb) aw[nb] = ldsb + 8192 + (16 * g + q) * 128 + 8 * p + (((4 * nh + nb) ^ hW) * 16);
    unsigned so = 0;
    v4f acc[4][2] = {};
    Frag fa, fb;
    BAR();
    STAMP(1);
    kstep(acc, fb, fa, false, so, ax0, ax1, aw);
    so += STAGE;
#pragma unroll 1
    for (int s = 1; s < 31; s += 2) {
      BAR();
      kstep(acc, fa, fb, true, so, ax0, ax1, aw);
      so += STAGE; if (so == DEPTH * STAGE) so = 0;
      BAR();
      kstep(acc, fb, fa, true, so, ax0, ax1, aw);
      so += STAGE; if (so == DEPTH * STAGE) so = 0;
    }
    BAR();
    kstep(acc, fa, fb, true, so, ax0, ax1, aw);
    BAR();
#pragma unroll
    for (int nb = 0; nb < 4; ++nb)
#pragma unroll
      for (int mb = 0; mb < 2; ++mb) acc[nb][mb] = __builtin_amdgcn_mfma_scale_f32_16x16x128_f8f6f4(fb.w[nb], fb.x[mb], acc[nb][mb], 0, 0, 0, 0, 0, 0);
    STAMP(2);
    if (tid == 256) misc[0] = ticket + 1;
    if (j == 0) {
#pragma unroll
      for (int nb = 0; nb < 4; ++nb)
#pragma unroll
        for (int mb = 0; mb < 2; ++mb) {
          v4f y = acc[nb][mb] * scale;
#pragma unroll
          for (int e = 0; e < 4; ++e) y[e] = y[e] <= 0.f ? 0.f : y[e];
          float* o = a.out + (size_t)(MS * a.rank + 64 * rb + 32 * mh + 16 * mb + (l & 15)) * NS + 128 * cb + 64 * nh + 16 * nb + 4 * g;
          st16_sys(o, __builtin_bit_cast(u32x4, y));
        }
    } else {
#pragma unroll
      for (int h = 0; h < 2; ++h) {
        const unsigned pb = ldsb + P_VAL + (2 * pw + h) * 4096;
        unsigned cnt = 0;
        unsigned mlo = 0, mhi = 0;
#pragma unroll
        for (int r = 0; r < 16; ++r) {
          const float y = acc[2 * h + (r >> 3)][(r >> 2) & 1][r & 3] * scale;
          const bool nz = !(y <= 0.f);
          const u64 m = __builtin_amdgcn_ballot_w64(nz);
          const unsigned pos = __builtin_amdgcn_mbcnt_hi((unsigned)(m >> 32), __builtin_amdgcn_mbcnt_lo((unsigned)m, cnt));
          if (nz) *(LAS float*)(size_t)(pb + 4 * pos) = y;
          cnt += __builtin_popcountll(m);
          mlo = llvm_writelane((unsigned)m, r, mlo);
          mhi = llvm_writelane((unsigned)(m >> 32), r, mhi);
        }
        if (l < 16) *(LAS v2i*)(size_t)(ldsb + P_HDR + ((2 * pw + h) * 16 + l) * 8) = v2i{(int)mlo, (int)mhi};
        if (l == 0) *(LAS unsigned short*)(size_t)(ldsb + P_HDR + 1024 + (2 * pw + h) * 2) = (unsigned short)cnt;
      }
    }
  }
  if (j == 0) return;
  LBAR();
  STAMP(3);
  const unsigned call = misc[0];
  const unsigned par = call & 1u;
  {
    const u32x4 cv = *(LAS u32x4*)(size_t)(ldsb + P_HDR + 1024);
    unsigned pre[9];
    pre[0] = HDR16;
#pragma unroll
    for (int i = 0; i < 8; ++i) pre[i + 1] = pre[i] + ((((cv[i >> 1] >> (16 * (i & 1))) & 0xffffu) + 3u) >> 2);
    const unsigned total = __builtin_amdgcn_readfirstlane(pre[8]);
    char* base = pull ? me + WS_OUT + ((size_t)(par * NR + dst) * 32 + tile) * SLOT
                      : a.ws[dst] + WS_IN + ((size_t)(par * NR + a.rank) * 32 + tile) * SLOT;
    for (unsigned c = tid; c < total; c += 512) {
      unsigned la = P_HDR + 16 * c;
#pragma unroll
      for (int i = 0; i < 8; ++i) la = c >= pre[i] ? P_VAL + i * 4096 + 16 * (c - pre[i]) : la;
      const u32x4 v = *(LAS u32x4*)(size_t)(ldsb + la);
      st16_sys(base + 16 * (size_t)c, v);
    }
    STAMP(4);
    wait_vm<0>();
    LBAR();
    if (tid == 0)
      __hip_atomic_store((GAS u64*)(a.ws[dst] + WS_FLAG + ((size_t)(par * NR + a.rank) * 32 + tile) * 128), ((u64)call << 32) | total,
                         __ATOMIC_RELAXED, __HIP_MEMORY_SCOPE_SYSTEM);
  }
  STAMP(5);
  {
    GAS unsigned* tmo = (GAS unsigned*)(me + WS_TMO);
    if (tid == 0) {
      GAS u64* flag = (GAS u64*)(me + WS_FLAG + ((size_t)(par * NR + src) * 32 + tile) * 128);
      const u64 t0 = __builtin_amdgcn_s_memrealtime();
      u64 fv;
      while ((unsigned)((fv = __hip_atomic_load(flag, __ATOMIC_RELAXED, __HIP_MEMORY_SCOPE_SYSTEM)) >> 32) < call) {
        if (__hip_atomic_load(tmo, __ATOMIC_RELAXED, __HIP_MEMORY_SCOPE_AGENT) == call || __builtin_amdgcn_s_memrealtime() - t0 > DEADLINE) {
          __hip_atomic_store(tmo, call, __ATOMIC_RELAXED, __HIP_MEMORY_SCOPE_AGENT);
          break;
        }
        __builtin_amdgcn_s_sleep(1);
      }
      const bool bad = __hip_atomic_load(tmo, __ATOMIC_RELAXED, __HIP_MEMORY_SCOPE_AGENT) == call;
      unsigned n = (unsigned)fv;
      if (n > (unsigned)(HDR16 + 2048)) n = HDR16 + 2048;
      misc[1] = bad ? 0u : n;
    }
    LBAR();
    STAMP(6);
    const unsigned total = __builtin_amdgcn_readfirstlane(misc[1]);
    const char* base = pull ? a.ws[src] + WS_OUT + ((size_t)(par * NR + a.rank) * 32 + tile) * SLOT
                            : me + WS_IN + ((size_t)(par * NR + src) * 32 + tile) * SLOT;
    for (unsigned c0 = 64 * wv; c0 < total; c0 += 512)
      if (c0 + l < total) glds_sys(ldsb + 16 * c0, base + 16 * (size_t)(c0 + l));
    wait_vm<0>();
    BAR();
    const int pw = wv >> 1, h = wv & 1, mh = pw & 1, nh = pw >> 1, g = l >> 4;
    float* o = a.out + (size_t)(MS * src + 64 * rb + 32 * mh + (l & 15)) * NS + 128 * cb + 64 * nh + 32 * h + 4 * g;
    if (total == 0) {
      const unsigned qn = 0x7fc00000u;
#pragma unroll
      for (int i = 0; i < 4; ++i) st16_sys(o + (i & 1) * 16 * NS + (i >> 1) * 16, u32x4{qn, qn, qn, qn});
      return;
    }
    const u32x4 cv = *(LAS u32x4*)(size_t)(ldsb + 1024);
    unsigned pre = HDR16;
#pragma unroll
    for (int i = 0; i < 7; ++i) pre += i < wv ? ((((cv[i >> 1] >> (16 * (i & 1))) & 0xffffu) + 3u) >> 2) : 0u;
    unsigned vb = __builtin_amdgcn_readfirstlane(ldsb + 16 * pre);
    const v2i mk = *(LAS v2i*)(size_t)(ldsb + (wv * 16 + (l & 15)) * 8);
    unsigned cnt = 0;
#pragma unroll
    for (int i = 0; i < 4; ++i) {
      u32x4 y;
#pragma unroll
      for (int e = 0; e < 4; ++e) {
        const unsigned lo = __builtin_amdgcn_readlane(mk[0], 4 * i + e), hi = __builtin_amdgcn_readlane(mk[1], 4 * i + e);
        const u64 m = ((u64)hi << 32) | lo;
        const unsigned pos = __builtin_amdgcn_mbcnt_hi(hi, __builtin_amdgcn_mbcnt_lo(lo, cnt));
        const unsigned v = *(LAS unsigned*)(size_t)(vb + 4 * pos);
        y[e] = __builtin_amdgcn_inverse_ballot_w64(m) ? v : 0u;
        cnt += __builtin_popcountll(m);
      }
      st16_sys(o + (i & 1) * 16 * NS + (i >> 1) * 16, y);
    }
    STAMP(7);
  }
}

}

extern "C" void kernel_launch(int rank, int nranks, ncclComm_t comm,
                              hipStream_t stream,
                              void* const* d_in, const int* in_sizes, int n_in,
                              void* d_out, int out_size,
                              const int* in_dtypes, int out_dtype,
                              void* const* d_out_all,
                              void* const* d_ws_all, size_t ws_bytes) {
  if (nranks != NR || (unsigned)rank >= (unsigned)NR || n_in != 4 || in_sizes[0] != MS * KD || in_sizes[1] != KD * ND ||
      out_size != NR * MS * NS || ws_bytes < WS_END) {
    fprintf(stderr, "kernel_launch: unexpected problem shape\n");
    return;
  }
  HIP_CHECK(hipFuncSetAttribute((const void*)fused_gemm_a2a, hipFuncAttributeMaxDynamicSharedMemorySize, LDS_BYTES));
  Args a;
  a.x = (const unsigned char*)d_in[0];
  a.w = (const unsigned char*)d_in[1];
  a.sx = (const float*)d_in[2];
  a.sw = (const float*)d_in[3];
  a.out = (float*)d_out;
  for (int p = 0; p < NR; ++p) a.ws[p] = (char*)d_ws_all[p];
  a.rank = rank;
  a.pad = 0;
  hipLaunchKernelGGL(fused_gemm_a2a, dim3(256), dim3(512), LDS_BYTES, stream, a);
  HIP_CHECK(hipGetLastError());
}
```

```cpp
#include <hip/hip_runtime.h>
#include <rccl/rccl.h>
#include <cstdio>

#define HIP_CHECK(expr) do { hipError_t _e = (expr); if (_e != hipSuccess) { \
    fprintf(stderr, "HIP error %s at %s:%d\n", hipGetErrorString(_e), __FILE__, __LINE__); \
    return; } } while (0)

namespace {

typedef unsigned u32x4 __attribute__((ext_vector_type(4)));
typedef int v2i __attribute__((ext_vector_type(2)));
typedef int v4i __attribute__((ext_vector_type(4)));
typedef int v8i __attribute__((ext_vector_type(8)));
typedef float v4f __attribute__((ext_vector_type(4)));
typedef unsigned long long u64;
#define LAS __attribute__((address_space(3)))
#define GAS __attribute__((address_space(1)))

constexpr int NR = 8;
constexpr int MS = 512, KD = 4096, ND = 4096, NS = 512;
constexpr int DEPTH = 5, STAGE = 24576, NSTEP = 32;
constexpr int LDS_MISC = DEPTH * STAGE;
constexpr int LDS_BYTES = LDS_MISC + 128 + 1024;
constexpr int DENSE = 36992;
constexpr int SLOT = 1024 + DENSE;
constexpr int L_TX = 0, L_RX = DENSE, L_TXM = 2 * DENSE, L_HAND = L_TXM + 1024;
static_assert(L_HAND + 16384 <= DEPTH * STAGE && SLOT % 128 == 0, "");
constexpr size_t WS_CTR = 0;
constexpr size_t WS_TMO = 32768;
constexpr size_t WS_FLAG = 65536;
constexpr size_t WS_DBG = 524288;
constexpr size_t WS_IN = 1u << 20;
constexpr size_t WS_OUT = WS_IN + (size_t)2 * NR * 32 * SLOT;
constexpr size_t WS_END = WS_OUT + (size_t)2 * NR * 32 * SLOT;
constexpr u64 DEADLINE = 20000000ull * 2;

#define T1 0
#define T2 0
#define T3 0
#define TR 0
#define PF 3
#define PF_DEPTH 3
#define NPUSH 24
#define PUSH_DEPTH 2
#define PULL_LANES 64
#define PULL_DEPTH 1
#define LOCAL_DELAY 0

#define LOOP_ALIGN [[clang::code_align(64)]]

#define POLL_GAP "2"

struct Args {
  const unsigned char* x;
  const unsigned char* w;
  const float* sx;
  const float* sw;
  float* out;
  char* ws[NR];
  int rank;
  int pad;
};

__device__ __forceinline__ void glds(unsigned m0, unsigned voff, __amdgpu_buffer_rsrc_t rs, unsigned soff) {
  asm volatile("s_mov_b32 m0, %0\n\ts_nop 0\n\tbuffer_load_dwordx4 %1, %2, %3 offen lds" ::"s"(m0), "v"(voff), "s"(rs), "s"(soff) : "memory");
}
__device__ __forceinline__ void glds_nt(unsigned m0, unsigned voff, __amdgpu_buffer_rsrc_t rs, unsigned soff) {
  asm volatile("s_mov_b32 m0, %0\n\ts_nop 0\n\tbuffer_load_dwordx4 %1, %2, %3 offen nt lds" ::"s"(m0), "v"(voff), "s"(rs), "s"(soff) : "memory");
}
#define WNT 0
#define GLDS_W glds
#define GLDS_PW glds
__device__ __forceinline__ void glds_sys(unsigned m0, const void* p) {
  asm volatile("s_mov_b32 m0, %0\n\ts_nop 0\n\tglobal_load_lds_dwordx4 %1, off sc0 sc1" ::"s"(m0), "v"(p) : "memory");
}
__device__ __forceinline__ void st16_sys(void* p, u32x4 v) {
  asm volatile("global_store_dwordx4 %0, %1, off sc0 sc1\n\ts_nop 1" ::"v"(p), "v"(v) : "memory");
}
template <int OFF> __device__ __forceinline__ void bst16_sys(__amdgpu_buffer_rsrc_t rs, unsigned voff, unsigned soff, u32x4 v) {
  asm volatile("buffer_store_dwordx4 %0, %1, %2, %3 offen offset:%4 sc0 sc1\n\ts_nop 1" ::"v"(v), "v"(voff), "s"(rs), "s"(soff), "i"(OFF) : "memory");
}
template <int OFF> __device__ __forceinline__ void bglds_sys(unsigned m0, __amdgpu_buffer_rsrc_t rs, unsigned voff, unsigned soff) {
  asm volatile("s_mov_b32 m0, %0\n\ts_nop 0\n\tbuffer_load_dwordx4 %1, %2, %3 offen offset:%4 sc0 sc1 lds" ::"s"(m0), "v"(voff), "s"(rs), "s"(soff), "i"(OFF) : "memory");
}
template <int N> __device__ __forceinline__ void wait_vm() { asm volatile("s_waitcnt vmcnt(%0)" ::"i"(N) : "memory"); }
#define SB() __builtin_amdgcn_sched_barrier(0)
#define BAR() do { asm volatile("s_barrier" ::: "memory"); SB(); } while (0)
#define WAIT_LDS() do { __builtin_amdgcn_s_waitcnt(0xC07F); SB(); } while (0)
#define LBAR() do { asm volatile("s_waitcnt lgkmcnt(0)\n\ts_barrier" ::: "memory"); SB(); } while (0)

#define STAMP(i) do {} while (0)
#define STAMP_FLUSH() do {} while (0)

struct Frag { v8i x[2]; v8i w[4]; };

__device__ __forceinline__ void kstep(v4f (&acc)[4][2], const Frag& c, Frag& n, bool mm, unsigned so, unsigned ax0, unsigned ax1, const unsigned (&aw)[4]) {
  v4i xa[2], xb[2]; v2i t[4][4];
#define RX(d, a) d = *(LAS v4i*)(size_t)(a)
#define RW(nb, u, off) t[nb][u] = __builtin_amdgcn_ds_read_tr8_b64_v2i32((LAS v2i*)(size_t)(aw[nb] + so + off))
#define MM(nb, mb) if (mm) acc[nb][mb] = __builtin_amdgcn_mfma_scale_f32_16x16x128_f8f6f4(c.w[nb], c.x[mb], acc[nb][mb], 0, 0, 0, 0, 0, 0)
  MM(0, 0); SB();
  RX(xa[0], ax0 + so); RX(xb[0], ax1 + so); RW(0, 0, 0); SB();
  MM(0, 1); SB();
  RW(0, 1, 1024); RW(0, 2, 8192); RW(0, 3, 9216); SB();
  MM(1, 0); SB();
  RX(xa[1], ax0 + so + 2048); RX(xb[1], ax1 + so + 2048); RW(1, 0, 0); SB();
  MM(1, 1); SB();
  RW(1, 1, 1024); RW(1, 2, 8192); RW(1, 3, 9216); SB();
  MM(2, 0); SB();
  RW(2, 0, 0); RW(2, 1, 1024); RW(2, 2, 8192); RW(2, 3, 9216); SB();
  MM(2, 1); SB();
  RW(3, 0, 0); RW(3, 1, 1024); RW(3, 2, 8192); RW(3, 3, 9216); SB();
  MM(3, 0); SB();
  MM(3, 1); SB();
#pragma unroll
  for (int mb = 0; mb < 2; ++mb) n.x[mb] = __builtin_shufflevector(xa[mb], xb[mb], 0, 1, 2, 3, 4, 5, 6, 7);
#pragma unroll
  for (int nb = 0; nb < 4; ++nb) n.w[nb] = v8i{t[nb][0][0], t[nb][0][1], t[nb][1][0], t[nb][1][1], t[nb][2][0], t[nb][2][1], t[nb][3][0], t[nb][3][1]};
  WAIT_LDS();
#undef RX
#undef RW
#undef MM
}

__device__ __forceinline__ unsigned lds_u32(unsigned a) { return *(LAS volatile unsigned*)(size_t)a; }
__device__ __forceinline__ unsigned rfl(unsigned v) { return __builtin_amdgcn_readfirstlane(v); }
template <int CTRL, int ROWS> __device__ __forceinline__ unsigned dpp0(unsigned v) { return __builtin_amdgcn_update_dpp(0u, v, CTRL, ROWS, 0xf, true); }
__device__ __forceinline__ unsigned wave_scan(unsigned v) {
  v += dpp0<0x111, 0xf>(v); v += dpp0<0x112, 0xf>(v); v += dpp0<0x114, 0xf>(v); v += dpp0<0x118, 0xf>(v);
  v += dpp0<0x142, 0xa>(v); v += dpp0<0x143, 0xc>(v);
  return v;
}
__device__ __forceinline__ unsigned wave_max(unsigned v) {
  v = max(v, dpp0<0x111, 0xf>(v)); v = max(v, dpp0<0x112, 0xf>(v)); v = max(v, dpp0<0x114, 0xf>(v)); v = max(v, dpp0<0x118, 0xf>(v));
  v = max(v, dpp0<0x142, 0xa>(v)); v = max(v, dpp0<0x143, 0xc>(v));
  return v;
}
__device__ __forceinline__ unsigned mbcnt64(u64 m) { return __builtin_amdgcn_mbcnt_hi((unsigned)(m >> 32), __builtin_amdgcn_mbcnt_lo((unsigned)m, 0u)); }
__device__ __forceinline__ void push_bits(unsigned (&b)[4], u64 m0, u64 m1, u64 m2, u64 m3) {
  asm("s_nop 1\n\tv_addc_co_u32_e64 %0, vcc, %0, %0, %4\n\tv_addc_co_u32_e64 %1, vcc, %1, %1, %5\n\t"
      "v_addc_co_u32_e64 %2, vcc, %2, %2, %6\n\tv_addc_co_u32_e64 %3, vcc, %3, %3, %7"
      : "+v"(b[0]), "+v"(b[1]), "+v"(b[2]), "+v"(b[3]) : "s"(m0), "s"(m1), "s"(m2), "s"(m3) : "vcc");
}

__device__ __forceinline__ unsigned wait_flag(GAS u64* flag, GAS unsigned* tmo, unsigned call, unsigned lds_word) {
  unsigned n, hi;
#define POLL(f) "global_load_dwordx2 " f ", %[flag], off sc0 sc1\n\t"
#define TEST(f0, f1) "s_waitcnt vmcnt(4)\n\tv_readfirstlane_b32 %[hi], " f1 "\n\tv_readfirstlane_b32 %[n], " f0 "\n\ts_cmp_ge_u32 %[hi], %[call]\n\ts_cbranch_scc1 2f\n\t"
  asm volatile("s_memrealtime s[96:97]\n\t"
               POLL("v[100:101]") "s_sleep " POLL_GAP "\n\t" POLL("v[102:103]") "s_sleep " POLL_GAP "\n\t" POLL("v[104:105]") "s_sleep " POLL_GAP "\n\t" POLL("v[106:107]")
               "global_load_dword v108, %[tmo], off sc1\n"
               "1:\n\t"
               TEST("v100", "v101") POLL("v[100:101]") TEST("v102", "v103") POLL("v[102:103]")
               TEST("v104", "v105") POLL("v[104:105]") TEST("v106", "v107") POLL("v[106:107]")
               "s_memrealtime s[98:99]\n\ts_waitcnt vmcnt(4) lgkmcnt(0)\n\tv_readfirstlane_b32 %[hi], v108\n\t"
               "global_load_dword v108, %[tmo], off sc1\n\t"
               "s_sub_u32 %[n], s98, s96\n\ts_cmp_gt_u32 %[n], %[dl]\n\ts_cselect_b32 %[hi], %[call], %[hi]\n\t"
               "s_cmp_lg_u32 %[hi], %[call]\n\ts_cbranch_scc1 1b\n\t"
               "s_mov_b32 %[n], 0\n"
               "2:\n\t"
               "v_mov_b32 v109, %[lw]\n\tv_mov_b32 v110, %[n]\n\tds_write_b32 v109, v110\n\t"
               "s_waitcnt lgkmcnt(0)\n\ts_barrier\n\ts_waitcnt vmcnt(0)"
               : [n] "=&s"(n), [hi] "=&s"(hi)
               : [flag] "v"(flag), [tmo] "v"(tmo), [call] "s"(call), [dl] "s"((unsigned)DEADLINE), [lw] "s"(lds_word)
               : "memory", "scc", "s96", "s97", "s98", "s99", "v100", "v101", "v102", "v103", "v104", "v105", "v106", "v107", "v108", "v109", "v110");
#undef POLL
#undef TEST
  return n;
}

__global__ void __launch_bounds__(512) fused_gemm_a2a(Args a) {
  extern __shared__ __attribute__((aligned(16))) char lds[];
  const int tid = threadIdx.x, l = tid & 63;
  const int wv = __builtin_amdgcn_readfirstlane(tid >> 6);
  const int j = blockIdx.x & 7, idx = blockIdx.x >> 3;
  const int rb = idx & 7, cb = idx >> 3;
  const unsigned dst = (unsigned)(a.rank + j) & 7u, src = (unsigned)(a.rank - j) & 7u;
  const bool pull = idx >= NPUSH;
  const unsigned ldsb = (unsigned)(size_t)lds;
  char* const me = a.ws[a.rank];
  char* const peer = a.ws[dst];
  STAMP(0);
  const u64 t0 = __builtin_amdgcn_s_memrealtime();
  const __amdgpu_buffer_rsrc_t rsW = __builtin_amdgcn_make_buffer_rsrc((void*)a.w, 0, KD * ND, 0x00020000);
  const __amdgpu_buffer_rsrc_t rsX = __builtin_amdgcn_make_buffer_rsrc((void*)a.x, 0, MS * KD, 0x00020000);
  const int xdelay = j == 0 ? LOCAL_DELAY : 0;
  if (cb >= 2 && PF) {
    const unsigned pvx = (unsigned)((l >> 3) * KD + 16 * (l & 7)), pvw = (unsigned)((l >> 4) * ND + 16 * (l & 15));
    const int id = (idx - 16) * 8 + wv;
    const unsigned ox0 = (unsigned)(8 * (id & 63) * KD + 128 * (id >> 6)), ow = (unsigned)(4 * id * ND) + NS * dst;
#pragma unroll 1
    for (int t = 0; t < 8; ++t) {
      glds(ldsb + LDS_MISC + 128, pvx, rsX, ox0 + 512 * t);
      GLDS_PW(ldsb + LDS_MISC + 128, pvw, rsW, ow + (unsigned)t * (512 * ND));
      glds(ldsb + LDS_MISC + 128, pvx, rsX, ox0 + 512 * t + 256);
      wait_vm<PF_DEPTH>();
    }
    if (PF != 3 && (cb == 3 || PF == 2)) {
      const int n2 = PF == 2 ? 128 : 64;
#pragma unroll 1
      for (int t = id & (n2 - 1); t < 1024; t += n2) {
        GLDS_PW(ldsb + LDS_MISC + 128, pvw, rsW, (unsigned)(4 * t * ND) + NS * dst + 256);
        wait_vm<PF_DEPTH>();
      }
    }
  }
  LAS volatile unsigned* misc = (LAS volatile unsigned*)(size_t)(ldsb + LDS_MISC);

  v4f v[4];
  unsigned call, us, upb;
  int sh;
  bool plain;
  auto setup = [&](unsigned m0, unsigned m2, unsigned m4) {
    call = rfl(m0) + 1; us = rfl(m2);
    sh = 24 - ((int)rfl(m4) - 126);
    sh = sh > 18 ? 18 : sh;
    const int ec = (int)((us >> 23) & 0xffu);
    plain = sh < 10 || ec == 0xff || ec - (sh + 8) < 1;
    upb = plain ? us : (us & 0x80000000u) | (unsigned)(127 + 8 + sh) << 23;
  };

  if (wv < 4) {
    const int xr = 16 * wv + (l >> 3);
    const unsigned vx0 = (unsigned)(xr * KD + 16 * ((l & 7) ^ ((xr >> 1) & 7)));
    const unsigned vx1 = (unsigned)((xr + 8) * KD + 16 * ((l & 7) ^ (((xr + 8) >> 1) & 7))) - 1024;
    const unsigned cw = (unsigned)((l & 7) ^ ((l >> 4) & 3));
    const unsigned vw0 = (unsigned)((l >> 3) * ND) + 16 * cw, vw1 = vw0 + 8 * ND - 1024;
    const unsigned vw2 = (unsigned)((l >> 3) * ND) + 16 * (cw ^ 4) + 16 * ND - 2048, vw3 = vw2 + 8 * ND - 1024;
    const unsigned k0 = (PF == 3 && cb >= 2) ? 4 * rb : 0;
    unsigned left = 32 - k0;
    unsigned sx = (unsigned)(64 * rb * KD) + 128 * k0, sw = (unsigned)(32 * wv * ND) + NS * dst + 128 * cb + k0 * (128 * ND);
    unsigned mx = ldsb + wv * 2048;
    const unsigned dmw = 8192 + wv * 2048, mend = mx + DEPTH * STAGE;
    auto issue = [&]() {
      asm volatile("s_mov_b32 m0, %0\n\ts_nop 0\n\tbuffer_load_dwordx4 %4, %10, %1 offen lds\n\tbuffer_load_dwordx4 %5, %10, %1 offen offset:1024 lds\n\t"
                   "s_add_u32 m0, %0, %3\n\ts_addk_i32 %0, 0x6000\n\t"
                   "buffer_load_dwordx4 %6, %11, %2 offen lds\n\tbuffer_load_dwordx4 %7, %11, %2 offen offset:1024 lds\n\t"
                   "buffer_load_dwordx4 %8, %11, %2 offen offset:2048 lds\n\tbuffer_load_dwordx4 %9, %11, %2 offen offset:3072 lds\n\t"
                   "s_addk_i32 %1, 0x80\n\ts_add_u32 %2, %2, 0x80000"
                   : "+s"(mx), "+s"(sx), "+s"(sw)
                   : "s"(dmw), "v"(vx0), "v"(vx1), "v"(vw0), "v"(vw1), "v"(vw2), "v"(vw3), "s"(rsX), "s"(rsW) : "memory", "scc");
      if (mx == mend) mx -= DEPTH * STAGE;
      if (PF == 3 && --left == 0) { sx -= KD; sw -= (unsigned)KD * ND; }
    };
    static_assert(STAGE == 0x6000 && 128 * ND == 0x80000, "");
    const int delay = (cb == 0 ? 0 : cb == 1 ? T1 : cb == 2 ? T2 : T3) + rb * TR + xdelay;
    while ((int)(__builtin_amdgcn_s_memrealtime() - t0) < delay) __builtin_amdgcn_s_sleep(1);
    STAMP(12);
#pragma unroll
    for (int i = 0; i < DEPTH - 1; ++i) issue();
    STAMP(13);
#pragma unroll 1
    LOOP_ALIGN for (int s = 0; s < NSTEP - (DEPTH - 1); ++s) {
      wait_vm<(DEPTH - 2) * 6>();
      BAR();
      issue();
    }
    static_assert(DEPTH == 5, "tail below is written for DEPTH 5");
    wait_vm<18>(); BAR();
    wait_vm<12>(); BAR();
    wait_vm<6>(); BAR();
    setup(misc[0], misc[2], misc[4 + wv]);
    asm volatile("" : "+s"(call), "+s"(us), "+s"(upb), "+s"(sh));
    wait_vm<0>(); BAR();
    BAR();
    if (j == 0) { STAMP_FLUSH(); return; }
    BAR();
#pragma unroll
    for (int i = 0; i < 4; ++i) v[i] = *(LAS v4f*)(size_t)(ldsb + L_HAND + wv * 4096 + i * 1024 + 16 * l);
  } else {
    unsigned ticket = 0;
    if (tid == 256) {
      GAS unsigned* ctr = (GAS unsigned*)(me + WS_CTR + blockIdx.x * 128);
      asm volatile("" : "+v"(ctr));
      ticket = __hip_atomic_fetch_add(ctr, 1u, __ATOMIC_RELAXED, __HIP_MEMORY_SCOPE_AGENT);
    }
    float sxv = a.sx[0], swv = a.sw[0];
    const int pw = wv - 4, mh = pw & 1, nh = pw >> 1;
    const int g = l >> 4, q = (l & 15) >> 1, p = l & 1;
    const unsigned f = (l & 15) >> 1;
    const unsigned ax0 = ldsb + (32 * mh + (l & 15)) * 128 + ((g ^ f) * 16), ax1 = ldsb + (32 * mh + (l & 15)) * 128 + ((g ^ f ^ 4) * 16);
    const unsigned hW = ((q >> 1) & 3) | ((g & 1) << 2);
    unsigned aw[4];
#pragma unroll
    for (int nb = 0; nb < 4; ++nb) aw[nb] = ldsb + 8192 + (16 * g + q) * 128 + 8 * p + (((4 * nh + nb) ^ hW) * 16);
    v4f acc[4][2] = {};
    unsigned so = 0;
    Frag fa, fb;
    BAR();
    STAMP(1);
    kstep(acc, fb, fa, false, so, ax0, ax1, aw);
    so += STAGE;
#pragma unroll 1
    LOOP_ALIGN for (int s = 1; s < 31; s += 2) {
      BAR();
      if (s == 23) {
        float mx = 0.f;
#pragma unroll
        for (int i = 0; i < 32; i += 2) mx = fmaxf(fmaxf(mx, fabsf(acc[i >> 3][(i >> 2) & 1][i & 3])), fabsf(acc[i >> 3][(i >> 2) & 1][(i & 3) + 1]));
        const unsigned mu = wave_max(__float_as_uint(mx));
        if (l == 63) misc[4 + pw] = mu >> 23;
        if (tid == 256) { misc[0] = ticket; misc[2] = __float_as_uint(sxv * swv); misc[3] = 0; }
        SB();
      }
      kstep(acc, fa, fb, true, so, ax0, ax1, aw);
      so += STAGE; if (so == DEPTH * STAGE) so = 0;
      BAR();
      kstep(acc, fb, fa, true, so, ax0, ax1, aw);
      so += STAGE; if (so == DEPTH * STAGE) so = 0;
    }
    BAR();
    const unsigned m0 = misc[0], m2 = misc[2], m4 = misc[4 + pw];
    kstep(acc, fa, fb, true, so, ax0, ax1, aw);
    BAR();
#pragma unroll
    for (int nb = 3; nb >= 0; --nb)
#pragma unroll
      for (int mb = 0; mb < 2; ++mb) acc[nb][mb] = __builtin_amdgcn_mfma_scale_f32_16x16x128_f8f6f4(fb.w[nb], fb.x[mb], acc[nb][mb], 0, 0, 0, 0, 0, 0);
    setup(m0, m2, m4);
    STAMP(2);
    if (j == 0) {
      const float scale = sxv * swv;
#pragma unroll
      for (int nb = 0; nb < 4; ++nb)
#pragma unroll
        for (int mb = 0; mb < 2; ++mb) {
          v4f y = acc[nb][mb] * scale;
#pragma unroll
          for (int e = 0; e < 4; ++e) y[e] = y[e] <= 0.f ? 0.f : y[e];
          float* o = a.out + (size_t)(MS * a.rank + 64 * rb + 32 * mh + 16 * mb + (l & 15)) * NS + 128 * cb + 64 * nh + 16 * nb + 4 * g;
          st16_sys(o, __builtin_bit_cast(u32x4, y));
        }
      STAMP_FLUSH();
      return;
    }
    SB();
#pragma unroll
    for (int i = 0; i < 4; ++i) *(LAS v4f*)(size_t)(ldsb + L_HAND + pw * 4096 + i * 1024 + 16 * l) = acc[2 + (i >> 1)][i & 1];
    LBAR();
#pragma unroll
    for (int i = 0; i < 4; ++i) v[i] = acc[i >> 1][i & 1];
  }
  const unsigned par = call & 1u;
  {
    const float scale = __uint_as_float(us), up = __uint_as_float(upb);
    v4f t[4];
    u64 m[16];
    unsigned q[16], b4[4] = {0, 0, 0, 0}, excm = 0, exco = 0;
#pragma unroll
    for (int i = 0; i < 4; ++i) t[i] = v[i] * up;
#pragma unroll
    for (int r = 0; r < 16; ++r) m[r] = __builtin_amdgcn_ballot_w64(!(t[r >> 2][r & 3] <= 0.f));
#pragma unroll
    for (int e = 0; e < 4; ++e) push_bits(b4, m[e], m[4 + e], m[8 + e], m[12 + e]);
    const unsigned bits = (b4[0] << 4 | b4[1]) << 8 | (b4[2] << 4 | b4[3]);
    const unsigned c = __builtin_popcount(bits);
    const unsigned pre = wave_scan(c);
    if (l == 63) *(LAS unsigned short*)(size_t)(ldsb + LDS_MISC + 32 + 2 * wv) = (unsigned short)pre;
    *(LAS unsigned short*)(size_t)(ldsb + L_TXM + 128 * wv + 2 * l) = (unsigned short)bits;
    STAMP(3);
    LBAR();
    unsigned zs = *(LAS unsigned short*)(size_t)(ldsb + LDS_MISC + 32 + 2 * (l & 7));
    SB();
#pragma unroll
    for (int r = 0; r < 16; ++r) asm("v_cvt_u32_f32 %0, %1" : "=v"(q[r]) : "v"(t[r >> 2][r & 3]));
    const v4f ts = (t[0] + t[1]) + (t[2] + t[3]);
    const float tn = (ts[0] + ts[1]) + (ts[2] + ts[3]);
    u64 bq[4];
#pragma unroll
    for (int i = 0; i < 4; ++i) bq[i] = __builtin_amdgcn_ballot_w64(((q[4 * i] | q[4 * i + 1] | q[4 * i + 2] | q[4 * i + 3]) & 0xffu) != 0);
    bool vote = plain | (__builtin_amdgcn_ballot_w64(tn != tn) != 0);
    SB();
    zs += dpp0<0x111, 0xf>(zs); zs += dpp0<0x112, 0xf>(zs); zs += dpp0<0x114, 0xf>(zs);
    const unsigned cnt = __builtin_amdgcn_readlane(pre, 63);
    const unsigned nrun = 16 + __builtin_amdgcn_readlane(zs, 7), off = 16 + __builtin_amdgcn_readlane(zs, wv) - cnt;
    const unsigned db = ldsb + L_TX;
    if (bq[0] | bq[1] | bq[2] | bq[3]) {
      const unsigned nq = __builtin_popcountll(bq[0]) + __builtin_popcountll(bq[1]) + __builtin_popcountll(bq[2]) + __builtin_popcountll(bq[3]);
      vote |= nq > 21 || 18 * nq > cnt;
      if (!vote) {
        unsigned o = 0;
        if (l == 0) o = __hip_atomic_fetch_add((LAS unsigned*)(size_t)(ldsb + LDS_MISC + 12), 6 * nq, __ATOMIC_RELAXED, __HIP_MEMORY_SCOPE_WORKGROUP);
        exco = nrun + rfl(o);
        unsigned sb = db + 4 * exco;
#pragma unroll
        for (int i = 0; i < 4; ++i)
          if (bq[i]) {
            excm |= 8u >> i;
            const u32x4 y = __builtin_bit_cast(u32x4, v[i] * scale);
#pragma unroll
            for (int e = 0; e < 4; ++e) q[4 * i + e] = (q[4 * i + e] & 0xffu) ? 0u : q[4 * i + e];
            if (__builtin_amdgcn_inverse_ballot_w64(bq[i])) {
              LAS unsigned* x = (LAS unsigned*)(size_t)(sb + 24 * mbcnt64(bq[i]));
              x[0] = y[0]; x[1] = y[1]; x[2] = y[2]; x[3] = y[3];
              x[4] = __builtin_amdgcn_perm(y[1], y[0], 0x04000c0cu); x[5] = __builtin_amdgcn_perm(y[3], y[2], 0x04000c0cu);
            }
            sb += 24 * __builtin_popcountll(bq[i]);
          }
      }
    }
    if (vote && l == 0) misc[4] = 0xffffffffu;
    const unsigned ad0 = db + 4 * (off + pre - c);
    auto scatter = [&]() {
      unsigned ad = ad0;
#pragma unroll
      for (int i = 0; i < 4; ++i)
        asm volatile("s_mov_b64 exec, %1\n\tds_write_b32 %0, %5\n\tv_add_u32 %0, 4, %0\n\t"
                     "s_mov_b64 exec, %2\n\tds_write_b32 %0, %6\n\tv_add_u32 %0, 4, %0\n\t"
                     "s_mov_b64 exec, %3\n\tds_write_b32 %0, %7\n\tv_add_u32 %0, 4, %0\n\t"
                     "s_mov_b64 exec, %4\n\tds_write_b32 %0, %8\n\tv_add_u32 %0, 4, %0\n\ts_mov_b64 exec, -1"
                     : "+v"(ad)
                     : "s"(m[4 * i]), "s"(m[4 * i + 1]), "s"(m[4 * i + 2]), "s"(m[4 * i + 3]),
                       "v"(q[4 * i]), "v"(q[4 * i + 1]), "v"(q[4 * i + 2]), "v"(q[4 * i + 3]) : "memory");
    };
    scatter();
    if (l == 0) *(LAS v2i*)(size_t)(db + 8 * wv) = v2i{(int)((off | (unsigned)sh << 16) << 8), (int)((exco | excm << 16) << 8)};
    char* const base = (pull ? me + WS_OUT + ((size_t)(par * NR + dst) * 32 + idx) * SLOT : peer + WS_IN + ((size_t)(par * NR + a.rank) * 32 + idx) * SLOT);
    const __amdgpu_buffer_rsrc_t rs = __builtin_amdgcn_make_buffer_rsrc(base, 0, SLOT, 0x00020000);
    STAMP(4);
    LBAR();
    const unsigned ne = misc[3], vt = misc[4];
    if (wv == 7) bst16_sys<0>(rs, 16 * l, 0, *(LAS u32x4*)(size_t)(ldsb + L_TXM + 16 * l));
    const bool raw = rfl(vt) == 0xffffffffu;
    if (raw) {
#pragma unroll
      for (int r = 0; r < 16; ++r) q[r] = __float_as_uint(v[r >> 2][r & 3] * scale);
      scatter();
      LBAR();
    }
    const unsigned n = raw ? nrun : nrun + rfl(ne), ng = (n + 15) >> 4;
    for (unsigned g0 = 64 * wv; g0 < ng; g0 += 512) {
      const unsigned nb = min(64u, ng - g0);
      if (l < nb) {
        u32x4 d[4];
#pragma unroll
        for (int i = 0; i < 4; ++i) d[i] = *(LAS u32x4*)(size_t)(db + 64 * g0 + 16 * (i * nb + l));
        if (raw) {
#pragma unroll
          for (int i = 0; i < 4; ++i) bst16_sys<0>(rs, 16 * l, 1024 + 64 * g0 + 16 * i * nb, d[i]);
        } else {
#pragma unroll
          for (int i = 0; i < 3; ++i) {
            u32x4 w;
#pragma unroll
            for (int e = 0; e < 4; ++e) w[e] = __builtin_amdgcn_perm(d[i][e], d[3][e], 0x07060503u - i);
            if (i >= PUSH_DEPTH && !pull) wait_vm<(PUSH_DEPTH < 3 ? PUSH_DEPTH : 3) - 1>();
            bst16_sys<0>(rs, 16 * l, 1024 + 48 * g0 + 16 * i * nb, w);
            if (i == 0) STAMP(5);
          }
        }
      }
    }
    STAMP(6);
    wait_vm<0>();
    BAR();
    if (tid == 0)
      __hip_atomic_store((GAS u64*)(peer + WS_FLAG + ((size_t)(par * NR + a.rank) * 32 + idx) * 128), ((u64)call << 32) | n | (raw ? 0x8000u : 0u),
                         __ATOMIC_RELAXED, __HIP_MEMORY_SCOPE_SYSTEM);
    STAMP(7);
  }
  {
    GAS unsigned* tmo = (GAS unsigned*)(me + WS_TMO);
    const char* base = pull ? a.ws[src] + WS_OUT + ((size_t)(par * NR + a.rank) * 32 + idx) * SLOT
                            : me + WS_IN + ((size_t)(par * NR + src) * 32 + idx) * SLOT;
    const __amdgpu_buffer_rsrc_t rs = __builtin_amdgcn_make_buffer_rsrc((void*)base, 0, SLOT, 0x00020000);
    float* o = a.out + (size_t)(MS * src + 64 * rb + 32 * (wv & 1) + (l & 15)) * NS + 128 * cb + 64 * (wv >> 1 & 1) + 32 * (wv < 4) + 4 * (l >> 4);
    unsigned keep = 0xffffff00u;
    asm volatile("" : "+v"(o), "+s"(keep));
    if (wv == 6) {
      GAS u64* flag = (GAS u64*)(me + WS_FLAG + ((size_t)(par * NR + src) * 32 + idx) * 128);
      if (wait_flag(flag, tmo, call, ldsb + LDS_MISC + 4) == 0 && l == 0)
        __hip_atomic_store(tmo, call, __ATOMIC_RELAXED, __HIP_MEMORY_SCOPE_AGENT);
    } else {
      BAR();
    }
    STAMP(8);
    const unsigned fl = rfl(misc[1]);
    const unsigned ng = min(((fl & 0x7fffu) + 15) >> 4, (unsigned)(DENSE / 64));
    const bool raw = fl & 0x8000u;
    const unsigned wo = 1024, db = ldsb + L_RX;
    const unsigned nb0 = raw || ng < 64u * wv ? 0u : min(64u, ng - 64 * wv);
    const unsigned vb = l < nb0 ? 16 * l : (unsigned)SLOT, s0 = wo + 48 * 64 * wv, s1 = s0 + 16 * nb0, s2 = s1 + 16 * nb0;
    const bool paced = pull && PULL_DEPTH == 1;
    unsigned tv, bits;
    u32x4 w[3];
    asm volatile("global_load_dword %0, %5, off sc1\n\tbuffer_load_ushort %1, %6, %8, %9 offen sc0 sc1\n\t"
                 "buffer_load_dwordx4 %2, %7, %8, %10 offen sc0 sc1\n\tbuffer_load_dwordx4 %3, %7, %8, %11 offen sc0 sc1\n\t"
                 "buffer_load_dwordx4 %4, %7, %8, %12 offen sc0 sc1\n\ts_waitcnt vmcnt(3)"
                 : "=&v"(tv), "=&v"(bits), "=&v"(w[0]), "=&v"(w[1]), "=&v"(w[2])
                 : "v"(tmo), "v"(2 * l), "v"(vb), "s"(rs), "s"(fl ? 128 * wv : SLOT), "s"(s0), "s"(paced ? SLOT : s1), "s"(paced ? SLOT : s2) : "memory");
    const unsigned c = __builtin_popcount(bits);
    const unsigned pre = wave_scan(c) - c;
    u64 m[16];
    u32x4 y[4] = {};
    bits <<= 16;
#pragma unroll
    for (int i = 0; i < 4; ++i)
      asm volatile("v_add_co_u32_e64 %0, %1, %0, %0\n\tv_add_co_u32_e64 %0, %2, %0, %0\n\tv_add_co_u32_e64 %0, %3, %0, %0\n\tv_add_co_u32_e64 %0, %4, %0, %0\n\ts_nop 1"
                   : "+v"(bits), "=s"(m[4 * i]), "=s"(m[4 * i + 1]), "=s"(m[4 * i + 2]), "=s"(m[4 * i + 3]), "+v"(y[i]));
    asm volatile("s_waitcnt vmcnt(0)" : "+v"(w[0]), "+v"(w[1]), "+v"(w[2]) :: "memory");
    if (paced)
      asm volatile("buffer_load_dwordx4 %0, %2, %3, %4 offen sc0 sc1\n\ts_waitcnt vmcnt(0)\n\tbuffer_load_dwordx4 %1, %2, %3, %5 offen sc0 sc1\n\ts_waitcnt vmcnt(0)"
                   : "=&v"(w[1]), "=&v"(w[2]) : "v"(vb), "s"(rs), "s"(s1), "s"(s2) : "memory");
    auto widen = [&](unsigned g0, unsigned nb) {
      u32x4 d[4];
#pragma unroll
      for (int e = 0; e < 4; ++e) {
#pragma unroll
        for (int i = 0; i < 3; ++i) d[i][e] = w[i][e] & keep;
        d[3][e] = __builtin_amdgcn_perm(w[0][e], __builtin_amdgcn_perm(w[1][e], w[2][e], 0x0c04000cu), 0x0402010cu);
      }
#pragma unroll
      for (int i = 0; i < 4; ++i) *(LAS u32x4*)(size_t)(db + 64 * g0 + 16 * (i * nb + l)) = d[i];
    };
    if (l < nb0) widen(64 * wv, nb0);
    if (raw || ng > 512) {
      for (unsigned g0 = 64 * wv + (raw ? 0 : 512); g0 < ng; g0 += 512) {
        const unsigned nb = min(64u, ng - g0);
        if (l < nb) {
          if (raw) {
#pragma unroll
            for (int i = 0; i < 4; ++i) bglds_sys<0>(db + 64 * g0 + 16 * i * nb, rs, 16 * l, wo + 64 * g0 + 16 * i * nb);
          } else {
            asm volatile("buffer_load_dwordx4 %0, %3, %4, %5 offen sc0 sc1\n\tbuffer_load_dwordx4 %1, %3, %4, %6 offen sc0 sc1\n\t"
                         "buffer_load_dwordx4 %2, %3, %4, %7 offen sc0 sc1\n\ts_waitcnt vmcnt(0)"
                         : "=&v"(w[0]), "=&v"(w[1]), "=&v"(w[2])
                         : "v"(16 * l), "s"(rs), "s"(wo + 48 * g0), "s"(wo + 48 * g0 + 16 * nb), "s"(wo + 48 * g0 + 32 * nb) : "memory");
            widen(g0, nb);
          }
        }
      }
      wait_vm<0>();
    }
    const bool bad = fl == 0 || rfl(tv) == call;
    LBAR();
    STAMP(9);
    if (bad) {
#pragma unroll
      for (int i = 0; i < 4; ++i) st16_sys(o + (i & 1) * 16 * NS + (i >> 1) * 16, u32x4{0x7fc00000u, 0x7fc00000u, 0x7fc00000u, 0x7fc00000u});
      return;
    }
    const v2i info = *(LAS v2i*)(size_t)(db + 8 * wv);
    const unsigned i0 = rfl(info[0]) >> 8, i1 = rfl(info[1]) >> 8;
    unsigned ad = db + 4 * (min(i0 & 0xffffu, (unsigned)(DENSE / 4)) + pre);
#pragma unroll
    for (int i = 0; i < 4; ++i)
      asm volatile("s_mov_b64 exec, %5\n\tds_read_b32 %1, %0\n\tv_add_u32 %0, 4, %0\n\t"
                   "s_mov_b64 exec, %6\n\tds_read_b32 %2, %0\n\tv_add_u32 %0, 4, %0\n\t"
                   "s_mov_b64 exec, %7\n\tds_read_b32 %3, %0\n\tv_add_u32 %0, 4, %0\n\t"
                   "s_mov_b64 exec, %8\n\tds_read_b32 %4, %0\n\tv_add_u32 %0, 4, %0\n\ts_mov_b64 exec, -1"
                   : "+v"(ad), "+v"(y[i][0]), "+v"(y[i][1]), "+v"(y[i][2]), "+v"(y[i][3])
                   : "s"(m[4 * i]), "s"(m[4 * i + 1]), "s"(m[4 * i + 2]), "s"(m[4 * i + 3]) : "memory");
    STAMP(10);
    const float down = __uint_as_float((us & 0x7fffffffu) - (((i0 >> 16) + 8) << 23));
    unsigned sb = db + 4 * min(i1 & 0xffffu, (unsigned)(DENSE / 4));
#pragma unroll
    for (int i = 0; i < 4; ++i) {
      asm volatile("s_waitcnt lgkmcnt(%4)" : "+v"(y[i][0]), "+v"(y[i][1]), "+v"(y[i][2]), "+v"(y[i][3]) : "i"(12 - 4 * i) : "memory");
      u32x4 z = y[i];
      if (!raw) {
        z = __builtin_bit_cast(u32x4, v4f{(float)z[0], (float)z[1], (float)z[2], (float)z[3]} * down);
        if (i1 & (0x80000u >> i)) {
          u64 x = 0;
#pragma unroll
          for (int e = 0; e < 4; ++e) x |= m[4 * i + e] & __builtin_amdgcn_ballot_w64(y[i][e] == 0);
          if (__builtin_amdgcn_inverse_ballot_w64(x)) {
            const LAS unsigned* p = (const LAS unsigned*)(size_t)(sb + 24 * mbcnt64(x));
            const unsigned lo01 = p[4], lo23 = p[5];
            z = u32x4{p[0] | (lo01 >> 16 & 0xffu), p[1] | lo01 >> 24, p[2] | (lo23 >> 16 & 0xffu), p[3] | lo23 >> 24};
#pragma unroll
            for (int e = 0; e < 4; ++e) z[e] = __uint_as_float(z[e]) <= 0.f ? 0u : z[e];
          }
          sb += 24 * __builtin_popcountll(x);
        }
      }
      st16_sys(o + (i & 1) * 16 * NS + (i >> 1) * 16, z);
    }
    STAMP(11);
    STAMP_FLUSH();
  }
}

}

extern "C" void kernel_launch(int rank, int nranks, ncclComm_t comm,
                              hipStream_t stream,
                              void* const* d_in, const int* in_sizes, int n_in,
                              void* d_out, int out_size,
                              const int* in_dtypes, int out_dtype,
                              void* const* d_out_all,
                              void* const* d_ws_all, size_t ws_bytes) {
  if (nranks != NR || (unsigned)rank >= (unsigned)NR || n_in != 4 || in_sizes[0] != MS * KD || in_sizes[1] != KD * ND ||
      out_size != NR * MS * NS || ws_bytes < WS_END) {
    fprintf(stderr, "kernel_launch: unexpected problem shape\n");
    return;
  }
  HIP_CHECK(hipFuncSetAttribute((const void*)fused_gemm_a2a, hipFuncAttributeMaxDynamicSharedMemorySize, LDS_BYTES));
  Args a;
  a.x = (const unsigned char*)d_in[0];
  a.w = (const unsigned char*)d_in[1];
  a.sx = (const float*)d_in[2];
  a.sw = (const float*)d_in[3];
  a.out = (float*)d_out;
  for (int p = 0; p < NR; ++p) a.ws[p] = (char*)d_ws_all[p];
  a.rank = rank;
  a.pad = 0;
  hipLaunchKernelGGL(fused_gemm_a2a, dim3(256), dim3(512), LDS_BYTES, stream, a);
  HIP_CHECK(hipGetLastError());
}
```

```cpp
#include <hip/hip_runtime.h>
#include <rccl/rccl.h>
#include <cstdio>

#define HIP_CHECK(expr) do { hipError_t _e = (expr); if (_e != hipSuccess) { \
    fprintf(stderr, "HIP error %s at %s:%d\n", hipGetErrorString(_e), __FILE__, __LINE__); \
    return; } } while (0)

namespace {

typedef unsigned u32x4 __attribute__((ext_vector_type(4)));
typedef int v2i __attribute__((ext_vector_type(2)));
typedef int v4i __attribute__((ext_vector_type(4)));
typedef int v8i __attribute__((ext_vector_type(8)));
typedef float v4f __attribute__((ext_vector_type(4)));
typedef unsigned long long u64;
#define LAS __attribute__((address_space(3)))
#define GAS __attribute__((address_space(1)))

constexpr int NR = 8;
constexpr int MS = 512, KD = 4096, ND = 4096, NS = 512;
constexpr int DEPTH = 5, STAGE = 24576, NSTEP = 32;
constexpr int LDS_MISC = DEPTH * STAGE;
constexpr int LDS_BYTES = LDS_MISC + 128 + 1024;
constexpr int DENSE = 32896;
constexpr int T_REC = 1024, T_BLK = T_REC + 2048;
#define SLOT_PAD 0
constexpr int SLOT = T_BLK + DENSE + SLOT_PAD;
constexpr int L_TX = 0, L_RX = DENSE, L_TXM = 2 * DENSE, L_REC = L_TXM + 1024;
static_assert(L_REC + 2048 <= DEPTH * STAGE && SLOT % 128 == 0 && DENSE % 64 == 0, "");
constexpr size_t WS_CTR = 0;
constexpr size_t WS_TMO = 32768;
constexpr size_t WS_FLAG = 65536;
constexpr size_t WS_DBG = 524288;
constexpr size_t WS_IN = 1u << 20;
constexpr size_t WS_OUT = WS_IN + (size_t)2 * NR * 32 * SLOT;
constexpr size_t WS_END = WS_OUT + (size_t)2 * NR * 32 * SLOT;
constexpr u64 DEADLINE = 20000000ull * 2;

#define T1 0
#define T2 0
#define T3 0
#define TR 0
#define PF 3
#define PF_DEPTH 3
#define NPUSH 24
#define PUSH_DEPTH 2
#define PULL_DEPTH 1
#define LOCAL_DELAY 0

#define POLL_GAP "2"

#define MASKS_FIRST ""

struct Args {
  const unsigned char* x;
  const unsigned char* w;
  const float* sx;
  const float* sw;
  float* out;
  char* ws[NR];
  int rank;
  int pad;
};

__device__ __forceinline__ void glds(unsigned m0, unsigned voff, __amdgpu_buffer_rsrc_t rs, unsigned soff) {
  asm volatile("s_mov_b32 m0, %0\n\ts_nop 0\n\tbuffer_load_dwordx4 %1, %2, %3 offen lds" ::"s"(m0), "v"(voff), "s"(rs), "s"(soff) : "memory");
}
__device__ __forceinline__ void st16_sys(void* p, u32x4 v) {
  asm volatile("global_store_dwordx4 %0, %1, off sc0 sc1\n\ts_nop 1" ::"v"(p), "v"(v) : "memory");
}
template <int OFF> __device__ __forceinline__ void bst16_sys(__amdgpu_buffer_rsrc_t rs, unsigned voff, unsigned soff, u32x4 v) {
  asm volatile("buffer_store_dwordx4 %0, %1, %2, %3 offen offset:%4 sc0 sc1\n\ts_nop 1" ::"v"(v), "v"(voff), "s"(rs), "s"(soff), "i"(OFF) : "memory");
}
template <int OFF> __device__ __forceinline__ void bglds_sys(unsigned m0, __amdgpu_buffer_rsrc_t rs, unsigned voff, unsigned soff) {
  asm volatile("s_mov_b32 m0, %0\n\ts_nop 0\n\tbuffer_load_dwordx4 %1, %2, %3 offen offset:%4 sc0 sc1 lds" ::"s"(m0), "v"(voff), "s"(rs), "s"(soff), "i"(OFF) : "memory");
}
template <int N> __device__ __forceinline__ void wait_vm() { asm volatile("s_waitcnt vmcnt(%0)" ::"i"(N) : "memory"); }
#define SB() __builtin_amdgcn_sched_barrier(0)
#define BAR() do { asm volatile("s_barrier" ::: "memory"); SB(); } while (0)
#define WAIT_LDS() do { __builtin_amdgcn_s_waitcnt(0xC07F); SB(); } while (0)
#define LBAR() do { asm volatile("s_waitcnt lgkmcnt(0)\n\ts_barrier" ::: "memory"); SB(); } while (0)

#define STAMP(i) do {} while (0)
#define STAMP_FLUSH() do {} while (0)

struct Frag { v8i x[2]; v8i w[4]; };

__device__ __forceinline__ void kstep(v4f (&acc)[4][2], const Frag& c, Frag& n, bool mm, unsigned so, unsigned ax0, unsigned ax1, const unsigned (&aw)[4]) {
  v4i xa[2], xb[2]; v2i t[4][4];
#define RX(d, a) d = *(LAS v4i*)(size_t)(a)
#define RW(nb, u, off) t[nb][u] = __builtin_amdgcn_ds_read_tr8_b64_v2i32((LAS v2i*)(size_t)(aw[nb] + so + off))
#define MM(nb, mb) if (mm) acc[nb][mb] = __builtin_amdgcn_mfma_scale_f32_16x16x128_f8f6f4(c.w[nb], c.x[mb], acc[nb][mb], 0, 0, 0, 0, 0, 0)
  MM(0, 0); SB();
  RX(xa[0], ax0 + so); RX(xb[0], ax1 + so); RW(0, 0, 0); SB();
  MM(0, 1); SB();
  RW(0, 1, 1024); RW(0, 2, 8192); RW(0, 3, 9216); SB();
  MM(1, 0); SB();
  RX(xa[1], ax0 + so + 2048); RX(xb[1], ax1 + so + 2048); RW(1, 0, 0); SB();
  MM(1, 1); SB();
  RW(1, 1, 1024); RW(1, 2, 8192); RW(1, 3, 9216); SB();
  MM(2, 0); SB();
  RW(2, 0, 0); RW(2, 1, 1024); RW(2, 2, 8192); RW(2, 3, 9216); SB();
  MM(2, 1); SB();
  RW(3, 0, 0); RW(3, 1, 1024); RW(3, 2, 8192); RW(3, 3, 9216); SB();
  MM(3, 0); SB();
  MM(3, 1); SB();
#pragma unroll
  for (int mb = 0; mb < 2; ++mb) n.x[mb] = __builtin_shufflevector(xa[mb], xb[mb], 0, 1, 2, 3, 4, 5, 6, 7);
#pragma unroll
  for (int nb = 0; nb < 4; ++nb) n.w[nb] = v8i{t[nb][0][0], t[nb][0][1], t[nb][1][0], t[nb][1][1], t[nb][2][0], t[nb][2][1], t[nb][3][0], t[nb][3][1]};
  WAIT_LDS();
#undef RX
#undef RW
#undef MM
}

__device__ __forceinline__ unsigned lds_u32(unsigned a) { return *(LAS volatile unsigned*)(size_t)a; }
__device__ __forceinline__ unsigned rfl(unsigned v) { return __builtin_amdgcn_readfirstlane(v); }
template <int CTRL, int ROWS> __device__ __forceinline__ unsigned dpp0(unsigned v) { return __builtin_amdgcn_update_dpp(0u, v, CTRL, ROWS, 0xf, true); }
__device__ __forceinline__ unsigned wave_scan(unsigned v) {
  v += dpp0<0x111, 0xf>(v); v += dpp0<0x112, 0xf>(v); v += dpp0<0x114, 0xf>(v); v += dpp0<0x118, 0xf>(v);
  v += dpp0<0x142, 0xa>(v); v += dpp0<0x143, 0xc>(v);
  return v;
}
__device__ __forceinline__ unsigned wave_max(unsigned v) {
  v = max(v, dpp0<0x111, 0xf>(v)); v = max(v, dpp0<0x112, 0xf>(v)); v = max(v, dpp0<0x114, 0xf>(v)); v = max(v, dpp0<0x118, 0xf>(v));
  v = max(v, dpp0<0x142, 0xa>(v)); v = max(v, dpp0<0x143, 0xc>(v));
  return v;
}
__device__ __forceinline__ unsigned mbcnt64(u64 m) { return __builtin_amdgcn_mbcnt_hi((unsigned)(m >> 32), __builtin_amdgcn_mbcnt_lo((unsigned)m, 0u)); }
__device__ __forceinline__ unsigned scan8(unsigned v) {
  v += dpp0<0x111, 0xf>(v); v += dpp0<0x112, 0xf>(v); v += dpp0<0x114, 0xf>(v);
  return v;
}

__device__ __forceinline__ unsigned wait_flag(GAS u64* flag, GAS unsigned* tmo, unsigned call, unsigned lds_word) {
  unsigned n, hi;
#define POLL(f) "global_load_dwordx2 " f ", %[flag], off sc0 sc1\n\t"
#define TEST(f0, f1) "s_waitcnt vmcnt(4)\n\tv_readfirstlane_b32 %[hi], " f1 "\n\tv_readfirstlane_b32 %[n], " f0 "\n\ts_cmp_ge_u32 %[hi], %[call]\n\ts_cbranch_scc1 2f\n\t"
  asm volatile("s_memrealtime s[96:97]\n\t"
               POLL("v[100:101]") "s_sleep " POLL_GAP "\n\t" POLL("v[102:103]") "s_sleep " POLL_GAP "\n\t" POLL("v[104:105]") "s_sleep " POLL_GAP "\n\t" POLL("v[106:107]")
               "global_load_dword v108, %[tmo], off sc1\n"
               "1:\n\t"
               TEST("v100", "v101") POLL("v[100:101]") TEST("v102", "v103") POLL("v[102:103]")
               TEST("v104", "v105") POLL("v[104:105]") TEST("v106", "v107") POLL("v[106:107]")
               "s_memrealtime s[98:99]\n\ts_waitcnt vmcnt(4) lgkmcnt(0)\n\tv_readfirstlane_b32 %[hi], v108\n\t"
               "global_load_dword v108, %[tmo], off sc1\n\t"
               "s_sub_u32 %[n], s98, s96\n\ts_cmp_gt_u32 %[n], %[dl]\n\ts_cselect_b32 %[hi], %[call], %[hi]\n\t"
               "s_cmp_lg_u32 %[hi], %[call]\n\ts_cbranch_scc1 1b\n\t"
               "s_mov_b32 %[n], 0\n"
               "2:\n\t"
               "v_mov_b32 v109, %[lw]\n\tv_mov_b32 v110, %[n]\n\tds_write_b32 v109, v110\n\t"
               "s_waitcnt lgkmcnt(0)\n\ts_barrier\n\ts_waitcnt vmcnt(0)"
               : [n] "=&s"(n), [hi] "=&s"(hi)
               : [flag] "v"(flag), [tmo] "v"(tmo), [call] "s"(call), [dl] "s"((unsigned)DEADLINE), [lw] "s"(lds_word)
               : "memory", "scc", "s96", "s97", "s98", "s99", "v100", "v101", "v102", "v103", "v104", "v105", "v106", "v107", "v108", "v109", "v110");
#undef POLL
#undef TEST
  return n;
}

__global__ void __launch_bounds__(512) fused_gemm_a2a(Args a) {
  extern __shared__ __attribute__((aligned(16))) char lds[];
  const int tid = threadIdx.x, l = tid & 63;
  const int wv = __builtin_amdgcn_readfirstlane(tid >> 6);
  const int j = blockIdx.x & 7, idx = blockIdx.x >> 3;
  const int rb = idx & 7, cb = idx >> 3;
  const unsigned dst = (unsigned)(a.rank + j) & 7u, src = (unsigned)(a.rank - j) & 7u;
  const bool pull = idx >= NPUSH;
  const unsigned ldsb = (unsigned)(size_t)lds;
  char* const me = a.ws[0];
  char* const peer = a.ws[j];
  STAMP(0);
  const u64 t0 = __builtin_amdgcn_s_memrealtime();
  const __amdgpu_buffer_rsrc_t rsW = __builtin_amdgcn_make_buffer_rsrc((void*)a.w, 0, KD * ND, 0x00020000);
  const __amdgpu_buffer_rsrc_t rsX = __builtin_amdgcn_make_buffer_rsrc((void*)a.x, 0, MS * KD, 0x00020000);
  const int xdelay = j == 0 ? LOCAL_DELAY : 0;
  if (cb >= 2 && PF) {
    const unsigned pvx = (unsigned)((l >> 3) * KD + 16 * (l & 7)), pvw = (unsigned)((l >> 4) * ND + 16 * (l & 15));
    const int id = (idx - 16) * 8 + wv;
    const unsigned ox0 = (unsigned)(8 * (id & 63) * KD + 128 * (id >> 6)), ow = (unsigned)(4 * id * ND) + NS * dst;
#pragma unroll 1
    for (int t = 0; t < 8; ++t) {
      glds(ldsb + LDS_MISC + 128, pvx, rsX, ox0 + 512 * t);
      glds(ldsb + LDS_MISC + 128, pvw, rsW, ow + (unsigned)t * (512 * ND));
      glds(ldsb + LDS_MISC + 128, pvx, rsX, ox0 + 512 * t + 256);
      wait_vm<PF_DEPTH>();
    }
    if (PF != 3 && (cb == 3 || PF == 2)) {
      const int n2 = PF == 2 ? 128 : 64;
#pragma unroll 1
      for (int t = id & (n2 - 1); t < 1024; t += n2) {
        glds(ldsb + LDS_MISC + 128, pvw, rsW, (unsigned)(4 * t * ND) + NS * dst + 256);
        wait_vm<PF_DEPTH>();
      }
    }
  }
  LAS volatile unsigned* misc = (LAS volatile unsigned*)(size_t)(ldsb + LDS_MISC);

  unsigned call, us, fl_out = 0;
  __amdgpu_buffer_rsrc_t rsT;
  auto setup = [&](unsigned m0, unsigned m2) {
    call = rfl(m0) + 1; us = rfl(m2);
    const size_t slot = ((size_t)((call & 1u) * NR + (pull ? dst : a.rank)) * 32 + idx) * SLOT;
    rsT = __builtin_amdgcn_make_buffer_rsrc((pull ? me + WS_OUT : peer + WS_IN) + slot, 0, SLOT, 0x00020000);
  };
  auto send_block = [&](unsigned g0, unsigned nb, bool raw) {
    u32x4 d[4];
    const unsigned vo = l < nb ? 16 * l : (unsigned)SLOT;
#pragma unroll
    for (int i = 0; i < 4; ++i) d[i] = *(LAS u32x4*)(size_t)(ldsb + L_TX + 64 * g0 + 16 * (i * nb + l));
    if (raw) {
#pragma unroll
      for (int i = 0; i < 4; ++i) bst16_sys<0>(rsT, vo, T_BLK + 64 * g0 + 16 * i * nb, d[i]);
    } else {
      u32x4 w[3];
#pragma unroll
      for (int i = 0; i < 3; ++i) {
#pragma unroll
        for (int e = 0; e < 4; ++e) w[i][e] = __builtin_amdgcn_perm(d[i][e], d[3][e], 0x07060503u - i);
        if (i == 2 && PUSH_DEPTH == 2 && !pull) wait_vm<1>();
        bst16_sys<0>(rsT, vo, T_BLK + 48 * g0 + 16 * i * nb, w[i]);
      }
      asm volatile("" :: "v"(w[0]), "v"(w[1]), "v"(w[2]));
    }
  };

  if (wv < 4) {
    const int xr = 16 * wv + (l >> 3);
    const unsigned vx0 = (unsigned)(xr * KD + 16 * ((l & 7) ^ ((xr >> 1) & 7)));
    const unsigned vx1 = (unsigned)((xr + 8) * KD + 16 * ((l & 7) ^ (((xr + 8) >> 1) & 7))) - 1024;
    const unsigned cw = (unsigned)((l & 7) ^ ((l >> 4) & 3));
    const unsigned vw0 = (unsigned)((l >> 3) * ND) + 16 * cw, vw1 = vw0 + 8 * ND - 1024;
    const unsigned vw2 = (unsigned)((l >> 3) * ND) + 16 * (cw ^ 4) + 16 * ND - 2048, vw3 = vw2 + 8 * ND - 1024;
    const unsigned k0 = (PF == 3 && cb >= 2) ? 4 * rb : 0;
    unsigned left = 32 - k0;
    unsigned sx = (unsigned)(64 * rb * KD) + 128 * k0, sw = (unsigned)(32 * wv * ND) + NS * dst + 128 * cb + k0 * (128 * ND);
    unsigned mx = ldsb + wv * 2048;
    const unsigned dmw = 8192 + wv * 2048, mend = mx + DEPTH * STAGE;
    auto issue = [&]() {
      asm volatile("s_mov_b32 m0, %0\n\ts_nop 0\n\tbuffer_load_dwordx4 %4, %10, %1 offen lds\n\tbuffer_load_dwordx4 %5, %10, %1 offen offset:1024 lds\n\t"
                   "s_add_u32 m0, %0, %3\n\ts_addk_i32 %0, 0x6000\n\t"
                   "buffer_load_dwordx4 %6, %11, %2 offen lds\n\tbuffer_load_dwordx4 %7, %11, %2 offen offset:1024 lds\n\t"
                   "buffer_load_dwordx4 %8, %11, %2 offen offset:2048 lds\n\tbuffer_load_dwordx4 %9, %11, %2 offen offset:3072 lds\n\t"
                   "s_addk_i32 %1, 0x80\n\ts_add_u32 %2, %2, 0x80000"
                   : "+s"(mx), "+s"(sx), "+s"(sw)
                   : "s"(dmw), "v"(vx0), "v"(vx1), "v"(vw0), "v"(vw1), "v"(vw2), "v"(vw3), "s"(rsX), "s"(rsW) : "memory", "scc");
      if (mx == mend) mx -= DEPTH * STAGE;
      if (PF == 3 && --left == 0) { sx -= KD; sw -= (unsigned)KD * ND; }
    };
    static_assert(STAGE == 0x6000 && 128 * ND == 0x80000, "");
    const int delay = (cb == 0 ? 0 : cb == 1 ? T1 : cb == 2 ? T2 : T3) + rb * TR + xdelay;
    while ((int)(__builtin_amdgcn_s_memrealtime() - t0) < delay) __builtin_amdgcn_s_sleep(1);
    STAMP(12);
#pragma unroll
    for (int i = 0; i < DEPTH - 1; ++i) issue();
    STAMP(13);
#pragma unroll 1
    for (int s = 0; s < NSTEP - (DEPTH - 1); ++s) {
      wait_vm<(DEPTH - 2) * 6>();
      BAR();
      issue();
    }
    static_assert(DEPTH == 5, "tail below is written for DEPTH 5");
    wait_vm<18>(); BAR();
    wait_vm<12>(); BAR();
    wait_vm<6>(); BAR();
    setup(misc[0], misc[2]);
    asm volatile("" : "+s"(call), "+s"(us), "+s"(rsT));
    wait_vm<0>(); BAR();
    BAR();
    if (j == 0) { STAMP_FLUSH(); return; }
    BAR();
    unsigned ng = (__builtin_amdgcn_readlane(scan8(*(LAS unsigned short*)(size_t)(ldsb + LDS_MISC + 32 + 2 * (l & 7))), 7) + 8 + 15) >> 4;
    BAR();
    const bool raw = rfl(misc[4]) == 0xffffffffu;
    if (raw) {
      BAR();
      ng = (__builtin_amdgcn_readlane(scan8(*(LAS unsigned short*)(size_t)(ldsb + LDS_MISC + 48 + 2 * (l & 7))), 7) + 8 + 15) >> 4;
      BAR();
    }
    if (64u * wv < ng) send_block(64 * wv, min(64u, ng - 64 * wv), raw);
    if (wv == 0 && ng > 512) send_block(512, 1, raw);
    STAMP(5);
    fl_out = ng | (raw ? 1u << 17 : 0u) | 1u << 18;
  } else {
    unsigned ticket = 0;
    if (tid == 256) {
      GAS unsigned* ctr = (GAS unsigned*)(me + WS_CTR + blockIdx.x * 128);
      asm volatile("" : "+v"(ctr));
      ticket = __hip_atomic_fetch_add(ctr, 1u, __ATOMIC_RELAXED, __HIP_MEMORY_SCOPE_AGENT);
    }
    float sxv = a.sx[0], swv = a.sw[0];
    const int pw = wv - 4, mh = pw & 1, nh = pw >> 1;
    const int g = l >> 4, lq = (l & 15) >> 1, p = l & 1;
    const unsigned f = (l & 15) >> 1;
    const unsigned ax0 = ldsb + (32 * mh + (l & 15)) * 128 + ((g ^ f) * 16), ax1 = ldsb + (32 * mh + (l & 15)) * 128 + ((g ^ f ^ 4) * 16);
    const unsigned hW = ((lq >> 1) & 3) | ((g & 1) << 2);
    unsigned aw[4];
#pragma unroll
    for (int nb = 0; nb < 4; ++nb) aw[nb] = ldsb + 8192 + (16 * g + lq) * 128 + 8 * p + (((4 * nh + nb) ^ hW) * 16);
    v4f acc[4][2] = {};
    unsigned so = 0, sexp = 0;
    Frag fa, fb;
    BAR();
    STAMP(1);
    kstep(acc, fb, fa, false, so, ax0, ax1, aw);
    so += STAGE;
#pragma unroll 1
    for (int s = 1; s < 31; s += 2) {
      BAR();
      if (s == 23) {
        float mx = 0.f;
#pragma unroll
        for (int i = 0; i < 32; i += 2) mx = fmaxf(fmaxf(mx, fabsf(acc[i >> 3][(i >> 2) & 1][i & 3])), fabsf(acc[i >> 3][(i >> 2) & 1][(i & 3) + 1]));
        sexp = __builtin_amdgcn_readlane(wave_max(__float_as_uint(mx * 0.875f)), 63) >> 23;
        if (tid == 256) { misc[0] = ticket; misc[2] = __float_as_uint(sxv * swv); misc[3] = 0; misc[4] = 0; }
        SB();
      }
      kstep(acc, fa, fb, true, so, ax0, ax1, aw);
      so += STAGE; if (so == DEPTH * STAGE) so = 0;
      BAR();
      kstep(acc, fb, fa, true, so, ax0, ax1, aw);
      so += STAGE; if (so == DEPTH * STAGE) so = 0;
    }
    BAR();
    const unsigned m0 = misc[0], m2 = misc[2];
    kstep(acc, fa, fb, true, so, ax0, ax1, aw);
    BAR();
#pragma unroll
    for (int nb = 0; nb < 4; ++nb)
#pragma unroll
      for (int mb = 0; mb < 2; ++mb) acc[nb][mb] = __builtin_amdgcn_mfma_scale_f32_16x16x128_f8f6f4(fb.w[nb], fb.x[mb], acc[nb][mb], 0, 0, 0, 0, 0, 0);
    setup(m0, m2);
    int sh = 24 - ((int)sexp - 126);
    sh = sh > 18 ? 18 : sh;
    const int ec = (int)((us >> 23) & 0xffu);
    const bool plain = sh < 12 || ec == 0xff || ec - (sh + 8) < 1;
    const float up = __uint_as_float(plain ? us : (us & 0x80000000u) | (unsigned)(127 + 8 + sh) << 23);
    STAMP(2);
    if (j == 0) {
      const float scale = sxv * swv;
#pragma unroll
      for (int nb = 0; nb < 4; ++nb)
#pragma unroll
        for (int mb = 0; mb < 2; ++mb) {
          v4f y = acc[nb][mb] * scale;
#pragma unroll
          for (int e = 0; e < 4; ++e) y[e] = y[e] <= 0.f ? 0.f : y[e];
          float* o = a.out + (size_t)(MS * a.rank + 64 * rb + 32 * mh + 16 * mb + (l & 15)) * NS + 128 * cb + 64 * nh + 16 * nb + 4 * g;
          st16_sys(o, __builtin_bit_cast(u32x4, y));
        }
      STAMP_FLUSH();
      return;
    }
    const float scale = __uint_as_float(us);
    v4f t[8];
    unsigned q[32], nz4[32], bits[2];
#pragma unroll
    for (int i = 0; i < 8; ++i) t[i] = acc[i >> 1][i & 1] * up;
#pragma unroll
    for (int r = 0; r < 32; ++r) asm("v_cvt_u32_f32 %0, %1" : "=v"(q[r]) : "v"(t[r >> 2][r & 3]));
#pragma unroll
    for (int r = 0; r < 32; ++r) nz4[r] = min(q[r], 4u);
#pragma unroll
    for (int h = 0; h < 2; ++h) {
      unsigned b[4];
#pragma unroll
      for (int i = 0; i < 4; ++i) {
        const unsigned* z = nz4 + 16 * h + 4 * i;
        b[i] = (z[0] << 1 | z[1]) << 2 | (z[2] << 1 | z[3]);
      }
      bits[h] = ((b[0] << 4 | b[1]) << 8 | (b[2] << 4 | b[3])) >> 2;
    }
    const unsigned c = __builtin_popcount(bits[0]) | __builtin_popcount(bits[1]) << 16;
    const unsigned pre = wave_scan(c);
    if (l == 63) *(LAS unsigned*)(size_t)(ldsb + LDS_MISC + 32 + 4 * pw) = pre;
    STAMP(3);
    LBAR();
    unsigned zs = *(LAS unsigned short*)(size_t)(ldsb + LDS_MISC + 32 + 2 * (l & 7));
    *(LAS unsigned short*)(size_t)(ldsb + L_TXM + 256 * pw + 2 * l) = (unsigned short)bits[0];
    *(LAS unsigned short*)(size_t)(ldsb + L_TXM + 256 * pw + 128 + 2 * l) = (unsigned short)bits[1];
    SB();
    const v4f ts = (t[0] + t[2]) + (t[4] + t[6]);
    const float tn = (ts[0] + ts[1]) + (ts[2] + ts[3]) + t[1][0];
    unsigned lo[8];
#pragma unroll
    for (int i = 0; i < 8; ++i) lo[i] = (q[4 * i] | q[4 * i + 1] | q[4 * i + 2] | q[4 * i + 3]) & 0xffu;
    const unsigned cnts = __builtin_amdgcn_readlane(pre, 63);
    unsigned nq = 0;
#pragma unroll
    for (int i = 0; i < 8; ++i) nq += __builtin_popcountll(__builtin_amdgcn_ballot_w64(lo[i] != 0));
    const bool vote = plain | (__builtin_amdgcn_ballot_w64(tn != tn) != 0) | (nq > 16) | (24 * nq > (cnts & 0xffffu) + (cnts >> 16));
    if (l == 0) {
      if (vote) misc[4] = 0xffffffffu;
      *(LAS unsigned char*)(size_t)(ldsb + LDS_MISC + 24 + pw) = (unsigned char)(sh - 12);
    }
    SB();
    zs = scan8(zs);
    unsigned nrun = 8 + __builtin_amdgcn_readlane(zs, 7);
    const unsigned off1 = 8 + __builtin_amdgcn_readlane(zs, 2 * pw), off0 = off1 - (cnts & 0xffffu);
    const unsigned db = ldsb + L_TX;
    const unsigned ex = pre - c;
    {
      unsigned ah[2] = {db + 4 * (off0 + (ex & 0xffffu)) + (1u << 20), db + 4 * (off1 + (ex >> 16)) + (1u << 20)};
#pragma unroll
      for (int i = 0; i < 8; ++i) {
        unsigned tmp, *z = nz4 + 4 * i;
        asm volatile("v_add_u32 %5, %4, %0\n\tv_mad_i32_i24 %0, %0, %6, %4\n\tv_add_u32 %4, %5, %1\n\tv_mad_i32_i24 %1, %1, %6, %5\n\t"
                     "v_add_u32 %5, %4, %2\n\tv_mad_i32_i24 %2, %2, %6, %4\n\tv_add_u32 %4, %5, %3\n\tv_mad_i32_i24 %3, %3, %6, %5"
                     : "+v"(z[0]), "+v"(z[1]), "+v"(z[2]), "+v"(z[3]), "+v"(ah[i >> 2]), "=&v"(tmp) : "s"(-(1 << 18)));
      }
#pragma unroll
      for (int i = 0; i < 8; ++i) {
        unsigned *z = nz4 + 4 * i, *y = q + 4 * i;
        asm volatile("ds_write_b32 %0, %4\n\tds_write_b32 %1, %5\n\tds_write_b32 %2, %6\n\tds_write_b32 %3, %7"
                     :: "v"(z[0]), "v"(z[1]), "v"(z[2]), "v"(z[3]), "v"(y[0]), "v"(y[1]), "v"(y[2]), "v"(y[3]) : "memory");
      }
    }
    if (l == 0) *(LAS v2i*)(size_t)(db + 8 * pw) = v2i{(int)(off0 << 10), (int)(off1 << 10)};
    STAMP(4);
    LBAR();
    const bool raw = rfl(misc[4]) == 0xffffffffu;
    if (raw) {
      unsigned rb2[2] = {0, 0};
#pragma unroll
      for (int r = 0; r < 32; ++r) {
        q[r] = __float_as_uint(acc[r >> 3][(r >> 2) & 1][r & 3] * scale);
        rb2[r >> 4] = 2 * rb2[r >> 4] + (!(__uint_as_float(q[r]) <= 0.f) ? 1u : 0u);
      }
      const unsigned rc = __builtin_popcount(rb2[0]) | __builtin_popcount(rb2[1]) << 16, rp = wave_scan(rc);
      if (l == 63) *(LAS unsigned*)(size_t)(ldsb + LDS_MISC + 48 + 4 * pw) = rp;
      *(LAS unsigned short*)(size_t)(ldsb + L_TXM + 256 * pw + 2 * l) = (unsigned short)rb2[0];
      *(LAS unsigned short*)(size_t)(ldsb + L_TXM + 256 * pw + 128 + 2 * l) = (unsigned short)rb2[1];
      LBAR();
      const unsigned rz = scan8(*(LAS unsigned short*)(size_t)(ldsb + LDS_MISC + 48 + 2 * (l & 7)));
      nrun = 8 + __builtin_amdgcn_readlane(rz, 7);
      const unsigned o1 = 8 + __builtin_amdgcn_readlane(rz, 2 * pw), o0 = o1 - (__builtin_amdgcn_readlane(rp, 63) & 0xffffu);
      if (l == 0) *(LAS v2i*)(size_t)(db + 8 * pw) = v2i{(int)(o0 << 10), (int)(o1 << 10)};
#pragma unroll
      for (int h = 0; h < 2; ++h) {
        unsigned ad = db + 4 * (h ? o1 + ((rp - rc) >> 16) : o0 + ((rp - rc) & 0xffffu));
#pragma unroll
        for (int r = 0; r < 16; ++r)
          if ((rb2[h] >> (15 - r)) & 1u) {
            *(LAS unsigned*)(size_t)ad = q[16 * h + r];
            ad += 4;
          }
      }
      LBAR();
    }
    const unsigned ng = (nrun + 15) >> 4;
    if (wv == 7) bst16_sys<0>(rsT, 16 * l, 0, *(LAS u32x4*)(size_t)(ldsb + L_TXM + 16 * l));
    if (64u * wv < ng) send_block(64 * wv, min(64u, ng - 64 * wv), raw);
    if (nq && !raw) {
      unsigned o = 0;
      if (l == 0) o = __hip_atomic_fetch_add((LAS unsigned*)(size_t)(ldsb + LDS_MISC + 12), nq, __ATOMIC_RELAXED, __HIP_MEMORY_SCOPE_WORKGROUP);
      unsigned sb = T_REC + 32 * rfl(o);
#pragma unroll
      for (int i = 0; i < 8; ++i) {
        asm("" : "+v"(lo[i]));
        const u64 b = __builtin_amdgcn_ballot_w64(lo[i] != 0);
        if (b) {
          const unsigned vo = lo[i] ? 32 * mbcnt64(b) : (unsigned)SLOT;
          bst16_sys<0>(rsT, vo, sb, __builtin_bit_cast(u32x4, acc[i >> 1][i & 1] * scale));
          bst16_sys<16>(rsT, vo, sb, u32x4{(unsigned)(l | (8 * pw + i) << 6), 0u, 0u, 0u});
          sb += 32 * __builtin_popcountll(b);
        }
      }
    }
  }
  STAMP(6);
  wait_vm<0>();
  LBAR();
  const unsigned par = call & 1u;
  if (tid == 0) {
    const unsigned s4 = misc[6];
    fl_out |= misc[3] << 10 | (s4 & 7u) << 20 | (s4 >> 8 & 7u) << 23 | (s4 >> 16 & 7u) << 26 | (s4 >> 24 & 7u) << 29;
    __hip_atomic_store((GAS u64*)(peer + WS_FLAG + ((size_t)(par * NR + a.rank) * 32 + idx) * 128), ((u64)call << 32) | fl_out,
                       __ATOMIC_RELAXED, __HIP_MEMORY_SCOPE_SYSTEM);
  }
  STAMP(7);
  {
    GAS unsigned* tmo = (GAS unsigned*)(me + WS_TMO);
    const char* base = pull ? a.ws[(8 - j) & 7] + WS_OUT + ((size_t)(par * NR + a.rank) * 32 + idx) * SLOT
                            : me + WS_IN + ((size_t)(par * NR + src) * 32 + idx) * SLOT;
    const __amdgpu_buffer_rsrc_t rs = __builtin_amdgcn_make_buffer_rsrc((void*)base, 0, SLOT, 0x00020000);
    float* const ot = a.out + (size_t)(MS * src + 64 * rb + 32 * (wv >> 1 & 1)) * NS + 128 * cb + 64 * (wv >> 2) + 32 * (wv & 1);
    float* o = ot + (l & 7) * NS + 2 * (l & 8) + 4 * (l >> 4);
    float* oy = ot + ((l & 7) + 8) * NS + 2 * ((l & 8) ^ 8) + 4 * (l >> 4);
    float *o1 = o + 16 * NS, *oy1 = oy + 16 * NS;
    unsigned keep = 0xffffff00u;
    asm volatile("" : "+v"(o), "+v"(oy), "+v"(o1), "+v"(oy1), "+s"(keep));
    if (wv == 6) {
      GAS u64* flag = (GAS u64*)(me + WS_FLAG + ((size_t)(par * NR + src) * 32 + idx) * 128);
      if (wait_flag(flag, tmo, call, ldsb + LDS_MISC + 4) == 0 && l == 0)
        __hip_atomic_store(tmo, call, __ATOMIC_RELAXED, __HIP_MEMORY_SCOPE_AGENT);
    } else {
      BAR();
    }
    STAMP(8);
    const unsigned fl = rfl(misc[1]);
    const unsigned nrec = min(fl >> 10 & 0x7fu, 64u), ng = min(fl & 0x3ffu, (unsigned)(DENSE / 64));
    const bool raw = fl & (1u << 17);
    const unsigned wo = T_BLK, db = ldsb + L_RX;
    const unsigned nb0 = raw || ng < 64u * wv ? 0u : min(64u, ng - 64 * wv);
    const unsigned vb = l < nb0 ? 16 * l : (unsigned)SLOT, s0 = wo + 48 * 64 * wv, s1 = s0 + 16 * nb0, s2 = s1 + 16 * nb0;
    const bool paced = pull && PULL_DEPTH == 1, paced2 = pull && PULL_DEPTH == 2;
    unsigned tv, bits;
    u32x4 w[3];
    if (wv == 7 && l < 2 * nrec) bglds_sys<0>(ldsb + L_REC, rs, 16 * l, T_REC);
    if (wv == 6 && l + 64 < 2 * nrec) bglds_sys<0>(ldsb + L_REC + 1024, rs, 16 * l, T_REC + 1024);
    asm volatile("global_load_dword %0, %5, off sc1\n\tbuffer_load_ushort %1, %6, %8, %9 offen sc0 sc1\n\t" MASKS_FIRST
                 "buffer_load_dwordx4 %2, %7, %8, %10 offen sc0 sc1\n\tbuffer_load_dwordx4 %3, %7, %8, %11 offen sc0 sc1\n\t"
                 "buffer_load_dwordx4 %4, %7, %8, %12 offen sc0 sc1\n\ts_waitcnt vmcnt(3)"
                 : "=&v"(tv), "=&v"(bits), "=&v"(w[0]), "=&v"(w[1]), "=&v"(w[2])
                 : "v"(tmo), "v"(2 * l), "v"(vb), "s"(rs), "s"(fl ? 128 * wv : SLOT), "s"(s0), "s"(paced ? SLOT : s1), "s"(paced || paced2 ? SLOT : s2) : "memory");
    if (paced2)
      asm volatile("s_waitcnt vmcnt(2)\n\tbuffer_load_dwordx4 %0, %1, %2, %3 offen sc0 sc1" : "+v"(w[2]) : "v"(vb), "s"(rs), "s"(s2) : "memory");
    const unsigned c = __builtin_popcount(bits);
    const unsigned pre = wave_scan(c) - c;
    const unsigned late = __builtin_amdgcn_update_dpp(0u, (pre + __builtin_popcount(bits >> 8)) << 8 | (bits & 0xffu), 0x128, 0xf, 0xf, true);
    const unsigned early = pre << 8 | bits >> 8;
    const unsigned sx = (l & 8) ? late : early, sy = (l & 8) ? early : late;
    u64 m[16];
    unsigned bx = sx << 24, by = sy << 24, ax = db + 4 * (sx >> 8), ay = db + 4 * (sy >> 8);
#pragma unroll
    for (int i = 0; i < 4; ++i)
      asm volatile("v_add_co_u32_e64 %0, %2, %0, %0\n\tv_add_co_u32_e64 %1, %4, %1, %1\n\tv_add_co_u32_e64 %0, %3, %0, %0\n\tv_add_co_u32_e64 %1, %5, %1, %1"
                   : "+v"(bx), "+v"(by), "=s"(m[2 * i]), "=s"(m[2 * i + 1]), "=s"(m[8 + 2 * i]), "=s"(m[9 + 2 * i]));
    asm volatile("s_waitcnt vmcnt(0)" : "+v"(w[0]), "+v"(w[1]), "+v"(w[2]), "+v"(ax), "+v"(ay) :: "memory");
    if (paced)
      asm volatile("buffer_load_dwordx4 %0, %2, %3, %4 offen sc0 sc1\n\ts_waitcnt vmcnt(0)\n\tbuffer_load_dwordx4 %1, %2, %3, %5 offen sc0 sc1\n\ts_waitcnt vmcnt(0)"
                   : "=&v"(w[1]), "=&v"(w[2]) : "v"(vb), "s"(rs), "s"(s1), "s"(s2) : "memory");
    auto widen = [&](unsigned g0, unsigned nb) {
      u32x4 d[4];
#pragma unroll
      for (int e = 0; e < 4; ++e) {
#pragma unroll
        for (int i = 0; i < 3; ++i) d[i][e] = w[i][e] & keep;
        d[3][e] = __builtin_amdgcn_perm(w[0][e], __builtin_amdgcn_perm(w[1][e], w[2][e], 0x0c04000cu), 0x0402010cu);
      }
#pragma unroll
      for (int i = 0; i < 4; ++i) *(LAS u32x4*)(size_t)(db + 64 * g0 + 16 * (i * nb + l)) = d[i];
    };
    if (l < nb0) widen(64 * wv, nb0);
    if (raw || ng > 512) {
      for (unsigned g0 = 64 * wv + (raw ? 0 : 512); g0 < ng; g0 += 512) {
        const unsigned nb = min(64u, ng - g0);
        if (l < nb) {
          if (raw) {
#pragma unroll
            for (int i = 0; i < 4; ++i) bglds_sys<0>(db + 64 * g0 + 16 * i * nb, rs, 16 * l, wo + 64 * g0 + 16 * i * nb);
          } else {
            asm volatile("buffer_load_dwordx4 %0, %3, %4, %5 offen sc0 sc1\n\tbuffer_load_dwordx4 %1, %3, %4, %6 offen sc0 sc1\n\t"
                         "buffer_load_dwordx4 %2, %3, %4, %7 offen sc0 sc1\n\ts_waitcnt vmcnt(0)"
                         : "=&v"(w[0]), "=&v"(w[1]), "=&v"(w[2])
                         : "v"(16 * l), "s"(rs), "s"(wo + 48 * g0), "s"(wo + 48 * g0 + 16 * nb), "s"(wo + 48 * g0 + 32 * nb) : "memory");
            widen(g0, nb);
          }
        }
      }
      wait_vm<0>();
    }
    const bool bad = fl == 0 || rfl(tv) == call;
    unsigned down = (us & 0x7fffffffu) - (((fl >> (20 + 3 * (wv >> 1)) & 7u) + 20) << 23);
    asm volatile("" : "+s"(down));
    LBAR();
    STAMP(9);
    if (bad) {
#pragma unroll
      for (int i = 0; i < 4; ++i) st16_sys((i >> 1 ? oy : o) + (i & 1) * 16 * NS, u32x4{0x7fc00000u, 0x7fc00000u, 0x7fc00000u, 0x7fc00000u});
      return;
    }
    u32x4 rec = *(LAS u32x4*)(size_t)(ldsb + L_REC + 32 * l);
    const unsigned id = *(LAS unsigned*)(size_t)(ldsb + L_REC + 32 * l + 16);
#define Z2(a, b) "v_mov_b64 v[" #a ":" #b "], 0\n\t"
#define G2(mx, x, my, y) "s_mov_b64 exec, %" #mx "\n\tds_read_b32 v" #x ", %0\n\tv_add_u32 %0, 4, %0\n\ts_mov_b64 exec, %" #my "\n\tds_read_b32 v" #y ", %1\n\tv_add_u32 %1, 4, %1\n\t"
#define GATHER "ds_read_b32 v116, %24\n\t" Z2(100, 101) Z2(102, 103) Z2(104, 105) Z2(106, 107) Z2(108, 109) Z2(110, 111) Z2(112, 113) Z2(114, 115) \
               "s_waitcnt lgkmcnt(0)\n\tv_lshrrev_b32 v116, 8, v116\n\tv_min_u32 v116, %2, v116\n\tv_add_u32 %0, %0, v116\n\tv_add_u32 %1, %1, v116\n\t" \
               G2(4, 100, 12, 108) G2(5, 101, 13, 109) G2(6, 102, 14, 110) G2(7, 103, 15, 111) \
               G2(8, 104, 16, 112) G2(9, 105, 17, 113) G2(10, 106, 18, 114) G2(11, 107, 19, 115) "s_mov_b64 exec, -1\n\t"
#define CV(a) "v_cvt_f32_u32 v" #a ", v" #a "\n\t"
#define ML(a) "v_mul_f32 v" #a ", %3, v" #a "\n\t"
#define ST4(p, a, d) "global_store_dwordx4 %" #p ", v[" #a ":" #d "], off sc0 sc1\n\t"
#define OUT4(p, a, b, c, d) CV(a) CV(b) CV(c) CV(d) ML(a) ML(b) ML(c) ML(d) ST4(p, a, d)
#define G_OPS : "+v"(ax), "+v"(ay) : "s"(DENSE), "s"(down), "s"(m[0]), "s"(m[1]), "s"(m[2]), "s"(m[3]), "s"(m[4]), "s"(m[5]), "s"(m[6]), "s"(m[7]), \
                "s"(m[8]), "s"(m[9]), "s"(m[10]), "s"(m[11]), "s"(m[12]), "s"(m[13]), "s"(m[14]), "s"(m[15]), \
                "v"(o), "v"(o1), "v"(oy), "v"(oy1), "v"(db + 4 * wv) \
              : "memory", "v100", "v101", "v102", "v103", "v104", "v105", "v106", "v107", "v108", "v109", "v110", "v111", "v112", "v113", "v114", "v115", "v116"
    if (raw) {
      asm volatile(GATHER "s_waitcnt lgkmcnt(0)\n\t" ST4(20, 100, 103) ST4(21, 104, 107) ST4(22, 108, 111) ST4(23, 112, 115) "s_nop 1" G_OPS);
    } else {
      asm volatile(GATHER "s_waitcnt lgkmcnt(8)\n\t" OUT4(20, 100, 101, 102, 103) OUT4(22, 108, 109, 110, 111)
                   "s_waitcnt lgkmcnt(0)\n\t" OUT4(21, 104, 105, 106, 107) OUT4(23, 112, 113, 114, 115) "s_nop 1" G_OPS);
      STAMP(10);
      if (l < nrec && id >> 8 == (unsigned)wv) {
#pragma unroll
        for (int e = 0; e < 4; ++e) rec[e] = __uint_as_float(rec[e]) <= 0.f ? 0u : rec[e];
        st16_sys(ot + ((id & 15u) + (id >> 2 & 16u)) * NS + (id >> 3 & 16u) + (id >> 2 & 12u), rec);
      }
    }
    STAMP(11);
    STAMP_FLUSH();
  }
}

}

extern "C" void kernel_launch(int rank, int nranks, ncclComm_t comm,
                              hipStream_t stream,
                              void* const* d_in, const int* in_sizes, int n_in,
                              void* d_out, int out_size,
                              const int* in_dtypes, int out_dtype,
                              void* const* d_out_all,
                              void* const* d_ws_all, size_t ws_bytes) {
  if (nranks != NR || (unsigned)rank >= (unsigned)NR || n_in != 4 || in_sizes[0] != MS * KD || in_sizes[1] != KD * ND ||
      out_size != NR * MS * NS || ws_bytes < WS_END) {
    fprintf(stderr, "kernel_launch: unexpected problem shape\n");
    return;
  }
  HIP_CHECK(hipFuncSetAttribute((const void*)fused_gemm_a2a, hipFuncAttributeMaxDynamicSharedMemorySize, LDS_BYTES));
  Args a;
  a.x = (const unsigned char*)d_in[0];
  a.w = (const unsigned char*)d_in[1];
  a.sx = (const float*)d_in[2];
  a.sw = (const float*)d_in[3];
  a.out = (float*)d_out;
  for (int p = 0; p < NR; ++p) a.ws[p] = (char*)d_ws_all[(rank + p) % NR];
  a.rank = rank;
  a.pad = 0;
  hipLaunchKernelGGL(fused_gemm_a2a, dim3(256), dim3(512), LDS_BYTES, stream, a);
  HIP_CHECK(hipGetLastError());
}
```

```cpp
#include <hip/hip_runtime.h>
#include <rccl/rccl.h>
#include <cstdio>

#define HIP_CHECK(expr) do { hipError_t _e = (expr); if (_e != hipSuccess) { \
    fprintf(stderr, "HIP error %s at %s:%d\n", hipGetErrorString(_e), __FILE__, __LINE__); \
    return; } } while (0)

namespace {

typedef unsigned u32x4 __attribute__((ext_vector_type(4)));
typedef int v2i __attribute__((ext_vector_type(2)));
typedef int v4i __attribute__((ext_vector_type(4)));
typedef int v8i __attribute__((ext_vector_type(8)));
typedef float v4f __attribute__((ext_vector_type(4)));
typedef unsigned long long u64;
#define LAS __attribute__((address_space(3)))
#define GAS __attribute__((address_space(1)))

constexpr int NR = 8;
constexpr int MS = 512, KD = 4096, ND = 4096, NS = 512;
#define RDEPTH 3
constexpr int DEPTH = RDEPTH, STAGE = 24576, NSTEP = 32;
constexpr int LDS_MISC = (DEPTH < 4 ? 4 : DEPTH) * STAGE;
constexpr int LDS_BYTES = LDS_MISC + 128 + 1024;
constexpr int DENSE = 32896;
constexpr int T_REC = 1024, T_BLK = T_REC + 2048;
#define SLOT_PAD 0
constexpr int SLOT = T_BLK + DENSE + SLOT_PAD;
constexpr int L_TX = 0, L_RX = DENSE, L_TXM = 2 * DENSE, L_REC = L_TXM + 1024;
static_assert(L_REC + 2048 <= DEPTH * STAGE && SLOT % 128 == 0 && DENSE % 64 == 0, "");
constexpr size_t WS_CTR = 0;
constexpr size_t WS_TMO = 32768;
constexpr size_t WS_FLAG = 65536;
constexpr size_t WS_DBG = 524288;
constexpr size_t WS_IN = 1u << 20;
constexpr size_t WS_OUT = WS_IN + (size_t)2 * NR * 32 * SLOT;
constexpr size_t WS_END = WS_OUT + (size_t)2 * NR * 32 * SLOT;
constexpr u64 DEADLINE = 20000000ull * 2;

#define PF 3
#define PF0 0
#define PF_W 8
#define PF_DEPTH 3
#define PULLMASK 0xff000000u
#define PULL_DEPTH 1
#define POLL_GAP "2"

struct Args {
  const unsigned char* x;
  const unsigned char* w;
  const float* sx;
  const float* sw;
  float* out;
  char* ws[NR];
  int rank;
  int pad;
};

__device__ __forceinline__ void glds(unsigned m0, unsigned voff, __amdgpu_buffer_rsrc_t rs, unsigned soff) {
  asm volatile("s_mov_b32 m0, %0\n\ts_nop 0\n\tbuffer_load_dwordx4 %1, %2, %3 offen lds" ::"s"(m0), "v"(voff), "s"(rs), "s"(soff) : "memory");
}
__device__ __forceinline__ void st16_sys(void* p, u32x4 v) {
  asm volatile("global_store_dwordx4 %0, %1, off sc0 sc1\n\ts_nop 1" ::"v"(p), "v"(v) : "memory");
}
template <int OFF> __device__ __forceinline__ void bst16_sys(__amdgpu_buffer_rsrc_t rs, unsigned voff, unsigned soff, u32x4 v) {
  asm volatile("buffer_store_dwordx4 %0, %1, %2, %3 offen offset:%4 sc0 sc1\n\ts_nop 1" ::"v"(v), "v"(voff), "s"(rs), "s"(soff), "i"(OFF) : "memory");
}
template <int OFF> __device__ __forceinline__ void bglds_sys(unsigned m0, __amdgpu_buffer_rsrc_t rs, unsigned voff, unsigned soff) {
  asm volatile("s_mov_b32 m0, %0\n\ts_nop 0\n\tbuffer_load_dwordx4 %1, %2, %3 offen offset:%4 sc0 sc1 lds" ::"s"(m0), "v"(voff), "s"(rs), "s"(soff), "i"(OFF) : "memory");
}
template <int N> __device__ __forceinline__ void wait_vm() { asm volatile("s_waitcnt vmcnt(%0)" ::"i"(N) : "memory"); }
#define SB() __builtin_amdgcn_sched_barrier(0)
#define BAR() do { asm volatile("s_barrier" ::: "memory"); SB(); } while (0)
#define WAIT_LDS() do { __builtin_amdgcn_s_waitcnt(0xC07F); SB(); } while (0)
#define LBAR() do { asm volatile("s_waitcnt lgkmcnt(0)\n\ts_barrier" ::: "memory"); SB(); } while (0)

#define STAMP(i) do {} while (0)
#define STAMP_FLUSH() do {} while (0)

struct Frag { v8i x[2]; v8i w[4]; };

__device__ __forceinline__ void kstep(v4f (&acc)[4][2], const Frag& c, Frag& n, bool mm, unsigned so, unsigned ax0, unsigned ax1, const unsigned (&aw)[4]) {
  v4i xa[2], xb[2]; v2i t[4][4];
#define RX(d, a) d = *(LAS v4i*)(size_t)(a)
#define RW(nb, u, off) t[nb][u] = __builtin_amdgcn_ds_read_tr8_b64_v2i32((LAS v2i*)(size_t)(aw[nb] + so + off))
#define MM(nb, mb) if (mm) acc[nb][mb] = __builtin_amdgcn_mfma_scale_f32_16x16x128_f8f6f4(c.w[nb], c.x[mb], acc[nb][mb], 0, 0, 0, 0, 0, 0)
  MM(0, 0); SB();
  RX(xa[0], ax0 + so); RX(xb[0], ax1 + so); RW(0, 0, 0); SB();
  MM(0, 1); SB();
  RW(0, 1, 1024); RW(0, 2, 8192); RW(0, 3, 9216); SB();
  MM(1, 0); SB();
  RX(xa[1], ax0 + so + 2048); RX(xb[1], ax1 + so + 2048); RW(1, 0, 0); SB();
  MM(1, 1); SB();
  RW(1, 1, 1024); RW(1, 2, 8192); RW(1, 3, 9216); SB();
  MM(2, 0); SB();
  RW(2, 0, 0); RW(2, 1, 1024); RW(2, 2, 8192); RW(2, 3, 9216); SB();
  MM(2, 1); SB();
  RW(3, 0, 0); RW(3, 1, 1024); RW(3, 2, 8192); RW(3, 3, 9216); SB();
  MM(3, 0); SB();
  MM(3, 1); SB();
#pragma unroll
  for (int mb = 0; mb < 2; ++mb) n.x[mb] = __builtin_shufflevector(xa[mb], xb[mb], 0, 1, 2, 3, 4, 5, 6, 7);
#pragma unroll
  for (int nb = 0; nb < 4; ++nb) n.w[nb] = v8i{t[nb][0][0], t[nb][0][1], t[nb][1][0], t[nb][1][1], t[nb][2][0], t[nb][2][1], t[nb][3][0], t[nb][3][1]};
  WAIT_LDS();
#undef RX
#undef RW
#undef MM
}

__device__ __forceinline__ unsigned lds_u32(unsigned a) { return *(LAS volatile unsigned*)(size_t)a; }
__device__ __forceinline__ unsigned rfl(unsigned v) { return __builtin_amdgcn_readfirstlane(v); }
template <int CTRL, int ROWS> __device__ __forceinline__ unsigned dpp0(unsigned v) { return __builtin_amdgcn_update_dpp(0u, v, CTRL, ROWS, 0xf, true); }
__device__ __forceinline__ unsigned wave_scan(unsigned v) {
  v += dpp0<0x111, 0xf>(v); v += dpp0<0x112, 0xf>(v); v += dpp0<0x114, 0xf>(v); v += dpp0<0x118, 0xf>(v);
  v += dpp0<0x142, 0xa>(v); v += dpp0<0x143, 0xc>(v);
  return v;
}
__device__ __forceinline__ unsigned wave_max(unsigned v) {
  v = max(v, dpp0<0x111, 0xf>(v)); v = max(v, dpp0<0x112, 0xf>(v)); v = max(v, dpp0<0x114, 0xf>(v)); v = max(v, dpp0<0x118, 0xf>(v));
  v = max(v, dpp0<0x142, 0xa>(v)); v = max(v, dpp0<0x143, 0xc>(v));
  return v;
}
__device__ __forceinline__ unsigned mbcnt64(u64 m) { return __builtin_amdgcn_mbcnt_hi((unsigned)(m >> 32), __builtin_amdgcn_mbcnt_lo((unsigned)m, 0u)); }
__device__ __forceinline__ unsigned scan8(unsigned v) {
  v += dpp0<0x111, 0xf>(v); v += dpp0<0x112, 0xf>(v); v += dpp0<0x114, 0xf>(v);
  return v;
}

__device__ __forceinline__ unsigned wait_flag(GAS u64* flag, GAS unsigned* tmo, unsigned call, unsigned lds_word) {
  unsigned n, hi;
#define POLL(f) "global_load_dwordx2 " f ", %[flag], off sc0 sc1\n\t"
#define TEST(f0, f1) "s_waitcnt vmcnt(4)\n\tv_readfirstlane_b32 %[hi], " f1 "\n\tv_readfirstlane_b32 %[n], " f0 "\n\ts_cmp_ge_u32 %[hi], %[call]\n\ts_cbranch_scc1 2f\n\t"
  asm volatile("s_memrealtime s[96:97]\n\t"
               POLL("v[100:101]") "s_sleep " POLL_GAP "\n\t" POLL("v[102:103]") "s_sleep " POLL_GAP "\n\t" POLL("v[104:105]") "s_sleep " POLL_GAP "\n\t" POLL("v[106:107]")
               "global_load_dword v108, %[tmo], off sc1\n"
               "1:\n\t"
               TEST("v100", "v101") POLL("v[100:101]") TEST("v102", "v103") POLL("v[102:103]")
               TEST("v104", "v105") POLL("v[104:105]") TEST("v106", "v107") POLL("v[106:107]")
               "s_memrealtime s[98:99]\n\ts_waitcnt vmcnt(4) lgkmcnt(0)\n\tv_readfirstlane_b32 %[hi], v108\n\t"
               "global_load_dword v108, %[tmo], off sc1\n\t"
               "s_sub_u32 %[n], s98, s96\n\ts_cmp_gt_u32 %[n], %[dl]\n\ts_cselect_b32 %[hi], %[call], %[hi]\n\t"
               "s_cmp_lg_u32 %[hi], %[call]\n\ts_cbranch_scc1 1b\n\t"
               "s_mov_b32 %[n], 0\n"
               "2:\n\t"
               "v_mov_b32 v109, %[lw]\n\tv_mov_b32 v110, %[n]\n\tds_write_b32 v109, v110\n\t"
               "s_waitcnt lgkmcnt(0)\n\ts_barrier\n\ts_waitcnt vmcnt(0)"
               : [n] "=&s"(n), [hi] "=&s"(hi)
               : [flag] "v"(flag), [tmo] "v"(tmo), [call] "s"(call), [dl] "s"((unsigned)DEADLINE), [lw] "s"(lds_word)
               : "memory", "scc", "s96", "s97", "s98", "s99", "v100", "v101", "v102", "v103", "v104", "v105", "v106", "v107", "v108", "v109", "v110");
#undef POLL
#undef TEST
  return n;
}

__global__ void __launch_bounds__(512) fused_gemm_a2a(Args a) {
  extern __shared__ __attribute__((aligned(16))) char lds[];
  const int tid = threadIdx.x, l = tid & 63;
  const int wv = __builtin_amdgcn_readfirstlane(tid >> 6);
  const int j = blockIdx.x & 7, idx = blockIdx.x >> 3;
  const int rb = idx & 7, cb = idx >> 3;
  const unsigned dst = (unsigned)(a.rank + j) & 7u, src = (unsigned)(a.rank - j) & 7u;
  const bool pull = (PULLMASK >> idx) & 1u;
  const unsigned ldsb = (unsigned)(size_t)lds;
  char* const me = a.ws[0];
  char* const peer = a.ws[j];
  STAMP(0);
  const __amdgpu_buffer_rsrc_t rsW = __builtin_amdgcn_make_buffer_rsrc((void*)a.w, 0, KD * ND, 0x00020000);
  const __amdgpu_buffer_rsrc_t rsX = __builtin_amdgcn_make_buffer_rsrc((void*)a.x, 0, MS * KD, 0x00020000);
  {
    const unsigned pvx = (unsigned)((l >> 3) * KD + 16 * (l & 7)), pvw = (unsigned)((l >> 4) * ND + 16 * (l & 15));
    if (PF0) {
      const int id = idx * 8 + wv;
      const unsigned ox = (unsigned)(8 * (id & 63) * KD + 128 * (id >> 6));
      glds(ldsb + LDS_MISC + 128, pvx, rsX, ox);
      glds(ldsb + LDS_MISC + 128, pvw, rsW, (unsigned)(4 * id * ND) + NS * dst);
      glds(ldsb + LDS_MISC + 128, pvx, rsX, ox + 512);
    }
    if (__builtin_expect(cb >= 2, 0)) {
      const int id = (idx - 16) * 8 + wv;
      const unsigned ox0 = (unsigned)(8 * (id & 63) * KD + 128 * (id >> 6)), ow = (unsigned)(4 * id * ND) + NS * dst;
#pragma unroll 1
      for (int t = PF0 ? 2 : 0; t < 8; ++t) {
        wait_vm<PF_DEPTH>();
        glds(ldsb + LDS_MISC + 128, pvx, rsX, ox0 + 512 * t);
        glds(ldsb + LDS_MISC + 128, pvw, rsW, ow + (unsigned)t * (512 * ND));
        glds(ldsb + LDS_MISC + 128, pvx, rsX, ox0 + 512 * t + 256);
      }
      if (PF == 2) {
#pragma unroll 1
        for (int t = 0; t < PF_W; ++t) {
          wait_vm<PF_DEPTH + 2>();
          glds(ldsb + LDS_MISC + 128, pvw, rsW, ow + (unsigned)t * (512 * ND) + 256);
        }
      }
    }
  }
  LAS volatile unsigned* misc = (LAS volatile unsigned*)(size_t)(ldsb + LDS_MISC);

  unsigned call, us, fl_out = 0;
  __amdgpu_buffer_rsrc_t rsT;
  auto setup = [&](unsigned m0, unsigned m2) {
    call = rfl(m0) + 1; us = rfl(m2);
    const size_t slot = ((size_t)((call & 1u) * NR + (pull ? dst : a.rank)) * 32 + idx) * SLOT;
    rsT = __builtin_amdgcn_make_buffer_rsrc((pull ? me + WS_OUT : peer + WS_IN) + slot, 0, SLOT, 0x00020000);
  };
  auto read_block = [&](u32x4 (&d)[4], unsigned la, unsigned nb) {
#pragma unroll
    for (int i = 0; i < 4; ++i) d[i] = *(LAS u32x4*)(size_t)(la + 16 * i * nb);
  };
  auto send_block = [&](const u32x4 (&d)[4], unsigned g0, unsigned nb, unsigned vo, bool raw) {
    if (raw) {
#pragma unroll
      for (int i = 0; i < 4; ++i) bst16_sys<0>(rsT, vo, T_BLK + 64 * g0 + 16 * i * nb, d[i]);
    } else {
      u32x4 w[3];
#pragma unroll
      for (int i = 0; i < 3; ++i) {
#pragma unroll
        for (int e = 0; e < 4; ++e) w[i][e] = __builtin_amdgcn_perm(d[i][e], d[3][e], 0x07060503u - i);
        if (i == 2 && !pull) wait_vm<1>();
        bst16_sys<0>(rsT, vo, T_BLK + 48 * g0 + 16 * i * nb, w[i]);
      }
      asm volatile("" :: "v"(w[0]), "v"(w[1]), "v"(w[2]));
    }
  };
  auto move_block = [&](unsigned g0, unsigned ng, bool raw) {
    const unsigned nb = min(64u, ng - g0);
    u32x4 d[4];
    read_block(d, ldsb + L_TX + 64 * g0 + 16 * l, nb);
    send_block(d, g0, nb, l < nb ? 16 * l : (unsigned)SLOT, raw);
  };

  if (__builtin_expect(wv < 4, 1)) {
    const int xr = 16 * wv + (l >> 3);
    const unsigned vx0 = (unsigned)(xr * KD + 16 * ((l & 7) ^ ((xr >> 1) & 7)));
    const unsigned vx1 = (unsigned)((xr + 8) * KD + 16 * ((l & 7) ^ (((xr + 8) >> 1) & 7))) - 1024;
    const unsigned cw = (unsigned)((l & 7) ^ ((l >> 4) & 3));
    const unsigned vw0 = (unsigned)((l >> 3) * ND) + 16 * cw, vw1 = vw0 + 8 * ND - 1024;
    const unsigned vw2 = (unsigned)((l >> 3) * ND) + 16 * (cw ^ 4) + 16 * ND - 2048, vw3 = vw2 + 8 * ND - 1024;
    const unsigned k0 = (PF == 3 && cb >= 2) ? 4 * rb : 0;
    unsigned left = 32 - k0;
    unsigned sx = (unsigned)(64 * rb * KD) + 128 * k0, sw = (unsigned)(32 * wv * ND) + NS * dst + 128 * cb + k0 * (128 * ND);
    unsigned mx = ldsb + wv * 2048;
    const unsigned dmw = 8192 + wv * 2048, mend = mx + DEPTH * STAGE;
    auto issue = [&]() {
      asm volatile("s_mov_b32 m0, %0\n\ts_nop 0\n\tbuffer_load_dwordx4 %4, %10, %1 offen lds\n\tbuffer_load_dwordx4 %5, %10, %1 offen offset:1024 lds\n\t"
                   "s_add_u32 m0, %0, %3\n\ts_addk_i32 %0, 0x6000\n\t"
                   "buffer_load_dwordx4 %6, %11, %2 offen lds\n\tbuffer_load_dwordx4 %7, %11, %2 offen offset:1024 lds\n\t"
                   "buffer_load_dwordx4 %8, %11, %2 offen offset:2048 lds\n\tbuffer_load_dwordx4 %9, %11, %2 offen offset:3072 lds\n\t"
                   "s_addk_i32 %1, 0x80\n\ts_add_u32 %2, %2, 0x80000"
                   : "+s"(mx), "+s"(sx), "+s"(sw)
                   : "s"(dmw), "v"(vx0), "v"(vx1), "v"(vw0), "v"(vw1), "v"(vw2), "v"(vw3), "s"(rsX), "s"(rsW) : "memory", "scc");
      if (mx == mend) mx -= DEPTH * STAGE;
      if (PF == 3 && --left == 0) { sx -= KD; sw -= (unsigned)KD * ND; }
    };
    static_assert(STAGE == 0x6000 && 128 * ND == 0x80000, "");
    STAMP(12);
#pragma unroll
    for (int i = 0; i < DEPTH - 1; ++i) issue();
    STAMP(13);
#pragma unroll 1
    for (int s = 0; s < NSTEP - (DEPTH - 1); ++s) {
      wait_vm<(DEPTH - 2) * 6>();
      BAR();
      issue();
    }
    if (DEPTH >= 6) { wait_vm<24>(); BAR(); }
    if (DEPTH >= 5) { wait_vm<18>(); BAR(); }
    if (DEPTH >= 4) { wait_vm<12>(); BAR(); }
    wait_vm<6>(); BAR();
    setup(misc[0], misc[2]);
    asm volatile("" : "+s"(call), "+s"(us), "+s"(rsT));
    wait_vm<0>(); BAR();
    BAR();
    if (j == 0) { STAMP_FLUSH(); return; }
    BAR();
    unsigned ng = (__builtin_amdgcn_readlane(scan8(*(LAS unsigned short*)(size_t)(ldsb + LDS_MISC + 32 + 2 * (l & 7))), 7) + 8 + 15) >> 4;
    const int left_g = (int)ng - 64 * wv;
    unsigned nb = left_g < 0 ? 0 : left_g;
    asm("" : "+s"(nb));
    nb = nb > 64 ? 64 : nb;
    unsigned vo = l < nb ? 16 * l : (unsigned)SLOT;
    unsigned la[5];
#pragma unroll
    for (int i = 0; i < 4; ++i) la[i] = ldsb + L_TX + 4096 * wv + 16 * (i * nb + l);
    la[4] = ldsb + LDS_MISC + 12;
    asm volatile("" : "+s"(ng), "+s"(nb));
    asm volatile("" : "+v"(vo), "+v"(la[0]), "+v"(la[1]), "+v"(la[2]), "+v"(la[3]), "+v"(la[4]));
    BAR();
    unsigned m3 = lds_u32(la[4]);
    u32x4 d[4];
#pragma unroll
    for (int i = 0; i < 4; ++i) d[i] = *(LAS u32x4*)(size_t)la[i];
    SB();
    m3 = rfl(m3);
    const bool raw = m3 & (7u << 17);
    if (__builtin_expect(raw, 0)) {
      BAR();
      ng = (__builtin_amdgcn_readlane(scan8(*(LAS unsigned short*)(size_t)(ldsb + LDS_MISC + 48 + 2 * (l & 7))), 7) + 8 + 15) >> 4;
      BAR();
      if (64u * wv < ng) move_block(64 * wv, ng, true);
    } else {
      send_block(d, 64 * wv, nb, vo, false);
    }
    if (wv == 0 && ng > 512) move_block(512, ng, raw);
    STAMP(5);
    fl_out = ng | m3;
  } else {
    unsigned ticket = 0;
    if (tid == 256) {
      GAS unsigned* ctr = (GAS unsigned*)(me + WS_CTR + blockIdx.x * 128);
      asm volatile("" : "+v"(ctr));
      ticket = __hip_atomic_fetch_add(ctr, 1u, __ATOMIC_RELAXED, __HIP_MEMORY_SCOPE_AGENT);
    }
    float sxv = a.sx[0], swv = a.sw[0];
    const int pw = wv - 4, mh = pw & 1, nh = pw >> 1;
    const int g = l >> 4, lq = (l & 15) >> 1, p = l & 1;
    const unsigned f = (l & 15) >> 1;
    const unsigned ax0 = ldsb + (32 * mh + (l & 15)) * 128 + ((g ^ f) * 16), ax1 = ldsb + (32 * mh + (l & 15)) * 128 + ((g ^ f ^ 4) * 16);
    const unsigned hW = ((lq >> 1) & 3) | ((g & 1) << 2);
    unsigned aw[4];
#pragma unroll
    for (int nb = 0; nb < 4; ++nb) aw[nb] = ldsb + 8192 + (16 * g + lq) * 128 + 8 * p + (((4 * nh + nb) ^ hW) * 16);
    v4f acc[4][2] = {};
    unsigned so = 0, upb = 0, note = 0;
    unsigned a_len = ldsb + LDS_MISC + 32 + 4 * pw + (l == 63 ? 0u : 1u << 20), a_all = ldsb + LDS_MISC + 32 + 2 * (l & 7);
    unsigned a_mask = ldsb + L_TXM + 256 * pw + 2 * l, a_note = ldsb + LDS_MISC + 12 + (l == 0 ? 0u : 1u << 20);
    unsigned a_start = ldsb + L_TX + 8 * pw + 4 * l + (l < 2 ? 0u : 1u << 20);
    asm volatile("" : "+v"(a_len), "+v"(a_all), "+v"(a_mask), "+v"(a_note), "+v"(a_start));
    Frag fa, fb;
    BAR();
    STAMP(1);
    kstep(acc, fb, fa, false, so, ax0, ax1, aw);
    so += STAGE;
#pragma unroll 1
    for (int s = 1; s < 31; s += 2) {
      BAR();
      if (s == 23) {
        float mx = 0.f;
#pragma unroll
        for (int i = 0; i < 32; i += 2) mx = fmaxf(fmaxf(mx, fabsf(acc[i >> 3][(i >> 2) & 1][i & 3])), fabsf(acc[i >> 3][(i >> 2) & 1][(i & 3) + 1]));
        const int sh = min(24 - ((int)(__builtin_amdgcn_readlane(wave_max(__float_as_uint(mx * 0.875f)), 63) >> 23) - 126), 18);
        const unsigned u = rfl(__float_as_uint(sxv * swv));
        const int ec = (int)((u >> 23) & 0xffu);
        const bool plain = sh < 12 || ec == 0xff || ec - (sh + 8) < 1;
        upb = plain ? u : (u & 0x80000000u) | (unsigned)(127 + 8 + sh) << 23;
        note = plain ? 1u << 17 : (unsigned)(sh - 12) << (20 + 3 * pw);
        if (tid == 256) { misc[0] = ticket; misc[2] = u; misc[3] = 0; }
        SB();
      }
      kstep(acc, fa, fb, true, so, ax0, ax1, aw);
      so += STAGE; if (so == DEPTH * STAGE) so = 0;
      BAR();
      kstep(acc, fb, fa, true, so, ax0, ax1, aw);
      so += STAGE; if (so == DEPTH * STAGE) so = 0;
    }
    BAR();
    const unsigned m0 = misc[0], m2 = misc[2];
    kstep(acc, fa, fb, true, so, ax0, ax1, aw);
    BAR();
#pragma unroll
    for (int nb = 0; nb < 4; ++nb)
#pragma unroll
      for (int mb = 0; mb < 2; ++mb) acc[nb][mb] = __builtin_amdgcn_mfma_scale_f32_16x16x128_f8f6f4(fb.w[nb], fb.x[mb], acc[nb][mb], 0, 0, 0, 0, 0, 0);
    setup(m0, m2);
    STAMP(2);
    if (j == 0) {
      const float scale = sxv * swv;
#pragma unroll
      for (int nb = 0; nb < 4; ++nb)
#pragma unroll
        for (int mb = 0; mb < 2; ++mb) {
          v4f y = acc[nb][mb] * scale;
#pragma unroll
          for (int e = 0; e < 4; ++e) y[e] = y[e] <= 0.f ? 0.f : y[e];
          float* o = a.out + (size_t)(MS * a.rank + 64 * rb + 32 * mh + 16 * mb + (l & 15)) * NS + 128 * cb + 64 * nh + 16 * nb + 4 * g;
          st16_sys(o, __builtin_bit_cast(u32x4, y));
        }
      STAMP_FLUSH();
      return;
    }
    const float scale = __uint_as_float(us), up = __uint_as_float(upb);
    v4f t[8];
    unsigned q[32], nz4[32], bits[2];
#pragma unroll
    for (int i = 0; i < 8; ++i) t[i] = acc[i >> 1][i & 1] * up;
#pragma unroll
    for (int r = 0; r < 32; ++r) asm("v_cvt_u32_f32 %0, %1" : "=v"(q[r]) : "v"(t[r >> 2][r & 3]));
#pragma unroll
    for (int r = 0; r < 32; ++r) nz4[r] = min(q[r], 4u);
#pragma unroll
    for (int h = 0; h < 2; ++h) {
      unsigned b[4];
#pragma unroll
      for (int i = 0; i < 4; ++i) {
        const unsigned* z = nz4 + 16 * h + 4 * i;
        b[i] = (z[0] << 1 | z[1]) << 2 | (z[2] << 1 | z[3]);
      }
      bits[h] = ((b[0] << 4 | b[1]) << 8 | (b[2] << 4 | b[3])) >> 2;
    }
    const unsigned c = __builtin_popcount(bits[0]) | __builtin_popcount(bits[1]) << 16;
    const unsigned pre = wave_scan(c);
    *(LAS unsigned*)(size_t)a_len = pre;
    STAMP(3);
    LBAR();
    unsigned zs = *(LAS unsigned short*)(size_t)a_all;
    *(LAS unsigned short*)(size_t)a_mask = (unsigned short)bits[0];
    *(LAS unsigned short*)(size_t)(a_mask + 128) = (unsigned short)bits[1];
    SB();
    const v4f ts = (t[0] + t[2]) + (t[4] + t[6]);
    const float tn = (ts[0] + ts[1]) + (ts[2] + ts[3]) + t[1][0];
    unsigned lo[8];
#pragma unroll
    for (int i = 0; i < 8; ++i) lo[i] = (q[4 * i] | q[4 * i + 1] | q[4 * i + 2] | q[4 * i + 3]) & 0xffu;
    const unsigned cnts = __builtin_amdgcn_readlane(pre, 63);
    unsigned nq = 0;
#pragma unroll
    for (int i = 0; i < 8; ++i) nq += __builtin_popcountll(__builtin_amdgcn_ballot_w64(lo[i] != 0));
    if (__builtin_amdgcn_ballot_w64(tn != tn) != 0) nq = 17;
    note |= nq > 16 ? 1u << 17 : nq << 10;
    SB();
    zs = scan8(zs);
    unsigned nrun = 8 + __builtin_amdgcn_readlane(zs, 7);
    const unsigned off1 = 8 + __builtin_amdgcn_readlane(zs, 2 * pw), off0 = off1 - (cnts & 0xffffu);
    const unsigned db = ldsb + L_TX;
    const unsigned ex = pre - c;
    *(LAS unsigned*)(size_t)a_start = (l == 0 ? off0 : off1) << 10;
    {
      unsigned ah[2] = {db + 4 * (off0 + (ex & 0xffffu)) + (1u << 20), db + 4 * (off1 + (ex >> 16)) + (1u << 20)};
#pragma unroll
      for (int i = 0; i < 8; ++i) {
        unsigned tmp, *z = nz4 + 4 * i;
        asm volatile("v_add_u32 %5, %4, %0\n\tv_mad_i32_i24 %0, %0, %6, %4\n\tv_add_u32 %4, %5, %1\n\tv_mad_i32_i24 %1, %1, %6, %5\n\t"
                     "v_add_u32 %5, %4, %2\n\tv_mad_i32_i24 %2, %2, %6, %4\n\tv_add_u32 %4, %5, %3\n\tv_mad_i32_i24 %3, %3, %6, %5"
                     : "+v"(z[0]), "+v"(z[1]), "+v"(z[2]), "+v"(z[3]), "+v"(ah[i >> 2]), "=&v"(tmp) : "s"(-(1 << 18)));
      }
#pragma unroll
      for (int i = 0; i < 8; ++i) {
        unsigned *z = nz4 + 4 * i, *y = q + 4 * i;
        asm volatile("ds_write_b32 %0, %4\n\tds_write_b32 %1, %5\n\tds_write_b32 %2, %6\n\tds_write_b32 %3, %7"
                     :: "v"(z[0]), "v"(z[1]), "v"(z[2]), "v"(z[3]), "v"(y[0]), "v"(y[1]), "v"(y[2]), "v"(y[3]) : "memory");
      }
    }
    STAMP(4);
    unsigned rec_at;
    asm volatile("ds_add_rtn_u32 %0, %1, %2\n\ts_waitcnt lgkmcnt(0)\n\ts_barrier" : "=&v"(rec_at) : "v"(a_note), "v"(note) : "memory");
    SB();
    const bool raw = rfl(misc[3]) & (7u << 17);
    if (raw) {
      unsigned rb2[2] = {0, 0};
#pragma unroll
      for (int r = 0; r < 32; ++r) {
        q[r] = __float_as_uint(acc[r >> 3][(r >> 2) & 1][r & 3] * scale);
        rb2[r >> 4] = 2 * rb2[r >> 4] + (!(__uint_as_float(q[r]) <= 0.f) ? 1u : 0u);
      }
      const unsigned rc = __builtin_popcount(rb2[0]) | __builtin_popcount(rb2[1]) << 16, rp = wave_scan(rc);
      if (l == 63) *(LAS unsigned*)(size_t)(ldsb + LDS_MISC + 48 + 4 * pw) = rp;
      *(LAS unsigned short*)(size_t)(ldsb + L_TXM + 256 * pw + 2 * l) = (unsigned short)rb2[0];
      *(LAS unsigned short*)(size_t)(ldsb + L_TXM + 256 * pw + 128 + 2 * l) = (unsigned short)rb2[1];
      LBAR();
      const unsigned rz = scan8(*(LAS unsigned short*)(size_t)(ldsb + LDS_MISC + 48 + 2 * (l & 7)));
      nrun = 8 + __builtin_amdgcn_readlane(rz, 7);
      const unsigned o1 = 8 + __builtin_amdgcn_readlane(rz, 2 * pw), o0 = o1 - (__builtin_amdgcn_readlane(rp, 63) & 0xffffu);
      if (l == 0) *(LAS v2i*)(size_t)(db + 8 * pw) = v2i{(int)(o0 << 10), (int)(o1 << 10)};
#pragma unroll
      for (int h = 0; h < 2; ++h) {
        unsigned ad = db + 4 * (h ? o1 + ((rp - rc) >> 16) : o0 + ((rp - rc) & 0xffffu));
#pragma unroll
        for (int r = 0; r < 16; ++r)
          if ((rb2[h] >> (15 - r)) & 1u) {
            *(LAS unsigned*)(size_t)ad = q[16 * h + r];
            ad += 4;
          }
      }
      LBAR();
    }
    const unsigned ng = (nrun + 15) >> 4;
    if (wv == 7) bst16_sys<0>(rsT, 16 * l, 0, *(LAS u32x4*)(size_t)(ldsb + L_TXM + 16 * l));
    if (64u * wv < ng) move_block(64 * wv, ng, raw);
    if (nq && !raw) {
      unsigned sb = T_REC + 32 * (rfl(rec_at) >> 10 & 0x7fu);
#pragma unroll
      for (int i = 0; i < 8; ++i) {
        asm("" : "+v"(lo[i]));
        const u64 b = __builtin_amdgcn_ballot_w64(lo[i] != 0);
        if (b) {
          const unsigned vo = lo[i] ? 32 * mbcnt64(b) : (unsigned)SLOT;
          bst16_sys<0>(rsT, vo, sb, __builtin_bit_cast(u32x4, acc[i >> 1][i & 1] * scale));
          bst16_sys<16>(rsT, vo, sb, u32x4{(unsigned)(l | (8 * pw + i) << 6), 0u, 0u, 0u});
          sb += 32 * __builtin_popcountll(b);
        }
      }
    }
  }
  STAMP(6);
  const unsigned par = call & 1u;
  GAS u64* flag_out = (GAS u64*)(peer + WS_FLAG + ((size_t)(par * NR + a.rank) * 32 + idx) * 128);
  asm volatile("" : "+s"(flag_out), "+s"(fl_out));
  wait_vm<0>();
  BAR();
  if (tid == 0)
    __hip_atomic_store(flag_out, ((u64)call << 32) | fl_out, __ATOMIC_RELAXED, __HIP_MEMORY_SCOPE_SYSTEM);
  STAMP(7);
  {
    GAS unsigned* tmo = (GAS unsigned*)(me + WS_TMO);
    const char* base = pull ? a.ws[(8 - j) & 7] + WS_OUT + ((size_t)(par * NR + a.rank) * 32 + idx) * SLOT
                            : me + WS_IN + ((size_t)(par * NR + src) * 32 + idx) * SLOT;
    float* const ot = a.out + (size_t)(MS * src + 64 * rb + 32 * (wv >> 1 & 1)) * NS + 128 * cb + 64 * (wv >> 2) + 32 * (wv & 1);
    float* o = ot + (l >> 3) * NS + 4 * (l & 7);
    float* oy = o + 8 * NS;
    unsigned from = 4 * ((l >> 3) + 16 * (l & 3));
    float *o1 = o + 16 * NS, *oy1 = oy + 16 * NS;
    unsigned keep = 0xffffff00u;
    asm volatile("" : "+v"(o), "+v"(oy), "+v"(o1), "+v"(oy1), "+v"(from));
    const unsigned db = ldsb + L_RX;
    unsigned s0 = T_BLK + 48 * 64 * wv, sm = 128 * wv, g_first = 64 * wv, lw = ldsb + LDS_MISC + 4, v2l = 2 * l, v16l = 16 * l;
    unsigned paced = pull && PULL_DEPTH == 1 ? (unsigned)SLOT : 0u;
    unsigned r_lane = l + 64 * (7 - wv), r_lds = wv >= 6 ? ldsb + L_REC + 1024 * (7 - wv) : ldsb + LDS_MISC + 128, r_mem = T_REC + 1024 * (7 - wv);
    GAS unsigned* vtmo = tmo;
    __amdgpu_buffer_rsrc_t rs = __builtin_amdgcn_make_buffer_rsrc((void*)base, 0, SLOT, 0x00020000);
    asm volatile("" : "+s"(keep), "+s"(s0), "+s"(sm), "+s"(g_first), "+s"(paced), "+s"(rs), "+s"(r_lds), "+s"(r_mem));
    asm volatile("" : "+v"(lw), "+v"(v2l), "+v"(v16l), "+v"(vtmo), "+v"(r_lane));
    if (wv == 6) {
      GAS u64* flag = (GAS u64*)(me + WS_FLAG + ((size_t)(par * NR + src) * 32 + idx) * 128);
      if (wait_flag(flag, tmo, call, ldsb + LDS_MISC + 4) == 0 && l == 0)
        __hip_atomic_store(tmo, call, __ATOMIC_RELAXED, __HIP_MEMORY_SCOPE_AGENT);
    } else {
      BAR();
    }
    STAMP(8);
    const unsigned fl = rfl(lds_u32(lw));
    const bool raw = fl & (7u << 17);
    const int left_g = (int)(raw ? 0u : fl & 0x3ffu) - (int)g_first;
    unsigned nb0 = left_g < 0 ? 0 : left_g;
    asm("" : "+s"(nb0));
    nb0 = nb0 > 64 ? 64 : nb0;
    const unsigned vb = l < nb0 ? v16l : (unsigned)SLOT, s1 = s0 + 16 * nb0, s2 = s1 + 16 * nb0;
    const unsigned nrec = raw ? 0u : min(fl >> 10 & 0x7fu, 64u);
    unsigned tv, bits;
    u32x4 w[3];
    asm volatile("buffer_load_ushort %1, %6, %9, %10 offen sc0 sc1\n\t"
                 "buffer_load_dwordx4 %2, %7, %9, %11 offen sc0 sc1\n\tbuffer_load_dwordx4 %3, %7, %9, %12 offen sc0 sc1\n\t"
                 "buffer_load_dwordx4 %4, %7, %9, %13 offen sc0 sc1\n\ts_mov_b32 m0, %14\n\tglobal_load_dword %0, %5, off sc1\n\t"
                 "buffer_load_dwordx4 %8, %9, %15 offen sc0 sc1 lds\n\ts_waitcnt vmcnt(5)"
                 : "=&v"(tv), "=&v"(bits), "=&v"(w[0]), "=&v"(w[1]), "=&v"(w[2])
                 : "v"(vtmo), "v"(fl ? v2l : (unsigned)SLOT), "v"(vb), "v"(r_lane < 2 * nrec ? v16l : (unsigned)SLOT),
                   "s"(rs), "s"(sm), "s"(s0), "s"(s1 + paced), "s"(s2 + paced), "s"(r_lds), "s"(r_mem) : "memory");
    const unsigned wo = T_BLK, ng = min(fl & 0x3ffu, (unsigned)(DENSE / 64));
    const unsigned c = __builtin_popcount(bits);
    const unsigned pre = wave_scan(c) - c;
    const unsigned px = __builtin_amdgcn_ds_bpermute(from, pre << 16 | bits), py = __builtin_amdgcn_ds_bpermute(from + 32, pre << 16 | bits);
    u64 m[16];
    unsigned bx = px << ((l & 4) ? 24 : 16), by = py << ((l & 4) ? 24 : 16);
    const unsigned skip = (l & 4) ? 0xff00u : 0u;
    unsigned ax = db + 4 * ((px >> 16) + __builtin_popcount(px & skip)), ay = db + 4 * ((py >> 16) + __builtin_popcount(py & skip));
#pragma unroll
    for (int i = 0; i < 4; ++i)
      asm volatile("v_add_co_u32_e64 %0, %2, %0, %0\n\tv_add_co_u32_e64 %1, %4, %1, %1\n\tv_add_co_u32_e64 %0, %3, %0, %0\n\tv_add_co_u32_e64 %1, %5, %1, %1"
                   : "+v"(bx), "+v"(by), "=s"(m[2 * i]), "=s"(m[2 * i + 1]), "=s"(m[8 + 2 * i]), "=s"(m[9 + 2 * i]));
    asm volatile("s_waitcnt vmcnt(0)" : "+v"(w[0]), "+v"(w[1]), "+v"(w[2]), "+v"(ax), "+v"(ay), "+v"(tv) :: "memory");
    if (paced)
      asm volatile("buffer_load_dwordx4 %0, %2, %3, %4 offen sc0 sc1\n\ts_waitcnt vmcnt(0)\n\tbuffer_load_dwordx4 %1, %2, %3, %5 offen sc0 sc1\n\ts_waitcnt vmcnt(0)"
                   : "=&v"(w[1]), "=&v"(w[2]) : "v"(vb), "s"(rs), "s"(s1), "s"(s2) : "memory");
    auto widen = [&](unsigned g0, unsigned nb) {
      u32x4 d[4];
#pragma unroll
      for (int e = 0; e < 4; ++e) {
#pragma unroll
        for (int i = 0; i < 3; ++i) d[i][e] = w[i][e] & keep;
        d[3][e] = __builtin_amdgcn_perm(w[0][e], __builtin_amdgcn_perm(w[1][e], w[2][e], 0x0c04000cu), 0x0402010cu);
      }
#pragma unroll
      for (int i = 0; i < 4; ++i) *(LAS u32x4*)(size_t)(db + 64 * g0 + 16 * (i * nb + l)) = d[i];
    };
    if (l < nb0) widen(64 * wv, nb0);
    if (raw || ng > 512) {
      for (unsigned g0 = 64 * wv + (raw ? 0 : 512); g0 < ng; g0 += 512) {
        const unsigned nb = min(64u, ng - g0);
        if (l < nb) {
          if (raw) {
#pragma unroll
            for (int i = 0; i < 4; ++i) bglds_sys<0>(db + 64 * g0 + 16 * i * nb, rs, 16 * l, wo + 64 * g0 + 16 * i * nb);
          } else {
            asm volatile("buffer_load_dwordx4 %0, %3, %4, %5 offen sc0 sc1\n\tbuffer_load_dwordx4 %1, %3, %4, %6 offen sc0 sc1\n\t"
                         "buffer_load_dwordx4 %2, %3, %4, %7 offen sc0 sc1\n\ts_waitcnt vmcnt(0)"
                         : "=&v"(w[0]), "=&v"(w[1]), "=&v"(w[2])
                         : "v"(16 * l), "s"(rs), "s"(wo + 48 * g0), "s"(wo + 48 * g0 + 16 * nb), "s"(wo + 48 * g0 + 32 * nb) : "memory");
            widen(g0, nb);
          }
        }
      }
      wait_vm<0>();
    }
    const bool bad = fl == 0 || rfl(tv) == call;
    unsigned down = (us & 0x7fffffffu) - (((fl >> (20 + 3 * (wv >> 1)) & 7u) + 20) << 23);
    asm volatile("" : "+s"(down));
    LBAR();
    STAMP(9);
    if (bad) {
#pragma unroll
      for (int i = 0; i < 4; ++i) st16_sys((i >> 1 ? oy : o) + (i & 1) * 16 * NS, u32x4{0x7fc00000u, 0x7fc00000u, 0x7fc00000u, 0x7fc00000u});
      return;
    }
    u32x4 rec = *(LAS u32x4*)(size_t)(ldsb + L_REC + 32 * l);
    const unsigned id = *(LAS unsigned*)(size_t)(ldsb + L_REC + 32 * l + 16);
#define Z2(a, b) "v_mov_b64 v[" #a ":" #b "], 0\n\t"
#define G2(mx, x, my, y) "s_mov_b64 exec, %" #mx "\n\tds_read_b32 v" #x ", %0\n\tv_add_u32 %0, 4, %0\n\ts_mov_b64 exec, %" #my "\n\tds_read_b32 v" #y ", %1\n\tv_add_u32 %1, 4, %1\n\t"
#define GATHER "ds_read_b32 v116, %24\n\t" Z2(100, 101) Z2(102, 103) Z2(104, 105) Z2(106, 107) Z2(108, 109) Z2(110, 111) Z2(112, 113) Z2(114, 115) \
               "s_waitcnt lgkmcnt(0)\n\tv_lshrrev_b32 v116, 8, v116\n\tv_min_u32 v116, %2, v116\n\tv_add_u32 %0, %0, v116\n\tv_add_u32 %1, %1, v116\n\t" \
               G2(4, 100, 12, 108) G2(5, 101, 13, 109) G2(6, 102, 14, 110) G2(7, 103, 15, 111) \
               G2(8, 104, 16, 112) G2(9, 105, 17, 113) G2(10, 106, 18, 114) G2(11, 107, 19, 115) "s_mov_b64 exec, -1\n\t"
#define CV(a) "v_cvt_f32_u32 v" #a ", v" #a "\n\t"
#define ML(a) "v_mul_f32 v" #a ", %3, v" #a "\n\t"
#define ST4(p, a, d) "global_store_dwordx4 %" #p ", v[" #a ":" #d "], off sc0 sc1\n\t"
#define OUT4(p, a, b, c, d) CV(a) CV(b) CV(c) CV(d) ML(a) ML(b) ML(c) ML(d) ST4(p, a, d)
#define G_OPS : "+v"(ax), "+v"(ay) : "s"(DENSE), "s"(down), "s"(m[0]), "s"(m[1]), "s"(m[2]), "s"(m[3]), "s"(m[4]), "s"(m[5]), "s"(m[6]), "s"(m[7]), \
                "s"(m[8]), "s"(m[9]), "s"(m[10]), "s"(m[11]), "s"(m[12]), "s"(m[13]), "s"(m[14]), "s"(m[15]), \
                "v"(o), "v"(o1), "v"(oy), "v"(oy1), "v"(db + 4 * wv) \
              : "memory", "v100", "v101", "v102", "v103", "v104", "v105", "v106", "v107", "v108", "v109", "v110", "v111", "v112", "v113", "v114", "v115", "v116"
    if (raw) {
      asm volatile(GATHER "s_waitcnt lgkmcnt(0)\n\t" ST4(20, 100, 103) ST4(21, 104, 107) ST4(22, 108, 111) ST4(23, 112, 115) "s_nop 1" G_OPS);
    } else {
      asm volatile(GATHER "s_waitcnt lgkmcnt(8)\n\t" OUT4(20, 100, 101, 102, 103) OUT4(22, 108, 109, 110, 111)
                   "s_waitcnt lgkmcnt(0)\n\t" OUT4(21, 104, 105, 106, 107) OUT4(23, 112, 113, 114, 115) "s_nop 1" G_OPS);
      STAMP(10);
      if (l < nrec && id >> 8 == (unsigned)wv) {
#pragma unroll
        for (int e = 0; e < 4; ++e) rec[e] = __uint_as_float(rec[e]) <= 0.f ? 0u : rec[e];
        st16_sys(ot + ((id & 15u) + (id >> 2 & 16u)) * NS + (id >> 3 & 16u) + (id >> 2 & 12u), rec);
      }
    }
    STAMP(11);
    STAMP_FLUSH();
  }
}

}

extern "C" void kernel_launch(int rank, int nranks, ncclComm_t comm,
                              hipStream_t stream,
                              void* const* d_in, const int* in_sizes, int n_in,
                              void* d_out, int out_size,
                              const int* in_dtypes, int out_dtype,
                              void* const* d_out_all,
                              void* const* d_ws_all, size_t ws_bytes) {
  if (nranks != NR || (unsigned)rank >= (unsigned)NR || n_in != 4 || in_sizes[0] != MS * KD || in_sizes[1] != KD * ND ||
      out_size != NR * MS * NS || ws_bytes < WS_END) {
    fprintf(stderr, "kernel_launch: unexpected problem shape\n");
    return;
  }
  HIP_CHECK(hipFuncSetAttribute((const void*)fused_gemm_a2a, hipFuncAttributeMaxDynamicSharedMemorySize, LDS_BYTES));
  Args a;
  a.x = (const unsigned char*)d_in[0];
  a.w = (const unsigned char*)d_in[1];
  a.sx = (const float*)d_in[2];
  a.sw = (const float*)d_in[3];
  a.out = (float*)d_out;
  for (int p = 0; p < NR; ++p) a.ws[p] = (char*)d_ws_all[(rank + p) % NR];
  a.rank = rank;
  a.pad = 0;
  hipLaunchKernelGGL(fused_gemm_a2a, dim3(256), dim3(512), LDS_BYTES, stream, a);
  HIP_CHECK(hipGetLastError());
}
```

```cpp
#include <hip/hip_runtime.h>
#include <rccl/rccl.h>
#include <cstdio>

#define HIP_CHECK(expr) do { hipError_t _e = (expr); if (_e != hipSuccess) { \
    fprintf(stderr, "HIP error %s at %s:%d\n", hipGetErrorString(_e), __FILE__, __LINE__); \
    return; } } while (0)

namespace {

typedef unsigned u32x4 __attribute__((ext_vector_type(4)));
typedef int v2i __attribute__((ext_vector_type(2)));
typedef int v4i __attribute__((ext_vector_type(4)));
typedef int v8i __attribute__((ext_vector_type(8)));
typedef float v4f __attribute__((ext_vector_type(4)));
typedef unsigned long long u64;
#define LAS __attribute__((address_space(3)))
#define GAS __attribute__((address_space(1)))

constexpr int NR = 8;
constexpr int MS = 512, KD = 4096, ND = 4096, NS = 512;
constexpr int DEPTH = 3, STAGE = 24576, NSTEP = 32;
constexpr int LDS_MISC = 4 * STAGE;
constexpr int LDS_BYTES = LDS_MISC + 128 + 1024;
constexpr int DENSE = 32896;
constexpr int T_REC = 1024, T_BLK = T_REC + 2048;
constexpr int SLOT = T_BLK + DENSE;
constexpr int L_TX = 0, L_RX = DENSE, L_TXM = 2 * DENSE, L_REC = L_TXM + 1024;
static_assert(L_REC + 2048 <= DEPTH * STAGE && SLOT % 128 == 0 && DENSE % 64 == 0, "");
constexpr size_t WS_CTR = 0;
constexpr size_t WS_TMO = 32768;
constexpr size_t WS_FLAG = 65536;
constexpr size_t WS_IN = 1u << 20;
constexpr size_t WS_OUT = WS_IN + (size_t)2 * NR * 32 * SLOT;
constexpr size_t WS_END = WS_OUT + (size_t)2 * NR * 32 * SLOT;
constexpr u64 DEADLINE = 20000000ull * 2;

constexpr unsigned PULLMASK = 0xff000000u;
#define POLL_GAP "2"

struct Args {
  const unsigned char* x;
  const unsigned char* w;
  const float* sx;
  const float* sw;
  float* out;
  char* ws[NR];
  int rank;
  int pad;
};

__device__ __forceinline__ void glds(unsigned m0, unsigned voff, __amdgpu_buffer_rsrc_t rs, unsigned soff) {
  asm volatile("s_mov_b32 m0, %0\n\ts_nop 0\n\tbuffer_load_dwordx4 %1, %2, %3 offen lds" ::"s"(m0), "v"(voff), "s"(rs), "s"(soff) : "memory");
}
__device__ __forceinline__ void st16_sys(void* p, u32x4 v) {
  asm volatile("global_store_dwordx4 %0, %1, off sc0 sc1\n\ts_nop 1" ::"v"(p), "v"(v) : "memory");
}
template <int OFF> __device__ __forceinline__ void bst16_sys(__amdgpu_buffer_rsrc_t rs, unsigned voff, unsigned soff, u32x4 v) {
  asm volatile("buffer_store_dwordx4 %0, %1, %2, %3 offen offset:%4 sc0 sc1\n\ts_nop 1" ::"v"(v), "v"(voff), "s"(rs), "s"(soff), "i"(OFF) : "memory");
}
template <int OFF> __device__ __forceinline__ void bglds_sys(unsigned m0, __amdgpu_buffer_rsrc_t rs, unsigned voff, unsigned soff) {
  asm volatile("s_mov_b32 m0, %0\n\ts_nop 0\n\tbuffer_load_dwordx4 %1, %2, %3 offen offset:%4 sc0 sc1 lds" ::"s"(m0), "v"(voff), "s"(rs), "s"(soff), "i"(OFF) : "memory");
}
template <int N> __device__ __forceinline__ void wait_vm() { asm volatile("s_waitcnt vmcnt(%0)" ::"i"(N) : "memory"); }
#define SB() __builtin_amdgcn_sched_barrier(0)
#define BAR() do { asm volatile("s_barrier" ::: "memory"); SB(); } while (0)
#define WAIT_LDS() do { __builtin_amdgcn_s_waitcnt(0xC07F); SB(); } while (0)
#define LBAR() do { asm volatile("s_waitcnt lgkmcnt(0)\n\ts_barrier" ::: "memory"); SB(); } while (0)

struct Frag { v8i x[2]; v8i w[4]; };

__device__ __forceinline__ void kstep(v4f (&acc)[4][2], const Frag& c, Frag& n, bool mm, unsigned so, unsigned ax0, unsigned ax1, const unsigned (&aw)[4]) {
  v4i xa[2], xb[2]; v2i t[4][4];
#define RX(d, a) d = *(LAS v4i*)(size_t)(a)
#define RW(nb, u, off) t[nb][u] = __builtin_amdgcn_ds_read_tr8_b64_v2i32((LAS v2i*)(size_t)(aw[nb] + so + off))
#define MM(nb, mb) if (mm) acc[nb][mb] = __builtin_amdgcn_mfma_scale_f32_16x16x128_f8f6f4(c.w[nb], c.x[mb], acc[nb][mb], 0, 0, 0, 0, 0, 0)
  MM(0, 0); SB();
  RX(xa[0], ax0 + so); RX(xb[0], ax1 + so); RW(0, 0, 0); SB();
  MM(0, 1); SB();
  RW(0, 1, 1024); RW(0, 2, 8192); RW(0, 3, 9216); SB();
  MM(1, 0); SB();
  RX(xa[1], ax0 + so + 2048); RX(xb[1], ax1 + so + 2048); RW(1, 0, 0); SB();
  MM(1, 1); SB();
  RW(1, 1, 1024); RW(1, 2, 8192); RW(1, 3, 9216); SB();
  MM(2, 0); SB();
  RW(2, 0, 0); RW(2, 1, 1024); RW(2, 2, 8192); RW(2, 3, 9216); SB();
  MM(2, 1); SB();
  RW(3, 0, 0); RW(3, 1, 1024); RW(3, 2, 8192); RW(3, 3, 9216); SB();
  MM(3, 0); SB();
  MM(3, 1); SB();
#pragma unroll
  for (int mb = 0; mb < 2; ++mb) n.x[mb] = __builtin_shufflevector(xa[mb], xb[mb], 0, 1, 2, 3, 4, 5, 6, 7);
#pragma unroll
  for (int nb = 0; nb < 4; ++nb) n.w[nb] = v8i{t[nb][0][0], t[nb][0][1], t[nb][1][0], t[nb][1][1], t[nb][2][0], t[nb][2][1], t[nb][3][0], t[nb][3][1]};
  WAIT_LDS();
#undef RX
#undef RW
#undef MM
}

__device__ __forceinline__ unsigned lds_u32(unsigned a) { return *(LAS volatile unsigned*)(size_t)a; }
__device__ __forceinline__ unsigned rfl(unsigned v) { return __builtin_amdgcn_readfirstlane(v); }
template <int CTRL, int ROWS> __device__ __forceinline__ unsigned dpp0(unsigned v) { return __builtin_amdgcn_update_dpp(0u, v, CTRL, ROWS, 0xf, true); }
__device__ __forceinline__ unsigned wave_scan(unsigned v) {
  v += dpp0<0x111, 0xf>(v); v += dpp0<0x112, 0xf>(v); v += dpp0<0x114, 0xf>(v); v += dpp0<0x118, 0xf>(v);
  v += dpp0<0x142, 0xa>(v); v += dpp0<0x143, 0xc>(v);
  return v;
}
__device__ __forceinline__ unsigned wave_max(unsigned v) {
  v = max(v, dpp0<0x111, 0xf>(v)); v = max(v, dpp0<0x112, 0xf>(v)); v = max(v, dpp0<0x114, 0xf>(v)); v = max(v, dpp0<0x118, 0xf>(v));
  v = max(v, dpp0<0x142, 0xa>(v)); v = max(v, dpp0<0x143, 0xc>(v));
  return v;
}
__device__ __forceinline__ unsigned mbcnt64(u64 m) { return __builtin_amdgcn_mbcnt_hi((unsigned)(m >> 32), __builtin_amdgcn_mbcnt_lo((unsigned)m, 0u)); }
__device__ __forceinline__ unsigned scan8(unsigned v) {
  v += dpp0<0x111, 0xf>(v); v += dpp0<0x112, 0xf>(v); v += dpp0<0x114, 0xf>(v);
  return v;
}

__device__ __forceinline__ unsigned wait_flag(GAS u64* flag, GAS unsigned* tmo, unsigned call, unsigned lds_word) {
  unsigned n, hi;
#define POLL(f) "global_load_dwordx2 " f ", %[flag], off sc0 sc1\n\t"
#define TEST(f0, f1) "s_waitcnt vmcnt(4)\n\tv_readfirstlane_b32 %[hi], " f1 "\n\tv_readfirstlane_b32 %[n], " f0 "\n\ts_cmp_ge_u32 %[hi], %[call]\n\ts_cbranch_scc1 2f\n\t"
  asm volatile("s_memrealtime s[96:97]\n\t"
               POLL("v[100:101]") "s_sleep " POLL_GAP "\n\t" POLL("v[102:103]") "s_sleep " POLL_GAP "\n\t" POLL("v[104:105]") "s_sleep " POLL_GAP "\n\t" POLL("v[106:107]")
               "global_load_dword v108, %[tmo], off sc1\n"
               "1:\n\t"
               TEST("v100", "v101") POLL("v[100:101]") TEST("v102", "v103") POLL("v[102:103]")
               TEST("v104", "v105") POLL("v[104:105]") TEST("v106", "v107") POLL("v[106:107]")
               "s_memrealtime s[98:99]\n\ts_waitcnt vmcnt(4) lgkmcnt(0)\n\tv_readfirstlane_b32 %[hi], v108\n\t"
               "global_load_dword v108, %[tmo], off sc1\n\t"
               "s_sub_u32 %[n], s98, s96\n\ts_cmp_gt_u32 %[n], %[dl]\n\ts_cselect_b32 %[hi], %[call], %[hi]\n\t"
               "s_cmp_lg_u32 %[hi], %[call]\n\ts_cbranch_scc1 1b\n\t"
               "s_mov_b32 %[n], 0\n"
               "2:\n\t"
               "v_mov_b32 v109, %[lw]\n\tv_mov_b32 v110, %[n]\n\tds_write_b32 v109, v110\n\t"
               "s_waitcnt lgkmcnt(0)\n\ts_barrier\n\ts_waitcnt vmcnt(0)"
               : [n] "=&s"(n), [hi] "=&s"(hi)
               : [flag] "v"(flag), [tmo] "v"(tmo), [call] "s"(call), [dl] "s"((unsigned)DEADLINE), [lw] "s"(lds_word)
               : "memory", "scc", "s96", "s97", "s98", "s99", "v100", "v101", "v102", "v103", "v104", "v105", "v106", "v107", "v108", "v109", "v110");
#undef POLL
#undef TEST
  return n;
}

__global__ void __launch_bounds__(512) fused_gemm_a2a(Args a) {
  extern __shared__ __attribute__((aligned(16))) char lds[];
  const int tid = threadIdx.x, l = tid & 63;
  const int wv = __builtin_amdgcn_readfirstlane(tid >> 6);
  const int j = blockIdx.x & 7, idx = blockIdx.x >> 3;
  const int rb = idx & 7, cb = idx >> 3;
  const unsigned dst = (unsigned)(a.rank + j) & 7u, src = (unsigned)(a.rank - j) & 7u;
  const bool pull = (PULLMASK >> idx) & 1u;
  const unsigned ldsb = (unsigned)(size_t)lds;
  char* const me = a.ws[0];
  char* const peer = a.ws[j];
  const __amdgpu_buffer_rsrc_t rsW = __builtin_amdgcn_make_buffer_rsrc((void*)a.w, 0, KD * ND, 0x00020000);
  const __amdgpu_buffer_rsrc_t rsX = __builtin_amdgcn_make_buffer_rsrc((void*)a.x, 0, MS * KD, 0x00020000);
  if (__builtin_expect(cb >= 2, 0)) {
    const unsigned pvx = (unsigned)((l >> 3) * KD + 16 * (l & 7)), pvw = (unsigned)((l >> 4) * ND + 16 * (l & 15));
#pragma unroll 1
    for (int t = (idx - 16) * 8 + wv; t < 96 * 32; t += 128) {
      const int kb = (t * 43691) >> 22, pc = t - 96 * kb;
      wait_vm<6>();
      if (pc < 64) glds(ldsb + LDS_MISC + 128, pvx, rsX, (unsigned)(8 * pc * KD + 128 * kb));
      else glds(ldsb + LDS_MISC + 128, pvw, rsW, (unsigned)((128 * kb + 4 * (pc - 64)) * ND) + NS * dst);
    }
  }
  LAS volatile unsigned* misc = (LAS volatile unsigned*)(size_t)(ldsb + LDS_MISC);

  unsigned call, us, fl_out = 0;
  __amdgpu_buffer_rsrc_t rsT;
  auto setup = [&](unsigned m0, unsigned m2) {
    call = rfl(m0) + 1; us = rfl(m2);
    const size_t slot = ((size_t)((call & 1u) * NR + (pull ? dst : a.rank)) * 32 + idx) * SLOT;
    rsT = __builtin_amdgcn_make_buffer_rsrc((pull ? me + WS_OUT : peer + WS_IN) + slot, 0, SLOT, 0x00020000);
  };
  auto read_block = [&](u32x4 (&d)[4], unsigned la, unsigned nb) {
#pragma unroll
    for (int i = 0; i < 4; ++i) d[i] = *(LAS u32x4*)(size_t)(la + 16 * i * nb);
  };
  auto send_block = [&](const u32x4 (&d)[4], unsigned g0, unsigned nb, unsigned vo, bool raw) {
    if (raw) {
#pragma unroll
      for (int i = 0; i < 4; ++i) bst16_sys<0>(rsT, vo, T_BLK + 64 * g0 + 16 * i * nb, d[i]);
    } else {
      u32x4 w[3];
#pragma unroll
      for (int i = 0; i < 3; ++i) {
#pragma unroll
        for (int e = 0; e < 4; ++e) w[i][e] = __builtin_amdgcn_perm(d[i][e], d[3][e], 0x07060503u - i);
        if (i == 2 && !pull) wait_vm<1>();
        bst16_sys<0>(rsT, vo, T_BLK + 48 * g0 + 16 * i * nb, w[i]);
      }
      asm volatile("" :: "v"(w[0]), "v"(w[1]), "v"(w[2]));
    }
  };
  auto move_block = [&](unsigned g0, unsigned ng, bool raw) {
    const unsigned nb = min(64u, ng - g0);
    u32x4 d[4];
    read_block(d, ldsb + L_TX + 64 * g0 + 16 * l, nb);
    send_block(d, g0, nb, l < nb ? 16 * l : (unsigned)SLOT, raw);
  };

  if (__builtin_expect(wv < 4, 1)) {
    const int xr = 16 * wv + (l >> 3);
    const unsigned vx0 = (unsigned)(xr * KD + 16 * ((l & 7) ^ ((xr >> 1) & 7)));
    const unsigned vx1 = (unsigned)((xr + 8) * KD + 16 * ((l & 7) ^ (((xr + 8) >> 1) & 7))) - 1024;
    const unsigned cw = (unsigned)((l & 7) ^ ((l >> 4) & 3));
    const unsigned vw0 = (unsigned)((l >> 3) * ND) + 16 * cw, vw1 = vw0 + 8 * ND - 1024;
    const unsigned vw2 = (unsigned)((l >> 3) * ND) + 16 * (cw ^ 4) + 16 * ND - 2048, vw3 = vw2 + 8 * ND - 1024;
    const unsigned k0 = cb >= 2 ? 4 * rb : 0;
    unsigned left = 32 - k0;
    unsigned sx = (unsigned)(64 * rb * KD) + 128 * k0, sw = (unsigned)(32 * wv * ND) + NS * dst + 128 * cb + k0 * (128 * ND);
    unsigned mx = ldsb + wv * 2048;
    const unsigned dmw = 8192 + wv * 2048, mend = mx + DEPTH * STAGE;
    auto issue = [&]() {
      asm volatile("s_mov_b32 m0, %0\n\ts_nop 0\n\tbuffer_load_dwordx4 %4, %10, %1 offen lds\n\tbuffer_load_dwordx4 %5, %10, %1 offen offset:1024 lds\n\t"
                   "s_add_u32 m0, %0, %3\n\ts_addk_i32 %0, 0x6000\n\t"
                   "buffer_load_dwordx4 %6, %11, %2 offen lds\n\tbuffer_load_dwordx4 %7, %11, %2 offen offset:1024 lds\n\t"
                   "buffer_load_dwordx4 %8, %11, %2 offen offset:2048 lds\n\tbuffer_load_dwordx4 %9, %11, %2 offen offset:3072 lds\n\t"
                   "s_addk_i32 %1, 0x80\n\ts_add_u32 %2, %2, 0x80000"
                   : "+s"(mx), "+s"(sx), "+s"(sw)
                   : "s"(dmw), "v"(vx0), "v"(vx1), "v"(vw0), "v"(vw1), "v"(vw2), "v"(vw3), "s"(rsX), "s"(rsW) : "memory", "scc");
      if (mx == mend) mx -= DEPTH * STAGE;
      if (--left == 0) { sx -= KD; sw -= (unsigned)KD * ND; }
    };
    static_assert(STAGE == 0x6000 && 128 * ND == 0x80000, "");
#pragma unroll
    for (int i = 0; i < DEPTH - 1; ++i) issue();
#pragma unroll 1
    for (int s = 0; s < NSTEP - (DEPTH - 1); ++s) {
      wait_vm<(DEPTH - 2) * 6>();
      BAR();
      issue();
    }
    static_assert(DEPTH == 3, "the last steps below");
    wait_vm<6>(); BAR();
    setup(misc[0], misc[2]);
    asm volatile("" : "+s"(call), "+s"(us), "+s"(rsT));
    wait_vm<0>(); BAR();
    BAR();
    if (j == 0) return;
    BAR();
    unsigned ng = (__builtin_amdgcn_readlane(scan8(*(LAS unsigned short*)(size_t)(ldsb + LDS_MISC + 32 + 2 * (l & 7))), 7) + 8 + 15) >> 4;
    const int left_g = (int)ng - 64 * wv;
    unsigned nb = left_g < 0 ? 0 : left_g;
    asm("" : "+s"(nb));
    nb = nb > 64 ? 64 : nb;
    unsigned vo = l < nb ? 16 * l : (unsigned)SLOT;
    unsigned la[5];
#pragma unroll
    for (int i = 0; i < 4; ++i) la[i] = ldsb + L_TX + 4096 * wv + 16 * (i * nb + l);
    la[4] = ldsb + LDS_MISC + 12;
    asm volatile("" : "+s"(ng), "+s"(nb));
    asm volatile("" : "+v"(vo), "+v"(la[0]), "+v"(la[1]), "+v"(la[2]), "+v"(la[3]), "+v"(la[4]));
    BAR();
    unsigned m3 = lds_u32(la[4]);
    u32x4 d[4];
#pragma unroll
    for (int i = 0; i < 4; ++i) d[i] = *(LAS u32x4*)(size_t)la[i];
    SB();
    m3 = rfl(m3);
    const bool raw = m3 & (7u << 17);
    if (__builtin_expect(raw, 0)) {
      BAR();
      ng = (__builtin_amdgcn_readlane(scan8(*(LAS unsigned short*)(size_t)(ldsb + LDS_MISC + 48 + 2 * (l & 7))), 7) + 8 + 15) >> 4;
      BAR();
      if (64u * wv < ng) move_block(64 * wv, ng, true);
    } else {
      send_block(d, 64 * wv, nb, vo, false);
    }
    if (wv == 0 && ng > 512) move_block(512, ng, raw);
    fl_out = ng | m3;
  } else {
    unsigned ticket = 0;
    if (tid == 256) {
      GAS unsigned* ctr = (GAS unsigned*)(me + WS_CTR + blockIdx.x * 128);
      asm volatile("" : "+v"(ctr));
      ticket = __hip_atomic_fetch_add(ctr, 1u, __ATOMIC_RELAXED, __HIP_MEMORY_SCOPE_AGENT);
    }
    float sxv = a.sx[0], swv = a.sw[0];
    const int pw = wv - 4, mh = pw & 1, nh = pw >> 1;
    const int g = l >> 4, lq = (l & 15) >> 1, p = l & 1;
    const unsigned f = (l & 15) >> 1;
    const unsigned ax0 = ldsb + (32 * mh + (l & 15)) * 128 + ((g ^ f) * 16), ax1 = ldsb + (32 * mh + (l & 15)) * 128 + ((g ^ f ^ 4) * 16);
    const unsigned hW = ((lq >> 1) & 3) | ((g & 1) << 2);
    unsigned aw[4];
#pragma unroll
    for (int nb = 0; nb < 4; ++nb) aw[nb] = ldsb + 8192 + (16 * g + lq) * 128 + 8 * p + (((4 * nh + nb) ^ hW) * 16);
    v4f acc[4][2] = {};
    unsigned so = 0, upb = 0, note = 0;
    unsigned a_len = ldsb + LDS_MISC + 32 + 4 * pw + (l == 63 ? 0u : 1u << 20), a_all = ldsb + LDS_MISC + 32 + 2 * (l & 7);
    unsigned a_mask = ldsb + L_TXM + 256 * pw + 2 * l, a_note = ldsb + LDS_MISC + 12 + (l == 0 ? 0u : 1u << 20);
    unsigned a_start = ldsb + L_TX + 8 * pw + 4 * l + (l < 2 ? 0u : 1u << 20);
    asm volatile("" : "+v"(a_len), "+v"(a_all), "+v"(a_mask), "+v"(a_note), "+v"(a_start));
    Frag fa, fb;
    BAR();
    kstep(acc, fb, fa, false, so, ax0, ax1, aw);
    so += STAGE;
#pragma unroll 1
    for (int s = 1; s < 31; s += 2) {
      BAR();
      if (s == 23) {
        float mx = 0.f;
#pragma unroll
        for (int i = 0; i < 32; i += 2) mx = fmaxf(fmaxf(mx, fabsf(acc[i >> 3][(i >> 2) & 1][i & 3])), fabsf(acc[i >> 3][(i >> 2) & 1][(i & 3) + 1]));
        const int sh = min(24 - ((int)(__builtin_amdgcn_readlane(wave_max(__float_as_uint(mx * 0.875f)), 63) >> 23) - 126), 18);
        const unsigned u = rfl(__float_as_uint(sxv * swv));
        const int ec = (int)((u >> 23) & 0xffu);
        const bool plain = sh < 12 || ec == 0xff || ec - (sh + 8) < 1;
        upb = plain ? u : (u & 0x80000000u) | (unsigned)(127 + 8 + sh) << 23;
        note = plain ? 1u << 17 : (unsigned)(sh - 12) << (20 + 3 * pw);
        if (tid == 256) { misc[0] = ticket; misc[2] = u; misc[3] = 0; }
        SB();
      }
      kstep(acc, fa, fb, true, so, ax0, ax1, aw);
      so += STAGE; if (so == DEPTH * STAGE) so = 0;
      BAR();
      kstep(acc, fb, fa, true, so, ax0, ax1, aw);
      so += STAGE; if (so == DEPTH * STAGE) so = 0;
    }
    BAR();
    const unsigned m0 = misc[0], m2 = misc[2];
    kstep(acc, fa, fb, true, so, ax0, ax1, aw);
    BAR();
#pragma unroll
    for (int nb = 0; nb < 4; ++nb)
#pragma unroll
      for (int mb = 0; mb < 2; ++mb) acc[nb][mb] = __builtin_amdgcn_mfma_scale_f32_16x16x128_f8f6f4(fb.w[nb], fb.x[mb], acc[nb][mb], 0, 0, 0, 0, 0, 0);
    setup(m0, m2);
    if (j == 0) {
      const float scale = sxv * swv;
#pragma unroll
      for (int nb = 0; nb < 4; ++nb)
#pragma unroll
        for (int mb = 0; mb < 2; ++mb) {
          v4f y = acc[nb][mb] * scale;
#pragma unroll
          for (int e = 0; e < 4; ++e) y[e] = y[e] <= 0.f ? 0.f : y[e];
          float* o = a.out + (size_t)(MS * a.rank + 64 * rb + 32 * mh + 16 * mb + (l & 15)) * NS + 128 * cb + 64 * nh + 16 * nb + 4 * g;
          st16_sys(o, __builtin_bit_cast(u32x4, y));
        }
      return;
    }
    const float scale = __uint_as_float(us), up = __uint_as_float(upb);
    v4f t[8];
    unsigned q[32], nz4[32], bits[2];
#pragma unroll
    for (int i = 0; i < 8; ++i) t[i] = acc[i >> 1][i & 1] * up;
#pragma unroll
    for (int r = 0; r < 32; ++r) asm("v_cvt_u32_f32 %0, %1" : "=v"(q[r]) : "v"(t[r >> 2][r & 3]));
#pragma unroll
    for (int r = 0; r < 32; ++r) nz4[r] = min(q[r], 4u);
#pragma unroll
    for (int h = 0; h < 2; ++h) {
      unsigned b[4];
#pragma unroll
      for (int i = 0; i < 4; ++i) {
        const unsigned* z = nz4 + 16 * h + 4 * i;
        b[i] = (z[0] << 1 | z[1]) << 2 | (z[2] << 1 | z[3]);
      }
      bits[h] = ((b[0] << 4 | b[1]) << 8 | (b[2] << 4 | b[3])) >> 2;
    }
    const unsigned c = __builtin_popcount(bits[0]) | __builtin_popcount(bits[1]) << 16;
    const unsigned pre = wave_scan(c);
    *(LAS unsigned*)(size_t)a_len = pre;
    LBAR();
    unsigned zs = *(LAS unsigned short*)(size_t)a_all;
    *(LAS unsigned short*)(size_t)a_mask = (unsigned short)bits[0];
    *(LAS unsigned short*)(size_t)(a_mask + 128) = (unsigned short)bits[1];
    SB();
    const v4f ts = (t[0] + t[2]) + (t[4] + t[6]);
    const float tn = (ts[0] + ts[1]) + (ts[2] + ts[3]) + t[1][0];
    unsigned lo[8];
#pragma unroll
    for (int i = 0; i < 8; ++i) lo[i] = (q[4 * i] | q[4 * i + 1] | q[4 * i + 2] | q[4 * i + 3]) & 0xffu;
    const unsigned cnts = __builtin_amdgcn_readlane(pre, 63);
    unsigned nq = 0;
#pragma unroll
    for (int i = 0; i < 8; ++i) nq += __builtin_popcountll(__builtin_amdgcn_ballot_w64(lo[i] != 0));
    if (__builtin_amdgcn_ballot_w64(tn != tn) != 0) nq = 17;
    note |= nq > 16 ? 1u << 17 : nq << 10;
    SB();
    zs = scan8(zs);
    unsigned nrun = 8 + __builtin_amdgcn_readlane(zs, 7);
    const unsigned off1 = 8 + __builtin_amdgcn_readlane(zs, 2 * pw), off0 = off1 - (cnts & 0xffffu);
    const unsigned db = ldsb + L_TX;
    const unsigned ex = pre - c;
    *(LAS unsigned*)(size_t)a_start = (l == 0 ? off0 : off1) << 10;
    {
      unsigned ah[2] = {db + 4 * (off0 + (ex & 0xffffu)) + (1u << 20), db + 4 * (off1 + (ex >> 16)) + (1u << 20)};
#pragma unroll
      for (int i = 0; i < 8; ++i) {
        unsigned tmp, *z = nz4 + 4 * i;
        asm volatile("v_add_u32 %5, %4, %0\n\tv_mad_i32_i24 %0, %0, %6, %4\n\tv_add_u32 %4, %5, %1\n\tv_mad_i32_i24 %1, %1, %6, %5\n\t"
                     "v_add_u32 %5, %4, %2\n\tv_mad_i32_i24 %2, %2, %6, %4\n\tv_add_u32 %4, %5, %3\n\tv_mad_i32_i24 %3, %3, %6, %5"
                     : "+v"(z[0]), "+v"(z[1]), "+v"(z[2]), "+v"(z[3]), "+v"(ah[i >> 2]), "=&v"(tmp) : "s"(-(1 << 18)));
      }
#pragma unroll
      for (int i = 0; i < 8; ++i) {
        unsigned *z = nz4 + 4 * i, *y = q + 4 * i;
        asm volatile("ds_write_b32 %0, %4\n\tds_write_b32 %1, %5\n\tds_write_b32 %2, %6\n\tds_write_b32 %3, %7"
                     :: "v"(z[0]), "v"(z[1]), "v"(z[2]), "v"(z[3]), "v"(y[0]), "v"(y[1]), "v"(y[2]), "v"(y[3]) : "memory");
      }
    }
    unsigned rec_at;
    asm volatile("ds_add_rtn_u32 %0, %1, %2\n\ts_waitcnt lgkmcnt(0)\n\ts_barrier" : "=&v"(rec_at) : "v"(a_note), "v"(note) : "memory");
    SB();
    const bool raw = rfl(misc[3]) & (7u << 17);
    if (raw) {
      unsigned rb2[2] = {0, 0};
#pragma unroll
      for (int r = 0; r < 32; ++r) {
        q[r] = __float_as_uint(acc[r >> 3][(r >> 2) & 1][r & 3] * scale);
        rb2[r >> 4] = 2 * rb2[r >> 4] + (!(__uint_as_float(q[r]) <= 0.f) ? 1u : 0u);
      }
      const unsigned rc = __builtin_popcount(rb2[0]) | __builtin_popcount(rb2[1]) << 16, rp = wave_scan(rc);
      if (l == 63) *(LAS unsigned*)(size_t)(ldsb + LDS_MISC + 48 + 4 * pw) = rp;
      *(LAS unsigned short*)(size_t)(ldsb + L_TXM + 256 * pw + 2 * l) = (unsigned short)rb2[0];
      *(LAS unsigned short*)(size_t)(ldsb + L_TXM + 256 * pw + 128 + 2 * l) = (unsigned short)rb2[1];
      LBAR();
      const unsigned rz = scan8(*(LAS unsigned short*)(size_t)(ldsb + LDS_MISC + 48 + 2 * (l & 7)));
      nrun = 8 + __builtin_amdgcn_readlane(rz, 7);
      const unsigned o1 = 8 + __builtin_amdgcn_readlane(rz, 2 * pw), o0 = o1 - (__builtin_amdgcn_readlane(rp, 63) & 0xffffu);
      if (l == 0) *(LAS v2i*)(size_t)(db + 8 * pw) = v2i{(int)(o0 << 10), (int)(o1 << 10)};
#pragma unroll
      for (int h = 0; h < 2; ++h) {
        unsigned ad = db + 4 * (h ? o1 + ((rp - rc) >> 16) : o0 + ((rp - rc) & 0xffffu));
#pragma unroll
        for (int r = 0; r < 16; ++r)
          if ((rb2[h] >> (15 - r)) & 1u) {
            *(LAS unsigned*)(size_t)ad = q[16 * h + r];
            ad += 4;
          }
      }
      LBAR();
    }
    const unsigned ng = (nrun + 15) >> 4;
    if (wv == 7) bst16_sys<0>(rsT, 16 * l, 0, *(LAS u32x4*)(size_t)(ldsb + L_TXM + 16 * l));
    if (64u * wv < ng) move_block(64 * wv, ng, raw);
    if (nq && !raw) {
      unsigned sb = T_REC + 32 * (rfl(rec_at) >> 10 & 0x7fu);
#pragma unroll
      for (int i = 0; i < 8; ++i) {
        asm("" : "+v"(lo[i]));
        const u64 b = __builtin_amdgcn_ballot_w64(lo[i] != 0);
        if (b) {
          const unsigned vo = lo[i] ? 32 * mbcnt64(b) : (unsigned)SLOT;
          bst16_sys<0>(rsT, vo, sb, __builtin_bit_cast(u32x4, acc[i >> 1][i & 1] * scale));
          bst16_sys<16>(rsT, vo, sb, u32x4{(unsigned)(l | (8 * pw + i) << 6), 0u, 0u, 0u});
          sb += 32 * __builtin_popcountll(b);
        }
      }
    }
  }
  const unsigned par = call & 1u;
  GAS u64* flag_out = (GAS u64*)(peer + WS_FLAG + ((size_t)(par * NR + a.rank) * 32 + idx) * 128);
  asm volatile("" : "+s"(flag_out), "+s"(fl_out));
  wait_vm<0>();
  BAR();
  if (tid == 0)
    __hip_atomic_store(flag_out, ((u64)call << 32) | fl_out, __ATOMIC_RELAXED, __HIP_MEMORY_SCOPE_SYSTEM);
  {
    GAS unsigned* tmo = (GAS unsigned*)(me + WS_TMO);
    const char* base = pull ? a.ws[(8 - j) & 7] + WS_OUT + ((size_t)(par * NR + a.rank) * 32 + idx) * SLOT
                            : me + WS_IN + ((size_t)(par * NR + src) * 32 + idx) * SLOT;
    float* const ot = a.out + (size_t)(MS * src + 64 * rb + 32 * (wv >> 1 & 1)) * NS + 128 * cb + 64 * (wv >> 2) + 32 * (wv & 1);
    float* o = ot + (l >> 3) * NS + 4 * (l & 7);
    float* oy = o + 8 * NS;
    unsigned from = 4 * ((l >> 3) + 16 * (l & 3));
    float *o1 = o + 16 * NS, *oy1 = oy + 16 * NS;
    unsigned keep = 0xffffff00u;
    asm volatile("" : "+v"(o), "+v"(oy), "+v"(o1), "+v"(oy1), "+v"(from));
    const unsigned db = ldsb + L_RX;
    unsigned s0 = T_BLK + 48 * 64 * wv, sm = 128 * wv, g_first = 64 * wv, lw = ldsb + LDS_MISC + 4, v2l = 2 * l, v16l = 16 * l;
    unsigned paced = pull ? (unsigned)SLOT : 0u;
    unsigned r_lane = l + 64 * (7 - wv), r_lds = wv >= 6 ? ldsb + L_REC + 1024 * (7 - wv) : ldsb + LDS_MISC + 128, r_mem = T_REC + 1024 * (7 - wv);
    GAS unsigned* vtmo = tmo;
    __amdgpu_buffer_rsrc_t rs = __builtin_amdgcn_make_buffer_rsrc((void*)base, 0, SLOT, 0x00020000);
    asm volatile("" : "+s"(keep), "+s"(s0), "+s"(sm), "+s"(g_first), "+s"(paced), "+s"(rs), "+s"(r_lds), "+s"(r_mem));
    asm volatile("" : "+v"(lw), "+v"(v2l), "+v"(v16l), "+v"(vtmo), "+v"(r_lane));
    if (wv == 6) {
      GAS u64* flag = (GAS u64*)(me + WS_FLAG + ((size_t)(par * NR + src) * 32 + idx) * 128);
      if (wait_flag(flag, tmo, call, ldsb + LDS_MISC + 4) == 0 && l == 0)
        __hip_atomic_store(tmo, call, __ATOMIC_RELAXED, __HIP_MEMORY_SCOPE_AGENT);
    } else {
      BAR();
    }
    const unsigned fl = rfl(lds_u32(lw));
    const bool raw = fl & (7u << 17);
    const int left_g = (int)(raw ? 0u : fl & 0x3ffu) - (int)g_first;
    unsigned nb0 = left_g < 0 ? 0 : left_g;
    asm("" : "+s"(nb0));
    nb0 = nb0 > 64 ? 64 : nb0;
    const unsigned vb = l < nb0 ? v16l : (unsigned)SLOT, s1 = s0 + 16 * nb0, s2 = s1 + 16 * nb0;
    const unsigned nrec = raw ? 0u : min(fl >> 10 & 0x7fu, 64u);
    unsigned tv, bits;
    u32x4 w[3];
    asm volatile("buffer_load_ushort %1, %6, %9, %10 offen sc0 sc1\n\t"
                 "buffer_load_dwordx4 %2, %7, %9, %11 offen sc0 sc1\n\tbuffer_load_dwordx4 %3, %7, %9, %12 offen sc0 sc1\n\t"
                 "buffer_load_dwordx4 %4, %7, %9, %13 offen sc0 sc1\n\ts_mov_b32 m0, %14\n\tglobal_load_dword %0, %5, off sc1\n\t"
                 "buffer_load_dwordx4 %8, %9, %15 offen sc0 sc1 lds\n\ts_waitcnt vmcnt(5)"
                 : "=&v"(tv), "=&v"(bits), "=&v"(w[0]), "=&v"(w[1]), "=&v"(w[2])
                 : "v"(vtmo), "v"(fl ? v2l : (unsigned)SLOT), "v"(vb), "v"(r_lane < 2 * nrec ? v16l : (unsigned)SLOT),
                   "s"(rs), "s"(sm), "s"(s0), "s"(s1 + paced), "s"(s2 + paced), "s"(r_lds), "s"(r_mem) : "memory");
    const unsigned wo = T_BLK, ng = min(fl & 0x3ffu, (unsigned)(DENSE / 64));
    const unsigned c = __builtin_popcount(bits);
    const unsigned pre = wave_scan(c) - c;
    const unsigned px = __builtin_amdgcn_ds_bpermute(from, pre << 16 | bits), py = __builtin_amdgcn_ds_bpermute(from + 32, pre << 16 | bits);
    u64 m[16];
    unsigned bx = px << ((l & 4) ? 24 : 16), by = py << ((l & 4) ? 24 : 16);
    const unsigned skip = (l & 4) ? 0xff00u : 0u;
    unsigned ax = db + 4 * ((px >> 16) + __builtin_popcount(px & skip)), ay = db + 4 * ((py >> 16) + __builtin_popcount(py & skip));
#pragma unroll
    for (int i = 0; i < 4; ++i)
      asm volatile("v_add_co_u32_e64 %0, %2, %0, %0\n\tv_add_co_u32_e64 %1, %4, %1, %1\n\tv_add_co_u32_e64 %0, %3, %0, %0\n\tv_add_co_u32_e64 %1, %5, %1, %1"
                   : "+v"(bx), "+v"(by), "=s"(m[2 * i]), "=s"(m[2 * i + 1]), "=s"(m[8 + 2 * i]), "=s"(m[9 + 2 * i]));
    asm volatile("s_waitcnt vmcnt(0)" : "+v"(w[0]), "+v"(w[1]), "+v"(w[2]), "+v"(ax), "+v"(ay), "+v"(tv) :: "memory");
    if (paced)
      asm volatile("buffer_load_dwordx4 %0, %2, %3, %4 offen sc0 sc1\n\ts_waitcnt vmcnt(0)\n\tbuffer_load_dwordx4 %1, %2, %3, %5 offen sc0 sc1\n\ts_waitcnt vmcnt(0)"
                   : "=&v"(w[1]), "=&v"(w[2]) : "v"(vb), "s"(rs), "s"(s1), "s"(s2) : "memory");
    auto widen = [&](unsigned g0, unsigned nb) {
      u32x4 d[4];
#pragma unroll
      for (int e = 0; e < 4; ++e) {
#pragma unroll
        for (int i = 0; i < 3; ++i) d[i][e] = w[i][e] & keep;
        d[3][e] = __builtin_amdgcn_perm(w[0][e], __builtin_amdgcn_perm(w[1][e], w[2][e], 0x0c04000cu), 0x0402010cu);
      }
#pragma unroll
      for (int i = 0; i < 4; ++i) *(LAS u32x4*)(size_t)(db + 64 * g0 + 16 * (i * nb + l)) = d[i];
    };
    if (l < nb0) widen(64 * wv, nb0);
    if (raw || ng > 512) {
      for (unsigned g0 = 64 * wv + (raw ? 0 : 512); g0 < ng; g0 += 512) {
        const unsigned nb = min(64u, ng - g0);
        if (l < nb) {
          if (raw) {
#pragma unroll
            for (int i = 0; i < 4; ++i) bglds_sys<0>(db + 64 * g0 + 16 * i * nb, rs, 16 * l, wo + 64 * g0 + 16 * i * nb);
          } else {
            asm volatile("buffer_load_dwordx4 %0, %3, %4, %5 offen sc0 sc1\n\tbuffer_load_dwordx4 %1, %3, %4, %6 offen sc0 sc1\n\t"
                         "buffer_load_dwordx4 %2, %3, %4, %7 offen sc0 sc1\n\ts_waitcnt vmcnt(0)"
                         : "=&v"(w[0]), "=&v"(w[1]), "=&v"(w[2])
                         : "v"(16 * l), "s"(rs), "s"(wo + 48 * g0), "s"(wo + 48 * g0 + 16 * nb), "s"(wo + 48 * g0 + 32 * nb) : "memory");
            widen(g0, nb);
          }
        }
      }
      wait_vm<0>();
    }
    const bool bad = fl == 0 || rfl(tv) == call;
    unsigned down = (us & 0x7fffffffu) - (((fl >> (20 + 3 * (wv >> 1)) & 7u) + 20) << 23);
    asm volatile("" : "+s"(down));
    LBAR();
    if (bad) {
#pragma unroll
      for (int i = 0; i < 4; ++i) st16_sys((i >> 1 ? oy : o) + (i & 1) * 16 * NS, u32x4{0x7fc00000u, 0x7fc00000u, 0x7fc00000u, 0x7fc00000u});
      return;
    }
    u32x4 rec = *(LAS u32x4*)(size_t)(ldsb + L_REC + 32 * l);
    const unsigned id = *(LAS unsigned*)(size_t)(ldsb + L_REC + 32 * l + 16);
#define Z2(a, b) "v_mov_b64 v[" #a ":" #b "], 0\n\t"
#define G2(mx, x, my, y) "s_mov_b64 exec, %" #mx "\n\tds_read_b32 v" #x ", %0\n\tv_add_u32 %0, 4, %0\n\ts_mov_b64 exec, %" #my "\n\tds_read_b32 v" #y ", %1\n\tv_add_u32 %1, 4, %1\n\t"
#define GATHER "ds_read_b32 v116, %24\n\t" Z2(100, 101) Z2(102, 103) Z2(104, 105) Z2(106, 107) Z2(108, 109) Z2(110, 111) Z2(112, 113) Z2(114, 115) \
               "s_waitcnt lgkmcnt(0)\n\tv_lshrrev_b32 v116, 8, v116\n\tv_min_u32 v116, %2, v116\n\tv_add_u32 %0, %0, v116\n\tv_add_u32 %1, %1, v116\n\t" \
               G2(4, 100, 12, 108) G2(5, 101, 13, 109) G2(6, 102, 14, 110) G2(7, 103, 15, 111) \
               G2(8, 104, 16, 112) G2(9, 105, 17, 113) G2(10, 106, 18, 114) G2(11, 107, 19, 115) "s_mov_b64 exec, -1\n\t"
#define CV(a) "v_cvt_f32_u32 v" #a ", v" #a "\n\t"
#define ML(a) "v_mul_f32 v" #a ", %3, v" #a "\n\t"
#define ST4(p, a, d) "global_store_dwordx4 %" #p ", v[" #a ":" #d "], off sc0 sc1\n\t"
#define OUT4(p, a, b, c, d) CV(a) CV(b) CV(c) CV(d) ML(a) ML(b) ML(c) ML(d) ST4(p, a, d)
#define G_OPS : "+v"(ax), "+v"(ay) : "s"(DENSE), "s"(down), "s"(m[0]), "s"(m[1]), "s"(m[2]), "s"(m[3]), "s"(m[4]), "s"(m[5]), "s"(m[6]), "s"(m[7]), \
                "s"(m[8]), "s"(m[9]), "s"(m[10]), "s"(m[11]), "s"(m[12]), "s"(m[13]), "s"(m[14]), "s"(m[15]), \
                "v"(o), "v"(o1), "v"(oy), "v"(oy1), "v"(db + 4 * wv) \
              : "memory", "v100", "v101", "v102", "v103", "v104", "v105", "v106", "v107", "v108", "v109", "v110", "v111", "v112", "v113", "v114", "v115", "v116"
    if (raw) {
      asm volatile(GATHER "s_waitcnt lgkmcnt(0)\n\t" ST4(20, 100, 103) ST4(21, 104, 107) ST4(22, 108, 111) ST4(23, 112, 115) "s_nop 1" G_OPS);
    } else {
      asm volatile(GATHER "s_waitcnt lgkmcnt(8)\n\t" OUT4(20, 100, 101, 102, 103) OUT4(22, 108, 109, 110, 111)
                   "s_waitcnt lgkmcnt(0)\n\t" OUT4(21, 104, 105, 106, 107) OUT4(23, 112, 113, 114, 115) "s_nop 1" G_OPS);
      if (l < nrec && id >> 8 == (unsigned)wv) {
#pragma unroll
        for (int e = 0; e < 4; ++e) rec[e] = __uint_as_float(rec[e]) <= 0.f ? 0u : rec[e];
        st16_sys(ot + ((id & 15u) + (id >> 2 & 16u)) * NS + (id >> 3 & 16u) + (id >> 2 & 12u), rec);
      }
    }
  }
}

}

extern "C" void kernel_launch(int rank, int nranks, ncclComm_t comm,
                              hipStream_t stream,
                              void* const* d_in, const int* in_sizes, int n_in,
                              void* d_out, int out_size,
                              const int* in_dtypes, int out_dtype,
                              void* const* d_out_all,
                              void* const* d_ws_all, size_t ws_bytes) {
  if (nranks != NR || (unsigned)rank >= (unsigned)NR || n_in != 4 || in_sizes[0] != MS * KD || in_sizes[1] != KD * ND ||
      out_size != NR * MS * NS || ws_bytes < WS_END) {
    fprintf(stderr, "kernel_launch: unexpected problem shape\n");
    return;
  }
  HIP_CHECK(hipFuncSetAttribute((const void*)fused_gemm_a2a, hipFuncAttributeMaxDynamicSharedMemorySize, LDS_BYTES));
  Args a;
  a.x = (const unsigned char*)d_in[0];
  a.w = (const unsigned char*)d_in[1];
  a.sx = (const float*)d_in[2];
  a.sw = (const float*)d_in[3];
  a.out = (float*)d_out;
  for (int p = 0; p < NR; ++p) a.ws[p] = (char*)d_ws_all[(rank + p) % NR];
  a.rank = rank;
  a.pad = 0;
  hipLaunchKernelGGL(fused_gemm_a2a, dim3(256), dim3(512), LDS_BYTES, stream, a);
  HIP_CHECK(hipGetLastError());
}
```

```cpp
#include <hip/hip_runtime.h>
#include <rccl/rccl.h>
#include <cstdio>

#define HIP_CHECK(expr) do { hipError_t _e = (expr); if (_e != hipSuccess) { \
    fprintf(stderr, "HIP error %s at %s:%d\n", hipGetErrorString(_e), __FILE__, __LINE__); \
    return; } } while (0)

namespace {

typedef unsigned u32x4 __attribute__((ext_vector_type(4)));
typedef int v2i __attribute__((ext_vector_type(2)));
typedef int v4i __attribute__((ext_vector_type(4)));
typedef int v8i __attribute__((ext_vector_type(8)));
typedef float v4f __attribute__((ext_vector_type(4)));
typedef unsigned long long u64;
#define LAS __attribute__((address_space(3)))
#define GAS __attribute__((address_space(1)))

constexpr int NR = 8;
constexpr int MS = 512, KD = 4096, ND = 4096, NS = 512;
constexpr int DEPTH = 3, STAGE = 24576, NSTEP = 32;
constexpr int LDS_MISC = 4 * STAGE;
constexpr int LDS_BYTES = LDS_MISC + 128 + 1024;
constexpr int DENSE = 32896;
constexpr int T_REC = 1024, T_BLK = T_REC + 2048;
constexpr int SLOT = T_BLK + DENSE;
constexpr int L_TX = 0, L_RX = DENSE, L_TXM = 2 * DENSE, L_REC = L_TXM + 1024;
static_assert(L_REC + 2048 <= DEPTH * STAGE && SLOT % 128 == 0 && DENSE % 64 == 0, "");
constexpr size_t WS_CTR = 0;
constexpr size_t WS_TMO = 32768;
constexpr size_t WS_FLAG = 65536;
constexpr size_t WS_IN = 1u << 20;
constexpr size_t WS_OUT = WS_IN + (size_t)2 * NR * 32 * SLOT;
constexpr size_t WS_END = WS_OUT + (size_t)2 * NR * 32 * SLOT;
constexpr u64 DEADLINE = 20000000ull * 2;

constexpr unsigned PULLMASK = 0xff000000u;
#define POLL_GAP "2"

struct Args {
  const unsigned char* x;
  const unsigned char* w;
  const float* sx;
  const float* sw;
  float* out;
  char* ws[NR];
  int rank;
  int pad;
};

__device__ __forceinline__ void glds(unsigned m0, unsigned voff, __amdgpu_buffer_rsrc_t rs, unsigned soff) {
  asm volatile("s_mov_b32 m0, %0\n\ts_nop 0\n\tbuffer_load_dwordx4 %1, %2, %3 offen lds" ::"s"(m0), "v"(voff), "s"(rs), "s"(soff) : "memory");
}
__device__ __forceinline__ void st16_sys(void* p, u32x4 v) {
  asm volatile("global_store_dwordx4 %0, %1, off sc0 sc1\n\ts_nop 1" ::"v"(p), "v"(v) : "memory");
}
template <int OFF> __device__ __forceinline__ void bst16_sys(__amdgpu_buffer_rsrc_t rs, unsigned voff, unsigned soff, u32x4 v) {
  asm volatile("buffer_store_dwordx4 %0, %1, %2, %3 offen offset:%4 sc0 sc1\n\ts_nop 1" ::"v"(v), "v"(voff), "s"(rs), "s"(soff), "i"(OFF) : "memory");
}
template <int OFF> __device__ __forceinline__ void bglds_sys(unsigned m0, __amdgpu_buffer_rsrc_t rs, unsigned voff, unsigned soff) {
  asm volatile("s_mov_b32 m0, %0\n\ts_nop 0\n\tbuffer_load_dwordx4 %1, %2, %3 offen offset:%4 sc0 sc1 lds" ::"s"(m0), "v"(voff), "s"(rs), "s"(soff), "i"(OFF) : "memory");
}
template <int N> __device__ __forceinline__ void wait_vm() { asm volatile("s_waitcnt vmcnt(%0)" ::"i"(N) : "memory"); }
#define SB() __builtin_amdgcn_sched_barrier(0)
#define BAR() do { asm volatile("s_barrier" ::: "memory"); SB(); } while (0)
#define WAIT_LDS() do { __builtin_amdgcn_s_waitcnt(0xC07F); SB(); } while (0)
#define LBAR() do { asm volatile("s_waitcnt lgkmcnt(0)\n\ts_barrier" ::: "memory"); SB(); } while (0)

struct Frag { v8i x[2]; v8i w[4]; };

__device__ __forceinline__ void kstep(v4f (&acc)[4][2], const Frag& c, Frag& n, bool mm, unsigned so, unsigned ax0, unsigned ax1, const unsigned (&aw)[4]) {
  v4i xa[2], xb[2]; v2i t[4][4];
#define RX(d, a) d = *(LAS v4i*)(size_t)(a)
#define RW(nb, u, off) t[nb][u] = __builtin_amdgcn_ds_read_tr8_b64_v2i32((LAS v2i*)(size_t)(aw[nb] + so + off))
#define MM(nb, mb) if (mm) acc[nb][mb] = __builtin_amdgcn_mfma_scale_f32_16x16x128_f8f6f4(c.w[nb], c.x[mb], acc[nb][mb], 0, 0, 0, 0, 0, 0)
  MM(0, 0); SB();
  RX(xa[0], ax0 + so); RX(xb[0], ax1 + so); RW(0, 0, 0); SB();
  MM(0, 1); SB();
  RW(0, 1, 1024); RW(0, 2, 8192); RW(0, 3, 9216); SB();
  MM(1, 0); SB();
  RX(xa[1], ax0 + so + 2048); RX(xb[1], ax1 + so + 2048); RW(1, 0, 0); SB();
  MM(1, 1); SB();
  RW(1, 1, 1024); RW(1, 2, 8192); RW(1, 3, 9216); SB();
  MM(2, 0); SB();
  RW(2, 0, 0); RW(2, 1, 1024); RW(2, 2, 8192); RW(2, 3, 9216); SB();
  MM(2, 1); SB();
  RW(3, 0, 0); RW(3, 1, 1024); RW(3, 2, 8192); RW(3, 3, 9216); SB();
  MM(3, 0); SB();
  MM(3, 1); SB();
#pragma unroll
  for (int mb = 0; mb < 2; ++mb) n.x[mb] = __builtin_shufflevector(xa[mb], xb[mb], 0, 1, 2, 3, 4, 5, 6, 7);
#pragma unroll
  for (int nb = 0; nb < 4; ++nb) n.w[nb] = v8i{t[nb][0][0], t[nb][0][1], t[nb][1][0], t[nb][1][1], t[nb][2][0], t[nb][2][1], t[nb][3][0], t[nb][3][1]};
  WAIT_LDS();
#undef RX
#undef RW
#undef MM
}

__device__ __forceinline__ unsigned lds_u32(unsigned a) { return *(LAS volatile unsigned*)(size_t)a; }
__device__ __forceinline__ unsigned rfl(unsigned v) { return __builtin_amdgcn_readfirstlane(v); }
template <int CTRL, int ROWS> __device__ __forceinline__ unsigned dpp0(unsigned v) { return __builtin_amdgcn_update_dpp(0u, v, CTRL, ROWS, 0xf, true); }
__device__ __forceinline__ unsigned wave_scan(unsigned v) {
  v += dpp0<0x111, 0xf>(v); v += dpp0<0x112, 0xf>(v); v += dpp0<0x114, 0xf>(v); v += dpp0<0x118, 0xf>(v);
  v += dpp0<0x142, 0xa>(v); v += dpp0<0x143, 0xc>(v);
  return v;
}
__device__ __forceinline__ unsigned wave_max(unsigned v) {
  v = max(v, dpp0<0x111, 0xf>(v)); v = max(v, dpp0<0x112, 0xf>(v)); v = max(v, dpp0<0x114, 0xf>(v)); v = max(v, dpp0<0x118, 0xf>(v));
  v = max(v, dpp0<0x142, 0xa>(v)); v = max(v, dpp0<0x143, 0xc>(v));
  return v;
}
__device__ __forceinline__ unsigned mbcnt64(u64 m) { return __builtin_amdgcn_mbcnt_hi((unsigned)(m >> 32), __builtin_amdgcn_mbcnt_lo((unsigned)m, 0u)); }
__device__ __forceinline__ unsigned scan8(unsigned v) {
  v += dpp0<0x111, 0xf>(v); v += dpp0<0x112, 0xf>(v); v += dpp0<0x114, 0xf>(v);
  return v;
}

__device__ __forceinline__ unsigned wait_flag(GAS u64* flag, GAS unsigned* tmo, unsigned call, unsigned lds_word) {
  unsigned n, hi;
#define POLL(f) "global_load_dwordx2 " f ", %[flag], off sc0 sc1\n\t"
#define TEST(f0, f1) "s_waitcnt vmcnt(4)\n\tv_readfirstlane_b32 %[hi], " f1 "\n\tv_readfirstlane_b32 %[n], " f0 "\n\ts_cmp_ge_u32 %[hi], %[call]\n\ts_cbranch_scc1 2f\n\t"
  asm volatile("s_memrealtime s[96:97]\n\t"
               POLL("v[100:101]") "s_sleep " POLL_GAP "\n\t" POLL("v[102:103]") "s_sleep " POLL_GAP "\n\t" POLL("v[104:105]") "s_sleep " POLL_GAP "\n\t" POLL("v[106:107]")
               "global_load_dword v108, %[tmo], off sc1\n"
               "1:\n\t"
               TEST("v100", "v101") POLL("v[100:101]") TEST("v102", "v103") POLL("v[102:103]")
               TEST("v104", "v105") POLL("v[104:105]") TEST("v106", "v107") POLL("v[106:107]")
               "s_memrealtime s[98:99]\n\ts_waitcnt vmcnt(4) lgkmcnt(0)\n\tv_readfirstlane_b32 %[hi], v108\n\t"
               "global_load_dword v108, %[tmo], off sc1\n\t"
               "s_sub_u32 %[n], s98, s96\n\ts_cmp_gt_u32 %[n], %[dl]\n\ts_cselect_b32 %[hi], %[call], %[hi]\n\t"
               "s_cmp_lg_u32 %[hi], %[call]\n\ts_cbranch_scc1 1b\n\t"
               "s_mov_b32 %[n], 0\n"
               "2:\n\t"
               "v_mov_b32 v109, %[lw]\n\tv_mov_b32 v110, %[n]\n\tds_write_b32 v109, v110\n\t"
               "s_waitcnt lgkmcnt(0)\n\ts_barrier\n\ts_waitcnt vmcnt(0)"
               : [n] "=&s"(n), [hi] "=&s"(hi)
               : [flag] "v"(flag), [tmo] "v"(tmo), [call] "s"(call), [dl] "s"((unsigned)DEADLINE), [lw] "s"(lds_word)
               : "memory", "scc", "s96", "s97", "s98", "s99", "v100", "v101", "v102", "v103", "v104", "v105", "v106", "v107", "v108", "v109", "v110");
#undef POLL
#undef TEST
  return n;
}

__global__ void __launch_bounds__(512) fused_gemm_a2a(Args a) {
  extern __shared__ __attribute__((aligned(16))) char lds[];
  const int tid = threadIdx.x, l = tid & 63;
  const int wv = __builtin_amdgcn_readfirstlane(tid >> 6);
  const int j = blockIdx.x & 7, idx = blockIdx.x >> 3;
  const int rb = idx & 7, cb = idx >> 3;
  const unsigned dst = (unsigned)(a.rank + j) & 7u, src = (unsigned)(a.rank - j) & 7u;
  const bool pull = (PULLMASK >> idx) & 1u;
  const unsigned ldsb = (unsigned)(size_t)lds;
  char* const me = a.ws[0];
  char* const peer = a.ws[j];
  const __amdgpu_buffer_rsrc_t rsW = __builtin_amdgcn_make_buffer_rsrc((void*)a.w, 0, KD * ND, 0x00020000);
  const __amdgpu_buffer_rsrc_t rsX = __builtin_amdgcn_make_buffer_rsrc((void*)a.x, 0, MS * KD, 0x00020000);
  if (__builtin_expect(cb >= 2, 0)) {
    const unsigned pvx = (unsigned)((l >> 3) * KD + 16 * (l & 7)), pvw = (unsigned)((l >> 4) * ND + 16 * (l & 15));
#pragma unroll 1
    for (int t = (idx - 16) * 8 + wv; t < 96 * 32; t += 128) {
      const int kb = (t * 43691) >> 22, pc = t - 96 * kb;
      wait_vm<6>();
      if (pc < 64) glds(ldsb + LDS_MISC + 128, pvx, rsX, (unsigned)(8 * pc * KD + 128 * kb));
      else glds(ldsb + LDS_MISC + 128, pvw, rsW, (unsigned)((128 * kb + 4 * (pc - 64)) * ND) + NS * dst);
    }
  }
  LAS volatile unsigned* misc = (LAS volatile unsigned*)(size_t)(ldsb + LDS_MISC);

  unsigned call, us, fl_out = 0;
  __amdgpu_buffer_rsrc_t rsT;
  auto setup = [&](unsigned m0, unsigned m2) {
    call = rfl(m0) + 1; us = rfl(m2);
    const size_t slot = ((size_t)((call & 1u) * NR + (pull ? dst : a.rank)) * 32 + idx) * SLOT;
    rsT = __builtin_amdgcn_make_buffer_rsrc((pull ? me + WS_OUT : peer + WS_IN) + slot, 0, SLOT, 0x00020000);
  };
  auto read_block = [&](u32x4 (&d)[4], unsigned la, unsigned nb) {
#pragma unroll
    for (int i = 0; i < 4; ++i) d[i] = *(LAS u32x4*)(size_t)(la + 16 * i * nb);
  };
  auto send_block = [&](const u32x4 (&d)[4], unsigned g0, unsigned nb, unsigned vo, bool raw, bool bar = false) {
    if (raw) {
#pragma unroll
      for (int i = 0; i < 4; ++i) bst16_sys<0>(rsT, vo, T_BLK + 64 * g0 + 16 * i * nb, d[i]);
    } else {
      u32x4 w[3];
#pragma unroll
      for (int i = 0; i < 3; ++i) {
#pragma unroll
        for (int e = 0; e < 4; ++e) w[i][e] = __builtin_amdgcn_perm(d[i][e], d[3][e], 0x07060503u - i);
        if (i == 2 && bar) BAR();
        if (i == 2 && !pull) wait_vm<1>();
        bst16_sys<0>(rsT, vo, T_BLK + 48 * g0 + 16 * i * nb, w[i]);
      }
      asm volatile("" :: "v"(w[0]), "v"(w[1]), "v"(w[2]));
    }
  };
  auto move_block = [&](unsigned g0, unsigned ng, bool raw) {
    const unsigned nb = min(64u, ng - g0);
    u32x4 d[4];
    read_block(d, ldsb + L_TX + 64 * g0 + 16 * l, nb);
    send_block(d, g0, nb, l < nb ? 16 * l : (unsigned)SLOT, raw);
  };

  if (__builtin_expect(wv < 4, 1)) {
    const int xr = 16 * wv + (l >> 3);
    const unsigned vx0 = (unsigned)(xr * KD + 16 * ((l & 7) ^ ((xr >> 1) & 7)));
    const unsigned vx1 = (unsigned)((xr + 8) * KD + 16 * ((l & 7) ^ (((xr + 8) >> 1) & 7))) - 1024;
    const unsigned cw = (unsigned)((l & 7) ^ ((l >> 4) & 3));
    const unsigned vw0 = (unsigned)((l >> 3) * ND) + 16 * cw, vw1 = vw0 + 8 * ND - 1024;
    const unsigned vw2 = (unsigned)((l >> 3) * ND) + 16 * (cw ^ 4) + 16 * ND - 2048, vw3 = vw2 + 8 * ND - 1024;
    const unsigned k0 = cb >= 2 ? 4 * rb : 0;
    unsigned left = 32 - k0;
    unsigned sx = (unsigned)(64 * rb * KD) + 128 * k0, sw = (unsigned)(32 * wv * ND) + NS * dst + 128 * cb + k0 * (128 * ND);
    unsigned mx = ldsb + wv * 2048;
    const unsigned dmw = 8192 + wv * 2048, mend = mx + DEPTH * STAGE;
    auto issue = [&]() {
      asm volatile("s_mov_b32 m0, %0\n\ts_nop 0\n\tbuffer_load_dwordx4 %4, %10, %1 offen lds\n\tbuffer_load_dwordx4 %5, %10, %1 offen offset:1024 lds\n\t"
                   "s_add_u32 m0, %0, %3\n\ts_addk_i32 %0, 0x6000\n\t"
                   "buffer_load_dwordx4 %6, %11, %2 offen lds\n\tbuffer_load_dwordx4 %7, %11, %2 offen offset:1024 lds\n\t"
                   "buffer_load_dwordx4 %8, %11, %2 offen offset:2048 lds\n\tbuffer_load_dwordx4 %9, %11, %2 offen offset:3072 lds\n\t"
                   "s_addk_i32 %1, 0x80\n\ts_add_u32 %2, %2, 0x80000"
                   : "+s"(mx), "+s"(sx), "+s"(sw)
                   : "s"(dmw), "v"(vx0), "v"(vx1), "v"(vw0), "v"(vw1), "v"(vw2), "v"(vw3), "s"(rsX), "s"(rsW) : "memory", "scc");
      if (mx == mend) mx -= DEPTH * STAGE;
      if (--left == 0) { sx -= KD; sw -= (unsigned)KD * ND; }
    };
    static_assert(STAGE == 0x6000 && 128 * ND == 0x80000, "");
#pragma unroll
    for (int i = 0; i < DEPTH - 1; ++i) issue();
#pragma unroll 1
    for (int s = 0; s < NSTEP - (DEPTH - 1); ++s) {
      wait_vm<(DEPTH - 2) * 6>();
      BAR();
      issue();
    }
    static_assert(DEPTH == 3, "the last steps below");
    wait_vm<6>(); BAR();
    setup(misc[0], misc[2]);
    asm volatile("" : "+s"(call), "+s"(us), "+s"(rsT));
    wait_vm<0>(); BAR();
    BAR();
    if (j == 0) return;
    BAR();
    const unsigned zs = scan8(*(LAS unsigned short*)(size_t)(ldsb + LDS_MISC + 32 + 4 * (l & 3) + 2 * (l >> 2 & 1)));
    unsigned ng = (__builtin_amdgcn_readlane(zs, 7) + 8 + 15) >> 4;
    unsigned nearly = (__builtin_amdgcn_readlane(zs, 3) + 8) >> 10;
    const int left_g = (int)ng - 64 * wv;
    unsigned nb = left_g < 0 ? 0 : left_g;
    asm("" : "+s"(nb));
    nb = nb > 64 ? 64 : nb;
    unsigned vo = l < nb ? 16 * l : (unsigned)SLOT;
    unsigned la[5];
#pragma unroll
    for (int i = 0; i < 4; ++i) la[i] = ldsb + L_TX + 4096 * wv + 16 * (i * nb + l);
    la[4] = ldsb + LDS_MISC + 12;
    asm volatile("" : "+s"(ng), "+s"(nb), "+s"(nearly));
    const bool early = (unsigned)wv < nearly;
    asm volatile("" : "+v"(vo), "+v"(la[0]), "+v"(la[1]), "+v"(la[2]), "+v"(la[3]), "+v"(la[4]));
    BAR();
    unsigned m3 = lds_u32(la[4]);
    if (!early) BAR();
    u32x4 d[4];
#pragma unroll
    for (int i = 0; i < 4; ++i) d[i] = *(LAS u32x4*)(size_t)la[i];
    SB();
    m3 = rfl(m3);
    const bool raw = m3 & (7u << 17);
    if (__builtin_expect(raw, 0)) {
      if (early) BAR();
      BAR();
      ng = (__builtin_amdgcn_readlane(scan8(*(LAS unsigned short*)(size_t)(ldsb + LDS_MISC + 48 + 2 * (l & 7))), 7) + 8 + 15) >> 4;
      BAR();
      if (64u * wv < ng) move_block(64 * wv, ng, true);
    } else {
      send_block(d, 64 * wv, nb, vo, false, early);
    }
    if (wv == 0 && ng > 512) move_block(512, ng, raw);
    fl_out = ng | m3;
  } else {
    unsigned ticket = 0;
    if (tid == 256) {
      GAS unsigned* ctr = (GAS unsigned*)(me + WS_CTR + blockIdx.x * 128);
      asm volatile("" : "+v"(ctr));
      ticket = __hip_atomic_fetch_add(ctr, 1u, __ATOMIC_RELAXED, __HIP_MEMORY_SCOPE_AGENT);
    }
    float sxv = a.sx[0], swv = a.sw[0];
    const int pw = wv - 4, mh = pw & 1, nh = pw >> 1;
    const int g = l >> 4, lq = (l & 15) >> 1, p = l & 1;
    const unsigned f = (l & 15) >> 1;
    const unsigned ax0 = ldsb + (32 * mh + (l & 15)) * 128 + ((g ^ f) * 16), ax1 = ldsb + (32 * mh + (l & 15)) * 128 + ((g ^ f ^ 4) * 16);
    const unsigned hW = ((lq >> 1) & 3) | ((g & 1) << 2);
    unsigned aw[4];
#pragma unroll
    for (int nb = 0; nb < 4; ++nb) aw[nb] = ldsb + 8192 + (16 * g + lq) * 128 + 8 * p + (((4 * nh + nb) ^ hW) * 16);
    v4f acc[4][2] = {};
    unsigned so = 0, upb = 0, note = 0;
    unsigned a_len = ldsb + LDS_MISC + 32 + 4 * pw + (l == 63 ? 0u : 1u << 20), a_all = ldsb + LDS_MISC + 32 + 4 * (l & 3) + 2 * (l >> 2 & 1);
    unsigned a_mask = ldsb + L_TXM + 256 * pw + 2 * l, a_note = ldsb + LDS_MISC + 12 + (l == 0 ? 0u : 1u << 20);
    unsigned a_start = ldsb + L_TX + 8 * pw + 4 * l + (l < 2 ? 0u : 1u << 20);
    asm volatile("" : "+v"(a_len), "+v"(a_all), "+v"(a_mask), "+v"(a_note), "+v"(a_start));
    Frag fa, fb;
    BAR();
    kstep(acc, fb, fa, false, so, ax0, ax1, aw);
    so += STAGE;
#pragma unroll 1
    for (int s = 1; s < 31; s += 2) {
      BAR();
      if (s == 23) {
        float mx = 0.f;
#pragma unroll
        for (int i = 0; i < 32; i += 2) mx = fmaxf(fmaxf(mx, fabsf(acc[i >> 3][(i >> 2) & 1][i & 3])), fabsf(acc[i >> 3][(i >> 2) & 1][(i & 3) + 1]));
        const int sh = min(24 - ((int)(__builtin_amdgcn_readlane(wave_max(__float_as_uint(mx * 0.875f)), 63) >> 23) - 126), 18);
        const unsigned u = rfl(__float_as_uint(sxv * swv));
        const int ec = (int)((u >> 23) & 0xffu);
        const bool plain = sh < 12 || ec == 0xff || ec - (sh + 8) < 1;
        upb = plain ? u : (u & 0x80000000u) | (unsigned)(127 + 8 + sh) << 23;
        note = plain ? 1u << 17 : (unsigned)(sh - 12) << (20 + 3 * pw);
        if (tid == 256) { misc[0] = ticket; misc[2] = u; misc[3] = 0; }
        SB();
      }
      kstep(acc, fa, fb, true, so, ax0, ax1, aw);
      so += STAGE; if (so == DEPTH * STAGE) so = 0;
      BAR();
      kstep(acc, fb, fa, true, so, ax0, ax1, aw);
      so += STAGE; if (so == DEPTH * STAGE) so = 0;
    }
    BAR();
    const unsigned m0 = misc[0], m2 = misc[2];
    kstep(acc, fa, fb, true, so, ax0, ax1, aw);
    BAR();
#pragma unroll
    for (int nb = 0; nb < 4; ++nb)
#pragma unroll
      for (int mb = 0; mb < 2; ++mb) acc[nb][mb] = __builtin_amdgcn_mfma_scale_f32_16x16x128_f8f6f4(fb.w[nb], fb.x[mb], acc[nb][mb], 0, 0, 0, 0, 0, 0);
    setup(m0, m2);
    if (j == 0) {
      const float scale = sxv * swv;
#pragma unroll
      for (int nb = 0; nb < 4; ++nb)
#pragma unroll
        for (int mb = 0; mb < 2; ++mb) {
          v4f y = acc[nb][mb] * scale;
#pragma unroll
          for (int e = 0; e < 4; ++e) y[e] = y[e] <= 0.f ? 0.f : y[e];
          float* o = a.out + (size_t)(MS * a.rank + 64 * rb + 32 * mh + 16 * mb + (l & 15)) * NS + 128 * cb + 64 * nh + 16 * nb + 4 * g;
          st16_sys(o, __builtin_bit_cast(u32x4, y));
        }
      return;
    }
    const float scale = __uint_as_float(us), up = __uint_as_float(upb);
    v4f t[8];
    unsigned q[32], nz4[32], bits[2];
#pragma unroll
    for (int i = 0; i < 8; ++i) t[i] = acc[i >> 1][i & 1] * up;
#pragma unroll
    for (int r = 0; r < 32; ++r) asm("v_cvt_u32_f32 %0, %1" : "=v"(q[r]) : "v"(t[r >> 2][r & 3]));
#pragma unroll
    for (int r = 0; r < 32; ++r) nz4[r] = min(q[r], 4u);
#pragma unroll
    for (int h = 0; h < 2; ++h) {
      unsigned b[4];
#pragma unroll
      for (int i = 0; i < 4; ++i) {
        const unsigned* z = nz4 + 16 * h + 4 * i;
        b[i] = (z[0] << 1 | z[1]) << 2 | (z[2] << 1 | z[3]);
      }
      bits[h] = ((b[0] << 4 | b[1]) << 8 | (b[2] << 4 | b[3])) >> 2;
    }
    const unsigned c = __builtin_popcount(bits[0]) | __builtin_popcount(bits[1]) << 16;
    const unsigned pre = wave_scan(c);
    *(LAS unsigned*)(size_t)a_len = pre;
    LBAR();
    unsigned zs = *(LAS unsigned short*)(size_t)a_all;
    *(LAS unsigned short*)(size_t)a_mask = (unsigned short)bits[0];
    *(LAS unsigned short*)(size_t)(a_mask + 128) = (unsigned short)bits[1];
    SB();
    const v4f ts = (t[0] + t[2]) + (t[4] + t[6]);
    const float tn = (ts[0] + ts[1]) + (ts[2] + ts[3]) + t[1][0];
    unsigned lo[8];
#pragma unroll
    for (int i = 0; i < 8; ++i) lo[i] = (q[4 * i] | q[4 * i + 1] | q[4 * i + 2] | q[4 * i + 3]) & 0xffu;
    const unsigned cnts = __builtin_amdgcn_readlane(pre, 63);
    unsigned nq = 0;
#pragma unroll
    for (int i = 0; i < 8; ++i) nq += __builtin_popcountll(__builtin_amdgcn_ballot_w64(lo[i] != 0));
    if (__builtin_amdgcn_ballot_w64(tn != tn) != 0) nq = 17;
    note |= nq > 16 ? 1u << 17 : nq << 10;
    SB();
    zs = scan8(zs);
    unsigned nrun = 8 + __builtin_amdgcn_readlane(zs, 7);
    const unsigned off0 = 8 + __builtin_amdgcn_readlane(zs, pw) - (cnts & 0xffffu), off1 = 8 + __builtin_amdgcn_readlane(zs, 4 + pw) - (cnts >> 16);
    const unsigned db = ldsb + L_TX;
    const unsigned ex = pre - c;
    *(LAS unsigned*)(size_t)a_start = (l == 0 ? off0 : off1) << 10;
    unsigned rec_at;
    {
      unsigned ah[2] = {db + 4 * (off0 + (ex & 0xffffu)) + (1u << 20), db + 4 * (off1 + (ex >> 16)) + (1u << 20)};
#pragma unroll
      for (int h = 0; h < 2; ++h) {
#pragma unroll
        for (int i = 4 * h; i < 4 * h + 4; ++i) {
          unsigned tmp, *z = nz4 + 4 * i;
          asm volatile("v_add_u32 %5, %4, %0\n\tv_mad_i32_i24 %0, %0, %6, %4\n\tv_add_u32 %4, %5, %1\n\tv_mad_i32_i24 %1, %1, %6, %5\n\t"
                       "v_add_u32 %5, %4, %2\n\tv_mad_i32_i24 %2, %2, %6, %4\n\tv_add_u32 %4, %5, %3\n\tv_mad_i32_i24 %3, %3, %6, %5"
                       : "+v"(z[0]), "+v"(z[1]), "+v"(z[2]), "+v"(z[3]), "+v"(ah[h]), "=&v"(tmp) : "s"(-(1 << 18)));
        }
#pragma unroll
        for (int i = 4 * h; i < 4 * h + 4; ++i) {
          unsigned *z = nz4 + 4 * i, *y = q + 4 * i;
          asm volatile("ds_write_b32 %0, %4\n\tds_write_b32 %1, %5\n\tds_write_b32 %2, %6\n\tds_write_b32 %3, %7"
                       :: "v"(z[0]), "v"(z[1]), "v"(z[2]), "v"(z[3]), "v"(y[0]), "v"(y[1]), "v"(y[2]), "v"(y[3]) : "memory");
        }
        if (h == 0)
          asm volatile("ds_add_rtn_u32 %0, %1, %2\n\ts_waitcnt lgkmcnt(0)\n\ts_barrier" : "=&v"(rec_at) : "v"(a_note), "v"(note) : "memory");
      }
    }
    LBAR();
    SB();
    const bool raw = rfl(misc[3]) & (7u << 17);
    if (raw) {
      unsigned rb2[2] = {0, 0};
#pragma unroll
      for (int r = 0; r < 32; ++r) {
        q[r] = __float_as_uint(acc[r >> 3][(r >> 2) & 1][r & 3] * scale);
        rb2[r >> 4] = 2 * rb2[r >> 4] + (!(__uint_as_float(q[r]) <= 0.f) ? 1u : 0u);
      }
      const unsigned rc = __builtin_popcount(rb2[0]) | __builtin_popcount(rb2[1]) << 16, rp = wave_scan(rc);
      if (l == 63) *(LAS unsigned*)(size_t)(ldsb + LDS_MISC + 48 + 4 * pw) = rp;
      *(LAS unsigned short*)(size_t)(ldsb + L_TXM + 256 * pw + 2 * l) = (unsigned short)rb2[0];
      *(LAS unsigned short*)(size_t)(ldsb + L_TXM + 256 * pw + 128 + 2 * l) = (unsigned short)rb2[1];
      LBAR();
      const unsigned rz = scan8(*(LAS unsigned short*)(size_t)(ldsb + LDS_MISC + 48 + 2 * (l & 7)));
      nrun = 8 + __builtin_amdgcn_readlane(rz, 7);
      const unsigned o1 = 8 + __builtin_amdgcn_readlane(rz, 2 * pw), o0 = o1 - (__builtin_amdgcn_readlane(rp, 63) & 0xffffu);
      if (l == 0) *(LAS v2i*)(size_t)(db + 8 * pw) = v2i{(int)(o0 << 10), (int)(o1 << 10)};
#pragma unroll
      for (int h = 0; h < 2; ++h) {
        unsigned ad = db + 4 * (h ? o1 + ((rp - rc) >> 16) : o0 + ((rp - rc) & 0xffffu));
#pragma unroll
        for (int r = 0; r < 16; ++r)
          if ((rb2[h] >> (15 - r)) & 1u) {
            *(LAS unsigned*)(size_t)ad = q[16 * h + r];
            ad += 4;
          }
      }
      LBAR();
    }
    const unsigned ng = (nrun + 15) >> 4;
    if (wv == 7) bst16_sys<0>(rsT, 16 * l, 0, *(LAS u32x4*)(size_t)(ldsb + L_TXM + 16 * l));
    if (64u * wv < ng) move_block(64 * wv, ng, raw);
    if (nq && !raw) {
      unsigned sb = T_REC + 32 * (rfl(rec_at) >> 10 & 0x7fu);
#pragma unroll
      for (int i = 0; i < 8; ++i) {
        asm("" : "+v"(lo[i]));
        const u64 b = __builtin_amdgcn_ballot_w64(lo[i] != 0);
        if (b) {
          const unsigned vo = lo[i] ? 32 * mbcnt64(b) : (unsigned)SLOT;
          bst16_sys<0>(rsT, vo, sb, __builtin_bit_cast(u32x4, acc[i >> 1][i & 1] * scale));
          bst16_sys<16>(rsT, vo, sb, u32x4{(unsigned)(l | (8 * pw + i) << 6), 0u, 0u, 0u});
          sb += 32 * __builtin_popcountll(b);
        }
      }
    }
  }
  const unsigned par = call & 1u;
  GAS u64* flag_out = (GAS u64*)(peer + WS_FLAG + ((size_t)(par * NR + a.rank) * 32 + idx) * 128);
  asm volatile("" : "+s"(flag_out), "+s"(fl_out));
  wait_vm<0>();
  BAR();
  if (tid == 0)
    __hip_atomic_store(flag_out, ((u64)call << 32) | fl_out, __ATOMIC_RELAXED, __HIP_MEMORY_SCOPE_SYSTEM);
  {
    GAS unsigned* tmo = (GAS unsigned*)(me + WS_TMO);
    const char* base = pull ? a.ws[(8 - j) & 7] + WS_OUT + ((size_t)(par * NR + a.rank) * 32 + idx) * SLOT
                            : me + WS_IN + ((size_t)(par * NR + src) * 32 + idx) * SLOT;
    float* const ot = a.out + (size_t)(MS * src + 64 * rb + 32 * (wv >> 1 & 1)) * NS + 128 * cb + 64 * (wv >> 2) + 32 * (wv & 1);
    float* o = ot + (l >> 3) * NS + 4 * (l & 7);
    float* oy = o + 8 * NS;
    unsigned from = 4 * ((l >> 3) + 16 * (l & 3));
    float *o1 = o + 16 * NS, *oy1 = oy + 16 * NS;
    unsigned keep = 0xffffff00u;
    asm volatile("" : "+v"(o), "+v"(oy), "+v"(o1), "+v"(oy1), "+v"(from));
    const unsigned db = ldsb + L_RX;
    unsigned s0 = T_BLK + 48 * 64 * wv, sm = 128 * wv, g_first = 64 * wv, lw = ldsb + LDS_MISC + 4, v2l = 2 * l, v16l = 16 * l;
    unsigned paced = pull ? (unsigned)SLOT : 0u;
    unsigned r_lane = l + 64 * (7 - wv), r_lds = wv >= 6 ? ldsb + L_REC + 1024 * (7 - wv) : ldsb + LDS_MISC + 128, r_mem = T_REC + 1024 * (7 - wv);
    GAS unsigned* vtmo = tmo;
    __amdgpu_buffer_rsrc_t rs = __builtin_amdgcn_make_buffer_rsrc((void*)base, 0, SLOT, 0x00020000);
    asm volatile("" : "+s"(keep), "+s"(s0), "+s"(sm), "+s"(g_first), "+s"(paced), "+s"(rs), "+s"(r_lds), "+s"(r_mem));
    asm volatile("" : "+v"(lw), "+v"(v2l), "+v"(v16l), "+v"(vtmo), "+v"(r_lane));
    if (wv == 6) {
      GAS u64* flag = (GAS u64*)(me + WS_FLAG + ((size_t)(par * NR + src) * 32 + idx) * 128);
      if (wait_flag(flag, tmo, call, ldsb + LDS_MISC + 4) == 0 && l == 0)
        __hip_atomic_store(tmo, call, __ATOMIC_RELAXED, __HIP_MEMORY_SCOPE_AGENT);
    } else {
      BAR();
    }
    const unsigned fl = rfl(lds_u32(lw));
    const bool raw = fl & (7u << 17);
    const int left_g = (int)(raw ? 0u : fl & 0x3ffu) - (int)g_first;
    unsigned nb0 = left_g < 0 ? 0 : left_g;
    asm("" : "+s"(nb0));
    nb0 = nb0 > 64 ? 64 : nb0;
    const unsigned vb = l < nb0 ? v16l : (unsigned)SLOT, s1 = s0 + 16 * nb0, s2 = s1 + 16 * nb0;
    const unsigned nrec = raw ? 0u : min(fl >> 10 & 0x7fu, 64u);
    unsigned tv, bits;
    u32x4 w[3];
    asm volatile("buffer_load_ushort %1, %6, %9, %10 offen sc0 sc1\n\t"
                 "buffer_load_dwordx4 %2, %7, %9, %11 offen sc0 sc1\n\tbuffer_load_dwordx4 %3, %7, %9, %12 offen sc0 sc1\n\t"
                 "buffer_load_dwordx4 %4, %7, %9, %13 offen sc0 sc1\n\ts_mov_b32 m0, %14\n\tglobal_load_dword %0, %5, off sc1\n\t"
                 "buffer_load_dwordx4 %8, %9, %15 offen sc0 sc1 lds\n\ts_waitcnt vmcnt(5)"
                 : "=&v"(tv), "=&v"(bits), "=&v"(w[0]), "=&v"(w[1]), "=&v"(w[2])
                 : "v"(vtmo), "v"(fl ? v2l : (unsigned)SLOT), "v"(vb), "v"(r_lane < 2 * nrec ? v16l : (unsigned)SLOT),
                   "s"(rs), "s"(sm), "s"(s0), "s"(s1 + paced), "s"(s2 + paced), "s"(r_lds), "s"(r_mem) : "memory");
    const unsigned wo = T_BLK, ng = min(fl & 0x3ffu, (unsigned)(DENSE / 64));
    const unsigned c = __builtin_popcount(bits);
    const unsigned pre = wave_scan(c) - c;
    const unsigned px = __builtin_amdgcn_ds_bpermute(from, pre << 16 | bits), py = __builtin_amdgcn_ds_bpermute(from + 32, pre << 16 | bits);
    u64 m[16];
    unsigned bx = px << ((l & 4) ? 24 : 16), by = py << ((l & 4) ? 24 : 16);
    const unsigned skip = (l & 4) ? 0xff00u : 0u;
    unsigned ax = db + 4 * ((px >> 16) + __builtin_popcount(px & skip)), ay = db + 4 * ((py >> 16) + __builtin_popcount(py & skip));
#pragma unroll
    for (int i = 0; i < 4; ++i)
      asm volatile("v_add_co_u32_e64 %0, %2, %0, %0\n\tv_add_co_u32_e64 %1, %4, %1, %1\n\tv_add_co_u32_e64 %0, %3, %0, %0\n\tv_add_co_u32_e64 %1, %5, %1, %1"
                   : "+v"(bx), "+v"(by), "=s"(m[2 * i]), "=s"(m[2 * i + 1]), "=s"(m[8 + 2 * i]), "=s"(m[9 + 2 * i]));
    asm volatile("s_waitcnt vmcnt(0)" : "+v"(w[0]), "+v"(w[1]), "+v"(w[2]), "+v"(ax), "+v"(ay), "+v"(tv) :: "memory");
    if (paced)
      asm volatile("buffer_load_dwordx4 %0, %2, %3, %4 offen sc0 sc1\n\ts_waitcnt vmcnt(0)\n\tbuffer_load_dwordx4 %1, %2, %3, %5 offen sc0 sc1\n\ts_waitcnt vmcnt(0)"
                   : "=&v"(w[1]), "=&v"(w[2]) : "v"(vb), "s"(rs), "s"(s1), "s"(s2) : "memory");
    auto widen = [&](unsigned g0, unsigned nb) {
      u32x4 d[4];
#pragma unroll
      for (int e = 0; e < 4; ++e) {
#pragma unroll
        for (int i = 0; i < 3; ++i) d[i][e] = w[i][e] & keep;
        d[3][e] = __builtin_amdgcn_perm(w[0][e], __builtin_amdgcn_perm(w[1][e], w[2][e], 0x0c04000cu), 0x0402010cu);
      }
#pragma unroll
      for (int i = 0; i < 4; ++i) *(LAS u32x4*)(size_t)(db + 64 * g0 + 16 * (i * nb + l)) = d[i];
    };
    if (l < nb0) widen(64 * wv, nb0);
    if (raw || ng > 512) {
      for (unsigned g0 = 64 * wv + (raw ? 0 : 512); g0 < ng; g0 += 512) {
        const unsigned nb = min(64u, ng - g0);
        if (l < nb) {
          if (raw) {
#pragma unroll
            for (int i = 0; i < 4; ++i) bglds_sys<0>(db + 64 * g0 + 16 * i * nb, rs, 16 * l, wo + 64 * g0 + 16 * i * nb);
          } else {
            asm volatile("buffer_load_dwordx4 %0, %3, %4, %5 offen sc0 sc1\n\tbuffer_load_dwordx4 %1, %3, %4, %6 offen sc0 sc1\n\t"
                         "buffer_load_dwordx4 %2, %3, %4, %7 offen sc0 sc1\n\ts_waitcnt vmcnt(0)"
                         : "=&v"(w[0]), "=&v"(w[1]), "=&v"(w[2])
                         : "v"(16 * l), "s"(rs), "s"(wo + 48 * g0), "s"(wo + 48 * g0 + 16 * nb), "s"(wo + 48 * g0 + 32 * nb) : "memory");
            widen(g0, nb);
          }
        }
      }
      wait_vm<0>();
    }
    const bool bad = fl == 0 || rfl(tv) == call;
    unsigned down = (us & 0x7fffffffu) - (((fl >> (20 + 3 * (wv >> 1)) & 7u) + 20) << 23);
    asm volatile("" : "+s"(down));
    LBAR();
    if (bad) {
#pragma unroll
      for (int i = 0; i < 4; ++i) st16_sys((i >> 1 ? oy : o) + (i & 1) * 16 * NS, u32x4{0x7fc00000u, 0x7fc00000u, 0x7fc00000u, 0x7fc00000u});
      return;
    }
    u32x4 rec = *(LAS u32x4*)(size_t)(ldsb + L_REC + 32 * l);
    const unsigned id = *(LAS unsigned*)(size_t)(ldsb + L_REC + 32 * l + 16);
#define Z2(a, b) "v_mov_b64 v[" #a ":" #b "], 0\n\t"
#define G2(mx, x, my, y) "s_mov_b64 exec, %" #mx "\n\tds_read_b32 v" #x ", %0\n\tv_add_u32 %0, 4, %0\n\ts_mov_b64 exec, %" #my "\n\tds_read_b32 v" #y ", %1\n\tv_add_u32 %1, 4, %1\n\t"
#define GATHER "ds_read_b32 v116, %24\n\t" Z2(100, 101) Z2(102, 103) Z2(104, 105) Z2(106, 107) Z2(108, 109) Z2(110, 111) Z2(112, 113) Z2(114, 115) \
               "s_waitcnt lgkmcnt(0)\n\tv_lshrrev_b32 v116, 8, v116\n\tv_min_u32 v116, %2, v116\n\tv_add_u32 %0, %0, v116\n\tv_add_u32 %1, %1, v116\n\t" \
               G2(4, 100, 12, 108) G2(5, 101, 13, 109) G2(6, 102, 14, 110) G2(7, 103, 15, 111) \
               G2(8, 104, 16, 112) G2(9, 105, 17, 113) G2(10, 106, 18, 114) G2(11, 107, 19, 115) "s_mov_b64 exec, -1\n\t"
#define CV(a) "v_cvt_f32_u32 v" #a ", v" #a "\n\t"
#define ML(a) "v_mul_f32 v" #a ", %3, v" #a "\n\t"
#define ST4(p, a, d) "global_store_dwordx4 %" #p ", v[" #a ":" #d "], off sc0 sc1\n\t"
#define OUT4(p, a, b, c, d) CV(a) CV(b) CV(c) CV(d) ML(a) ML(b) ML(c) ML(d) ST4(p, a, d)
#define G_OPS : "+v"(ax), "+v"(ay) : "s"(DENSE), "s"(down), "s"(m[0]), "s"(m[1]), "s"(m[2]), "s"(m[3]), "s"(m[4]), "s"(m[5]), "s"(m[6]), "s"(m[7]), \
                "s"(m[8]), "s"(m[9]), "s"(m[10]), "s"(m[11]), "s"(m[12]), "s"(m[13]), "s"(m[14]), "s"(m[15]), \
                "v"(o), "v"(o1), "v"(oy), "v"(oy1), "v"(db + 4 * wv) \
              : "memory", "v100", "v101", "v102", "v103", "v104", "v105", "v106", "v107", "v108", "v109", "v110", "v111", "v112", "v113", "v114", "v115", "v116"
    if (raw) {
      asm volatile(GATHER "s_waitcnt lgkmcnt(0)\n\t" ST4(20, 100, 103) ST4(21, 104, 107) ST4(22, 108, 111) ST4(23, 112, 115) "s_nop 1" G_OPS);
    } else {
      asm volatile(GATHER "s_waitcnt lgkmcnt(8)\n\t" OUT4(20, 100, 101, 102, 103) OUT4(22, 108, 109, 110, 111)
                   "s_waitcnt lgkmcnt(0)\n\t" OUT4(21, 104, 105, 106, 107) OUT4(23, 112, 113, 114, 115) "s_nop 1" G_OPS);
      if (l < nrec && id >> 8 == (unsigned)wv) {
#pragma unroll
        for (int e = 0; e < 4; ++e) rec[e] = __uint_as_float(rec[e]) <= 0.f ? 0u : rec[e];
        st16_sys(ot + ((id & 15u) + (id >> 2 & 16u)) * NS + (id >> 3 & 16u) + (id >> 2 & 12u), rec);
      }
    }
  }
}

}

extern "C" void kernel_launch(int rank, int nranks, ncclComm_t comm,
                              hipStream_t stream,
                              void* const* d_in, const int* in_sizes, int n_in,
                              void* d_out, int out_size,
                              const int* in_dtypes, int out_dtype,
                              void* const* d_out_all,
                              void* const* d_ws_all, size_t ws_bytes) {
  if (nranks != NR || (unsigned)rank >= (unsigned)NR || n_in != 4 || in_sizes[0] != MS * KD || in_sizes[1] != KD * ND ||
      out_size != NR * MS * NS || ws_bytes < WS_END) {
    fprintf(stderr, "kernel_launch: unexpected problem shape\n");
    return;
  }
  HIP_CHECK(hipFuncSetAttribute((const void*)fused_gemm_a2a, hipFuncAttributeMaxDynamicSharedMemorySize, LDS_BYTES));
  Args a;
  a.x = (const unsigned char*)d_in[0];
  a.w = (const unsigned char*)d_in[1];
  a.sx = (const float*)d_in[2];
  a.sw = (const float*)d_in[3];
  a.out = (float*)d_out;
  for (int p = 0; p < NR; ++p) a.ws[p] = (char*)d_ws_all[(rank + p) % NR];
  a.rank = rank;
  a.pad = 0;
  hipLaunchKernelGGL(fused_gemm_a2a, dim3(256), dim3(512), LDS_BYTES, stream, a);
  HIP_CHECK(hipGetLastError());
}
```
